```python
import numpy as np

kernel_name = "gpu_gemm_a2a_m4096_k4096_n16384_bf16_relu"


ABSMAX_RTOL = 0.008

COMPUTE_DTYPE = "bfloat16"
EPILOGUE = "relu"
IN_QUANT = None
OUT_QUANT = None
IN_DTYPES = {"x": "bfloat16", "w_mat": "bfloat16"}
WORLD_SIZE = 8

def _to_bf16(x):
    u = np.ascontiguousarray(x, dtype=np.float32).view(np.uint32)
    return ((u + 0x7FFF + ((u >> 16) & 1)) & 0xFFFF0000).view(np.float32)

def setup_inputs():

    x = _to_bf16(np.random.randn(4096, 4096).astype(np.float32))
    w_mat = _to_bf16((np.random.randn(4096, 16384) / np.sqrt(4096)).astype(np.float32))
    return {"x": x, "w_mat": w_mat}

def _epilogue(y):
    return np.maximum(y, 0.0).astype(np.float32)

def reference(x, w_mat):
    y = (x @ w_mat).astype(np.float32)
    return _epilogue(y)
```

```cpp
#define NEAR
#include <hip/hip_runtime.h>
#include <rccl/rccl.h>
#include <cstdio>
#pragma clang diagnostic ignored "-Wint-to-pointer-cast"

#define CFG_WIN 3
#define CFG_PAG 72
#define CFG_P 72
#define CFG_P2 44
#define CFG_OAUX 2
#define CFG_XMID 8
#define CFG_WMID 16
#define CFG_LSLP 0
#define CFG_SSLP 4
#define CFG_AGC 9
#define CFG_AGAUX 17
#define CFG_N0 9
#define CFG_N1 9
#define CFG_AGS 2
#define CFG_FLI 9
#define CFG_AGCK 6
#define ABL 0

namespace ga {
constexpr int MLOC = 512, KDIM = 4096, NDIM = 16384, NR = 8, NSH = 2048;
constexpr int BM = 128, BN = 256, BK = 64, NS = 3;
constexpr int XB = 16384, STAGE = 3 * XB, STG = NS * STAGE;
constexpr int LDS_BYTES = STG + 16384;
constexpr int NWG = 256, ROUNDS = 4, RSTEPS = 16, TSTEPS = ROUNDS * RSTEPS;
constexpr unsigned X_BYTES = MLOC * KDIM * 2u, W_BYTES = (unsigned)KDIM * NDIM * 2u, OUT_BYTES = 4096u * NSH * 2u;
constexpr unsigned long long DEADLINE = 100000000ull;
constexpr unsigned SPIN_CAP = 1u << 26;
constexpr unsigned WS_DONE = 0, WS_REG = 2048, WS_ARR = 4096, WS_ARR1 = 6144, WS_CNT = 8192, WS_TMO = 16384, WS_AGF = 32768, WS_DONE2 = 49152, WS_SCR = 196608, WS_DBG = 65536, WS_XF = 131072;
constexpr unsigned WS_XD = 1u << 20, XD_WG = 49152u, XD_BUF = NWG * XD_WG;
constexpr unsigned WS_G = 64u << 20, G_SRC = 1u << 20, G_BUF = NR * G_SRC;
constexpr unsigned WS_NEED = WS_G + 2 * G_BUF;
static_assert(WS_XD + ROUNDS * XD_BUF <= WS_G, "ws");
static_assert(LDS_BYTES <= 160 * 1024, "LDS");

typedef __attribute__((address_space(3))) char lds_char;
typedef short s16x4 __attribute__((ext_vector_type(4)));
typedef short s16x2 __attribute__((ext_vector_type(2)));
typedef __bf16 bf16x8 __attribute__((ext_vector_type(8)));
typedef __bf16 bf16x2 __attribute__((ext_vector_type(2)));
typedef float f32x4 __attribute__((ext_vector_type(4)));
typedef float f32x2 __attribute__((ext_vector_type(2)));
typedef unsigned u32x4 __attribute__((ext_vector_type(4)));
typedef __amdgpu_buffer_rsrc_t rsrc_t;
typedef __attribute__((address_space(3))) s16x4* lp4;
typedef const __attribute__((address_space(3))) bf16x8* lp8;
typedef volatile __attribute__((address_space(3))) u32x4* lpq;
typedef volatile __attribute__((address_space(3))) unsigned* lpu;
typedef __attribute__((address_space(1))) unsigned* gptr_u;

struct Params {
  const void* X; const void* W;
  char* out[NR]; char* ws[NR];
  int rank; int mode;
};

#define WAIT_V(n) asm volatile("s_waitcnt vmcnt(%0)" ::"n"(n) : "memory")
#define WAIT_L(n) asm volatile("s_waitcnt lgkmcnt(%0)" ::"n"(n) : "memory")
#define BARRIER() do { asm volatile("" ::: "memory"); __builtin_amdgcn_s_barrier(); asm volatile("" ::: "memory"); } while (0)
#define SCHED_FENCE() __builtin_amdgcn_sched_barrier(0)
#define CFENCE() asm volatile("" ::: "memory")
#define AINL __attribute__((always_inline))
#define PIN(x) asm volatile("" : "+v"(x))
constexpr int AUX_SYS = 17;
constexpr int AUX_OUT = CFG_OAUX;

__device__ __forceinline__ rsrc_t mkrs(const void* p, unsigned bytes) { return __builtin_amdgcn_make_buffer_rsrc((void*)p, 0, bytes, 0x00020000); }

struct Tile { int xsrc;   int xrow; int wcol; int peer; int orow; int ocol; };
static_assert(CFG_N0 % 3 == 0 && CFG_N1 % 3 == 0, "triples");
constexpr int N0 = CFG_N0, N1 = CFG_N1, N2 = 24 - N0 - N1;
__device__ __forceinline__ int nth_bit(unsigned m, int k) { for (int i = 0; i < k; ++i) m &= m - 1; return __builtin_ctz(m); }
__device__ __forceinline__ Tile tile_of(int b, int r, int rank) {
  const int h = (b >> 2) & 1, idx = b >> 3;
  const unsigned myset = ((0xb2u >> rank) & 1u) ? 0xb2u : 0x4du;
  Tile t;
  if (r < 3 && (h || idx < 24)) {
    const int i = h ? r * 32 + idx : r * 24 + idx, tq = i / 3;
    t.xsrc = -1; t.xrow = (i % 3) * BM;
    t.peer = h ? nth_bit(myset ^ 0xffu, tq & 3) : nth_bit(myset & ~(1u << rank), tq % 3);
    t.ocol = (h ? tq >> 2 : tq / 3) * BN; t.orow = rank * MLOC + t.xrow;
  } else {
    const int j = r < 3 ? r * 8 + idx - 24 : 24 + h * 32 + idx;
    if (j < 32) { t.xsrc = -1; t.xrow = (j & 3) * BM; t.ocol = (j >> 2) * BN; t.orow = rank * MLOC + t.xrow; }
    else { const int jj = j - 32; t.xsrc = (rank + 1 + jj % 7) & 7; t.xrow = 0; t.ocol = (jj / 7) * BN; t.orow = t.xsrc * MLOC + 3 * BM; }
    t.peer = rank;
  }
  t.wcol = t.peer * NSH + t.ocol;
  return t;
}
__device__ __forceinline__ unsigned pack2(float a, float b) {
  f32x2 v = {a, b};
  return __builtin_bit_cast(unsigned, __builtin_convertvector(v, bf16x2));
}
__device__ __forceinline__ unsigned relu2(unsigned x) {
  s16x2 s = __builtin_bit_cast(s16x2, x), z = {0, 0};
  return __builtin_bit_cast(unsigned, __builtin_elementwise_max(s, z));
}
__device__ __forceinline__ float unpk(unsigned d, int hi) { return __builtin_bit_cast(float, hi ? (d & 0xffff0000u) : (d << 16)); }
__device__ __forceinline__ int vm_outstanding() { int v; asm volatile("s_getreg_b32 %0, hwreg(HW_REG_IB_STS, 0, 4)" : "=s"(v)); return v; }
template <int N> struct IC { static constexpr int v = N; };

__device__ __forceinline__ bool give_up(char* myws, unsigned call, unsigned long long t0) {
  if (__hip_atomic_load((unsigned*)(myws + WS_TMO), __ATOMIC_RELAXED, __HIP_MEMORY_SCOPE_AGENT) == call ||
      __builtin_amdgcn_s_memrealtime() - t0 > DEADLINE) {
    __hip_atomic_store((unsigned*)(myws + WS_TMO), call, __ATOMIC_RELAXED, __HIP_MEMORY_SCOPE_AGENT);
    return true;
  }
  return false;
}
__global__ void __launch_bounds__(512, 2) gemm_a2a(Params p) {
  __shared__ __attribute__((aligned(1024))) char smem[LDS_BYTES];
  lds_char* lds = (lds_char*)smem;
  const unsigned lbase = (unsigned)(size_t)lds;
  const int tid = threadIdx.x, lane = tid & 63;
  const int wid = __builtin_amdgcn_readfirstlane(tid >> 6);
  const int b = blockIdx.x, q = b & 3, rank = p.rank;
  char* myws = p.ws[rank];
  const unsigned call = *(const unsigned*)(myws + WS_CNT + b * 4) + 1u;
  const unsigned lane16 = lane * 16;

  if (wid >= 4 && wid < 7) {
    const bool isx = wid == 4;
    unsigned v[2];
#pragma unroll
    for (int e = 0; e < 2; ++e) {
      const unsigned vx = (unsigned)(lane >> 3) * (KDIM * 2) + (unsigned)(((lane & 7) ^ ((4 * e + (lane >> 4)) & 7)) * 16);
      const int c16 = lane & 15, f = (lane >> 4) | (e << 2);
      const unsigned vw = (unsigned)(lane >> 4) * (NDIM * 2) + (unsigned)((((c16 >> 1) ^ f) * 32) + (c16 & 1) * 16);
      v[e] = isx ? vx : vw;
    }
    rsrc_t rs = mkrs(p.W, W_BYTES);
    unsigned sbase = 0;
    int slot = 0, ks = 0, rnd = 0;
    auto issue = [&]() AINL {
      if (ks == 0) {
        const Tile t = tile_of(b, rnd, rank);
        if (isx) {
          rs = t.xsrc < 0 ? mkrs(p.X, X_BYTES) : mkrs(myws + WS_G + (call & 1) * G_BUF + t.xsrc * G_SRC, G_SRC);
          sbase = (unsigned)t.xrow * (KDIM * 2) + (unsigned)q * 2048u;
        } else {
          sbase = (unsigned)(q * 1024) * (NDIM * 2) + (unsigned)(t.wcol + (wid - 5) * 128) * 2u;
        }
      }
      lds_char* base = lds + slot * STAGE + (wid - 4) * XB;
      if (isx) {
        const unsigned s = sbase + (unsigned)ks * (BK * 2);
#pragma unroll
        for (int i = 0; i < 16; ++i) {
          __builtin_amdgcn_raw_ptr_buffer_load_lds(rs, base + i * 1024, 16, v[i & 1],
                                                   s + (unsigned)(32 * (((i >> 2) + q) & 3) + 8 * (i & 3)) * (KDIM * 2), 0, 0);
        }
      } else {
        const unsigned s = sbase + (unsigned)ks * (BK * NDIM * 2);
#pragma unroll
        for (int i = 0; i < 16; ++i) {
          __builtin_amdgcn_raw_ptr_buffer_load_lds(rs, base + i * 1024, 16, v[(i >> 1) & 1], s + i * 4 * (NDIM * 2), 0, 0);
        }
      }
      slot = (slot + 1 == NS) ? 0 : slot + 1;
      if (++ks == RSTEPS) { ks = 0; ++rnd; }
    };
#pragma unroll 1
    for (int s = 0; s < NS; ++s) issue();
    WAIT_V(32);
    BARRIER();
#pragma unroll 1
    for (int t = 0; t < TSTEPS - NS; ++t) {
      WAIT_V(16);
      BARRIER();
      issue();
    }
    WAIT_V(16); BARRIER();
    WAIT_V(0); BARRIER();
    BARRIER();
    return;
  }

  if (wid == 7) {
    int t = -1;
    auto step = [&]() AINL { if (t < TSTEPS - 1) { BARRIER(); ++t; } };
    const bool remote = !(p.mode & 1);
    const unsigned st_voff = (unsigned)(lane & 15) * (NSH * 2) + (unsigned)(((lane >> 4) & 1) * 2 + (lane >> 5)) * 16u;
    rsrc_t rxd = mkrs(myws + WS_XD, (ROUNDS - 1) * XD_BUF), rxf = mkrs(myws + WS_XF + b * 64, 48);
    const unsigned goff = (unsigned)b * 4096u;
    size_t fptr = (size_t)((remote && lane < NR && lane != rank) ? p.ws[lane & 7] + (rank * NWG + b) * 4 : myws + WS_SCR - WS_AGF + b * 4);
    u32x4 d[4], ow[16], in[8][3], o, agf;
    unsigned xfv = 0;
    int agn = 0;
    auto ag_store = [&](const u32x4& dd, int j) AINL {
      const int peer = (rank + 1 + (agn + b) % 7) & 7;
      rsrc_t rd = mkrs(p.ws[peer] + WS_G + (call & 1) * G_BUF + rank * G_SRC, (remote && agn < 28) ? G_SRC : 0u);
      __builtin_amdgcn_raw_buffer_store_b128(dd, rd, lane16, goff + j * 1024, AUX_SYS);
      ++agn;
    };
    auto xf_load = [&]() AINL { CFENCE(); xfv = __builtin_amdgcn_raw_buffer_load_b32(rxf, lane < 12 ? lane * 4 : 0, 0, AUX_SYS); CFENCE(); };
    auto xf_check = [&](unsigned seq) AINL {
      if (__builtin_expect(!__all(xfv >= seq), 0)) {
        const unsigned long long t0 = __builtin_amdgcn_s_memrealtime();
        for (;;) {
          xf_load();
          if (__all(xfv >= seq)) break;
          if (give_up(myws, call, t0)) break;
        }
      }
      CFENCE();
    };
    auto in_load = [&](int r, int i) AINL {
#pragma unroll
      for (int k = 0; k < 3; ++k)
        in[i & 7][k] = __builtin_amdgcn_raw_buffer_load_b128(rxd, lane16, r * XD_BUF + (unsigned)b * XD_WG + k * 16384u + i * 1024u, AUX_SYS);
    };
    auto ow_load = [&]() AINL {
#pragma unroll
      for (int i = 0; i < 16; ++i) ow[i] = *(lpq)(lbase + STG + i * 1024 + lane16);
      WAIT_L(0);
    };
    const unsigned seq0 = (call - 1u) * ROUNDS + 1u;
    {
      rsrc_t rx = mkrs(p.X, X_BYTES);
#pragma unroll
      for (int j = 0; j < 4; ++j) d[j] = __builtin_amdgcn_raw_buffer_load_b128(rx, lane16, 3u * BM * KDIM * 2 + goff + j * 1024, 0);
      PIN(fptr);
      WAIT_V(0);
      BARRIER();
#pragma unroll
      for (int j = 0; j < 3; ++j) {
#pragma unroll 1
        for (int k = 0; k < (j < 2 ? 7 : 6); ++k) { ag_store(d[j], j); step(); }
      }
      xf_load(); ag_store(d[2], 2); step();
      xf_check(seq0); in_load(0, 0); ag_store(d[3], 3); step();
      in_load(0, 1); ag_store(d[3], 3); ow_load(); step();
    }
    const int agsrc = remote ? tile_of(b, ROUNDS - 1, rank).xsrc : -1;
#pragma unroll 1
    for (int r = 0; r < ROUNDS - 1; ++r) {
      const Tile tl = tile_of(b, r, rank);
      rsrc_t rd = mkrs(p.out[tl.peer], OUT_BYTES);
      const unsigned so = (unsigned)(tl.orow + 32 * q) * (NSH * 2) + (unsigned)tl.ocol * 2u;
      const bool more = r < ROUNDS - 2;
      const bool agchk = r == 1 && agsrc >= 0;
      rsrc_t raf = mkrs(myws + WS_AGF + (agsrc & 7) * (NWG * 4), agchk ? NWG * 4 : 0);
#pragma unroll
      for (int i = 0; i < 16; ++i) {
        if (i == 13) xf_load();
        if (i == CFG_AGCK - 1) { agf = __builtin_amdgcn_raw_buffer_load_b128(raf, lane16, 0, AUX_SYS); CFENCE(); }
        {
          float v[8];
#define GAP() asm volatile("s_nop 0")
#pragma unroll
          for (int w = 0; w < 8; ++w) {
            float a_ = unpk(ow[i][w >> 1], w & 1); PIN(a_); GAP();
#pragma unroll
            for (int k = 0; k < 3; ++k) { float u_ = unpk(in[i & 7][k][w >> 1], w & 1); PIN(u_); GAP(); a_ += u_; PIN(a_); GAP(); }
            v[w] = a_;
          }
          unsigned d0 = pack2(v[0], v[1]); PIN(d0); GAP(); unsigned d1 = pack2(v[2], v[3]); PIN(d1); GAP();
          unsigned e0 = pack2(v[4], v[5]); PIN(e0); GAP(); unsigned e1 = pack2(v[6], v[7]); PIN(e1); GAP();
          d0 = relu2(d0); PIN(d0); GAP(); d1 = relu2(d1); PIN(d1); GAP(); e0 = relu2(e0); PIN(e0); GAP(); e1 = relu2(e1); PIN(e1); GAP();
          auto s0 = __builtin_amdgcn_permlane16_swap(d0, e0, false, false);
          auto s1 = __builtin_amdgcn_permlane16_swap(d1, e1, false, false);
          o = (u32x4){s0[0], s1[0], s0[1], s1[1]};
          PIN(o);
        }
        if (i == CFG_AGCK) {
          if (__builtin_expect(agchk && !__all(agf[0] >= call && agf[1] >= call && agf[2] >= call && agf[3] >= call), 0)) {
            const unsigned long long t0 = __builtin_amdgcn_s_memrealtime();
            for (;;) {
              CFENCE();
              agf = __builtin_amdgcn_raw_buffer_load_b128(raf, lane16, 0, AUX_SYS);
              if (__all(agf[0] >= call && agf[1] >= call && agf[2] >= call && agf[3] >= call)) break;
              if (give_up(myws, call, t0)) break;
            }
          }
          CFENCE();
        }
        if (i == 14) xf_check(more ? seq0 + r + 1u : 0u);
        if (i < 6) { in_load(r, 2 * i + 2); in_load(r, 2 * i + 3); }
        else if (i < 8) in_load(r, i + 8);
        else if (i >= 14) in_load(r + 1, i - 14);
        CFENCE();
        __builtin_amdgcn_raw_buffer_store_b128(o, rd, st_voff, so + (unsigned)((i >> 1) & 1) * (16 * NSH * 2) + (i >> 2) * 128u + (i & 1) * 64u, AUX_SYS);
        if (i < 5 * CFG_AGS && i % CFG_AGS == 0) ag_store(d[3], 3);
        if (i == 4 * CFG_AGS + 3 && r == 0) {
          WAIT_V(3);
          __hip_atomic_store((gptr_u)(fptr + WS_AGF), call, __ATOMIC_RELAXED, __HIP_MEMORY_SCOPE_SYSTEM);
        }
        if (i == 15 && more) ow_load();
        SCHED_FENCE(); asm volatile("s_nop 1" ::: "memory"); SCHED_FENCE();
        step();
      }
    }
    WAIT_V(0);
    __hip_atomic_store((gptr_u)(fptr + WS_DONE2), call, __ATOMIC_RELAXED, __HIP_MEMORY_SCOPE_SYSTEM);
    return;
  }

  const int c = lane & 15, g = lane >> 4;
  unsigned ax0 = lbase + c * 128 + ((g ^ (c >> 1)) * 16), ax1 = 0, aw[4];
  {
    const int qq = c >> 2, pp = c & 3;
#pragma unroll
    for (int jn = 0; jn < 4; ++jn)
      aw[jn] = lbase + XB * (1 + (wid >> 1)) + (8 * g + qq) * 256 + (((4 * (wid & 1) + jn) ^ (qq | ((g & 1) << 2))) * 32) + 8 * pp;
  }
  f32x4 acc[8][4];
  bf16x8 x0[8], x1[8], wa[4], wb[4];
  u32x4 rb[13];
  unsigned flagv = 0;
#pragma unroll
  for (int mi = 0; mi < 8; ++mi)
#pragma unroll
    for (int jn = 0; jn < 4; ++jn) acc[mi][jn] = (f32x4){0.f, 0.f, 0.f, 0.f};
#pragma unroll
  for (int j = 0; j < 13; ++j) rb[j] = (u32x4){0u, 0u, 0u, 0u};

  s16x4 wlo;
#define RD_WL(KK, jn) wlo = __builtin_amdgcn_ds_read_tr16_b64_v4i16((lp4)(aw[jn] + (KK) * 8192))
#define RD_WH(D, KK, jn) D[jn] = __builtin_bit_cast(bf16x8, __builtin_shufflevector(wlo, (s16x4)__builtin_amdgcn_ds_read_tr16_b64_v4i16((lp4)(aw[jn] + (KK) * 8192 + 1024)), 0, 1, 2, 3, 4, 5, 6, 7))
#define RD_X0(mi) x0[mi] = *(lp8)(ax0 + (mi) * 2048)
#define RD_X1(mi) x1[mi] = *(lp8)(ax1 + (mi) * 2048)
#define MFMA(ACC, A, B) asm volatile("v_mfma_f32_16x16x32_bf16 %0, %1, %2, %0" : "+v"(ACC) : "v"(A), "v"(B))
#define MFMA0(ACC, A, B) asm volatile("v_mfma_f32_16x16x32_bf16 %0, %1, %2, 0" : "+v"(ACC) : "v"(A), "v"(B))
#define SLOT(G, S, mi, XV, WV, ZZ, FILL) do { \
    if constexpr (ZZ) MFMA0(acc[mi][S], WV[S], XV[mi]); else MFMA(acc[mi][S], WV[S], XV[mi]); \
    SCHED_FENCE(); FILL; extra(KIND_, IC<G>{}, IC<S>{}); SCHED_FENCE(); } while (0)
#define GROUP(G, mi, XV, WV, ZZ, F0, F1, F2, F3) do { \
    SLOT(G, 0, mi, XV, WV, ZZ, F0); SLOT(G, 1, mi, XV, WV, ZZ, F1); SLOT(G, 2, mi, XV, WV, ZZ, F2); SLOT(G, 3, mi, XV, WV, ZZ, F3); } while (0)
#define NOP_ (void)0
#define STPAD() do { SCHED_FENCE(); asm volatile("s_nop 1"); SCHED_FENCE(); } while (0)

  const int gb = b & ~3;
  rsrc_t rxd = mkrs(myws + WS_XD, 0);
  rsrc_t rxf = mkrs(myws + WS_XF, 0);
  unsigned xd_buf = 0, seq = 0;
  const unsigned stg_addr = lbase + STG + wid * 4096 + lane16;
  auto opaque_l16 = [&]() AINL { unsigned l = lane16; PIN(l); return l; };
  auto pack_part = [&](auto N_) AINL -> u32x4 {
    constexpr int n = decltype(N_)::v, mi = n >> 1, h = n & 1;
    const f32x4 v0 = acc[mi][2 * h], v1 = acc[mi][2 * h + 1];
    return (u32x4){pack2(v0[0], v0[1]), pack2(v0[2], v0[3]), pack2(v1[0], v1[1]), pack2(v1[2], v1[3])};
  };
  auto store_part = [&](auto N_, u32x4 d) AINL {
    constexpr int n = decltype(N_)::v, u = n >> 2;
    const unsigned so = xd_buf + (unsigned)(gb + ((q + u) & 3)) * XD_WG + (unsigned)((u - 1) * 4 + wid) * 4096u + (n & 3) * 1024u;
    __builtin_amdgcn_raw_buffer_store_b128(d, rxd, lane16, so, AUX_SYS);
    SCHED_FENCE(); asm volatile("s_nop 0");
  };
  auto load_part = [&](int slot_, int j) AINL -> u32x4 {
    return __builtin_amdgcn_raw_buffer_load_b128(rxd, lane16, xd_buf + (unsigned)b * XD_WG + (unsigned)(slot_ * 4 + wid) * 4096u + j * 1024u, AUX_SYS);
  };
  auto stage_own = [&](auto J_) AINL { *(lpq)(stg_addr + decltype(J_)::v * 1024) = pack_part(J_); };
  auto send_flags = [&]() AINL {
    WAIT_V(0);
    const unsigned l = opaque_l16() >> 4;
    const unsigned vo = (l >= 1 && l < 4) ? (((q + l) & 3) * 16 + (l - 1) * 4) * 4 : 0x80000000u;
    __builtin_amdgcn_raw_buffer_store_b32(seq, rxf, vo, (unsigned)(gb * 16 + wid) * 4u, AUX_SYS);
  };
  auto load_flags = [&]() AINL {
    const unsigned l = opaque_l16();
    flagv = __builtin_amdgcn_raw_buffer_load_b32(rxf, l < 32u ? l : 32u, (unsigned)(b * 16 + wid) * 4u, AUX_SYS);
  };

  int slot = 0;
  auto extra = [&](auto KIND_, auto G_, auto S_) AINL {
    constexpr int KIND = decltype(KIND_)::v, G = decltype(G_)::v, S = decltype(S_)::v, SL = G * 4 + S;
#define PK(sl, n) if constexpr (SL == sl) { rb[n - 3] = pack_part(IC<n>{}); PIN(rb[n - 3]); }
#define ST(sl, n) if constexpr (SL == sl) store_part(IC<n>{}, rb[n - 3]);
    if constexpr (KIND == 5 && !(ABL & 1)) {
      PK(39, 4) ST(41, 4) PK(42, 5) PK(43, 6) PK(47, 7) ST(49, 5) ST(57, 6)
      if constexpr (SL == 62) stage_own(IC<0>{});
      if constexpr (SL == 63) stage_own(IC<1>{});
    }
    if constexpr (KIND == 1 && !(ABL & 1)) {
      if constexpr (SL == 2) stage_own(IC<2>{});
      if constexpr (SL == 3) stage_own(IC<3>{});
      ST(1, 7) PK(5, 8) PK(6, 9) PK(7, 10) ST(9, 8) PK(11, 11) PK(15, 12) ST(17, 9) PK(19, 13) PK(23, 14) ST(25, 10) PK(26, 15)
      ST(33, 11) ST(41, 12) ST(49, 13) ST(56, 14) ST(63, 15)
    }
  };
  auto kstep = [&](auto KIND_) AINL {
    constexpr int KIND = decltype(KIND_)::v;
    constexpr bool Z = KIND == 1;
    slot = (slot + 1 == NS) ? 0 : slot + 1;
    const unsigned dlt = slot == 0 ? (unsigned)(-(NS - 1) * STAGE) : (unsigned)STAGE;
    if constexpr (Z) asm volatile("s_nop 3");
    SCHED_FENCE();
    GROUP(0, 0, x0, wa, Z, RD_X0(3), NOP_, NOP_, NOP_);
    GROUP(1, 1, x0, wa, Z, RD_X0(4), NOP_, NOP_, NOP_);
    GROUP(2, 2, x0, wa, Z, RD_X0(5), RD_WL(1, 0), RD_WH(wb, 1, 0), NOP_);
    GROUP(3, 3, x0, wa, Z, RD_X0(6), RD_WL(1, 1), RD_WH(wb, 1, 1), NOP_);
    GROUP(4, 4, x0, wa, Z, RD_X0(7), RD_WL(1, 2), RD_WH(wb, 1, 2), ax1 = ax0 ^ 64u; PIN(ax1));
    GROUP(5, 5, x0, wa, Z, RD_X1(2), RD_WL(1, 3), RD_WH(wb, 1, 3), NOP_);
    GROUP(6, 6, x0, wa, Z, RD_X1(3), NOP_, NOP_, NOP_);
    GROUP(7, 7, x0, wa, Z, RD_X1(4), NOP_, RD_X1(5), NOP_);
    GROUP(8, 2, x1, wb, false, RD_X1(6), NOP_, NOP_, NOP_);
    GROUP(9, 3, x1, wb, false, RD_X1(7), NOP_, NOP_, NOP_);
    GROUP(10, 4, x1, wb, false, RD_X1(0), NOP_, NOP_, NOP_);
    GROUP(11, 5, x1, wb, false, RD_X1(1), NOP_, NOP_, ax0 += dlt; PIN(ax0));
    GROUP(12, 6, x1, wb, false, aw[0] += dlt; PIN(aw[0]), aw[1] += dlt; PIN(aw[1]), aw[2] += dlt; PIN(aw[2]), aw[3] += dlt; PIN(aw[3]));
    WAIT_L(0);
    BARRIER();
    RD_X0(0);
    SCHED_FENCE();
    GROUP(13, 7, x1, wb, false, RD_WL(0, 0), RD_WH(wa, 0, 0), RD_WL(0, 1), RD_WH(wa, 0, 1));
    GROUP(14, 0, x1, wb, false, RD_WL(0, 2), RD_WH(wa, 0, 2), RD_WL(0, 3), RD_WH(wa, 0, 3));
    GROUP(15, 1, x1, wb, false, RD_X0(1), RD_X0(2), NOP_, NOP_);
  };

#define FSTAMP()
#define STAMP()
  BARRIER();
  STAMP();
  FSTAMP();
  RD_X0(0);
#pragma unroll
  for (int jn = 0; jn < 4; ++jn) { RD_WL(0, jn); RD_WH(wa, 0, jn); }
  RD_X0(1); RD_X0(2);
  SCHED_FENCE();
#pragma unroll 1
  for (int r = 0; r < ROUNDS; ++r) {
    kstep(IC<1>{}); FSTAMP();
    kstep(IC<0>{}); FSTAMP();
    send_flags();
#define KS0() do { kstep(IC<0>{}); FSTAMP(); } while (0)
    KS0(); KS0(); KS0(); KS0(); KS0(); KS0(); KS0(); KS0(); KS0(); KS0(); KS0(); KS0(); KS0();
    rxd = mkrs(myws + WS_XD, ROUNDS * XD_BUF);
    rxf = mkrs(myws + WS_XF, NWG * 64);
    xd_buf = r * XD_BUF;
    seq = (call - 1u) * ROUNDS + r + 1u;
    kstep(IC<5>{}); FSTAMP();
    STAMP();
  }
  {
    SCHED_FENCE();
    store_part(IC<7>{}, rb[4]); STPAD();
#define SP(n) store_part(IC<n>{}, pack_part(IC<n>{})); STPAD()
    SP(8); SP(9); SP(10); SP(11); SP(12); SP(13); SP(14); SP(15);
    FSTAMP();
    send_flags();
    FSTAMP();
    {
      const unsigned long long t0 = __builtin_amdgcn_s_memrealtime();
      for (;;) {
        CFENCE();
        load_flags();
        if (__all(flagv >= seq)) break;
        if (give_up(myws, call, t0)) break;
      }
      CFENCE();
    }
    FSTAMP();
    u32x4 r3[3][4];
#pragma unroll
    for (int s = 0; s < 3; ++s)
#pragma unroll
      for (int j = 0; j < 4; ++j) r3[s][j] = load_part(s, j);
    const bool bad = __hip_atomic_load((unsigned*)(myws + WS_TMO), __ATOMIC_RELAXED, __HIP_MEMORY_SCOPE_AGENT) == call;
    const Tile tl = tile_of(b, ROUNDS - 1, rank);
    rsrc_t rd = mkrs(p.out[tl.peer], OUT_BYTES);
    const unsigned so = (unsigned)(tl.orow + 32 * q) * (NSH * 2) + (unsigned)tl.ocol * 2u;
    const unsigned l = opaque_l16() >> 4, gg = l >> 4;
    const unsigned vo = (l & 15) * (NSH * 2) + (8 * wid + (gg & 1) * 2 + (gg >> 1)) * 16u;
#pragma unroll
    for (int j = 0; j < 4; ++j) {
      f32x4 v[2] = {acc[j >> 1][2 * (j & 1)], acc[j >> 1][2 * (j & 1) + 1]};
#pragma unroll
      for (int s = 0; s < 3; ++s)
#pragma unroll
        for (int w = 0; w < 8; ++w) v[w >> 2][w & 3] += unpk(r3[s][j][w >> 1], w & 1);
      const unsigned d0 = relu2(pack2(v[0][0], v[0][1])), d1 = relu2(pack2(v[0][2], v[0][3]));
      const unsigned e0 = relu2(pack2(v[1][0], v[1][1])), e1 = relu2(pack2(v[1][2], v[1][3]));
      auto s0 = __builtin_amdgcn_permlane16_swap(d0, e0, false, false);
      auto s1 = __builtin_amdgcn_permlane16_swap(d1, e1, false, false);
      u32x4 o = {s0[0], s1[0], s0[1], s1[1]};
      if (bad) o = (u32x4){0x7fc07fc0u, 0x7fc07fc0u, 0x7fc07fc0u, 0x7fc07fc0u};
      __builtin_amdgcn_raw_buffer_store_b128(o, rd, vo, so + (j >> 1) * (16 * NSH * 2) + (j & 1) * 64u, 0);
      STPAD();
    }
    FSTAMP();
  }
  STAMP();
  FSTAMP();
  FSTAMP();
  STAMP();
  STAMP();
  if (wid == 0) {
    if (lane == 0) __hip_atomic_store((unsigned*)(myws + WS_CNT) + b, call, __ATOMIC_RELAXED, __HIP_MEMORY_SCOPE_AGENT);
  } else if (wid == 1 && b < NR - 1 && !(p.mode & 1)) {
    rsrc_t frs2 = mkrs(myws + WS_DONE2 + ((rank + 1 + b) & 7) * (NWG * 4), NWG * 4);
    const unsigned long long t0 = __builtin_amdgcn_s_memrealtime();
    for (;;) {
      CFENCE();
      const u32x4 f = __builtin_amdgcn_raw_buffer_load_b128(frs2, lane16, 0, AUX_SYS);
      const bool ok = f[0] >= call && f[1] >= call && f[2] >= call && f[3] >= call;
      if (__all(ok)) break;
      if (give_up(myws, call, t0)) {
        if (lane == 0) *(volatile unsigned short*)(p.out[rank] + (size_t)rank * MLOC * NSH * 2) = 0x7fc0;
        break;
      }
    }
  }
}
}

extern "C" void kernel_launch(int rank, int nranks, ncclComm_t comm, hipStream_t stream,
                              void* const* d_in, const int* in_sizes, int n_in,
                              void* d_out, int out_size, const int* in_dtypes, int out_dtype,
                              void* const* d_out_all, void* const* d_ws_all, size_t ws_bytes) {
  if (nranks != ga::NR || ws_bytes < ga::WS_NEED) return;
  ga::Params p;
  p.X = d_in[0]; p.W = d_in[1];
  for (int i = 0; i < ga::NR; ++i) { p.out[i] = (char*)d_out_all[i]; p.ws[i] = (char*)d_ws_all[i]; }
  p.rank = rank; p.mode = 0;
  hipLaunchKernelGGL(ga::gemm_a2a, dim3(ga::NWG), dim3(512), 0, stream, p);
}
```

```cpp
#define ROT
#include <hip/hip_runtime.h>
#include <rccl/rccl.h>
#include <cstdio>
#pragma clang diagnostic ignored "-Wint-to-pointer-cast"

#define CFG_WIN 3
#define CFG_PAG 72
#define CFG_P 72
#define CFG_P2 44
#define CFG_OAUX 2
#define CFG_XMID 8
#define CFG_WMID 16
#define CFG_LSLP 0
#define CFG_SSLP 4
#define CFG_AGC 9
#define CFG_AGAUX 17
#define CFG_N0 9
#define CFG_N1 9
#define CFG_AGS 2
#define CFG_FLI 9
#define CFG_AGCK 6
#define CFG_RB 3
#define CFG_WAUX 0
#define CFG_XM 3
#define CFG_PH 1
#define ABL 0

namespace ga {
constexpr int MLOC = 512, KDIM = 4096, NDIM = 16384, NR = 8, NSH = 2048;
constexpr int BM = 128, BN = 256, BK = 64, NS = 3;
constexpr int XB = 16384, STAGE = 3 * XB, STG = NS * STAGE;
constexpr int LDS_BYTES = STG + 16384;
constexpr int NWG = 256, ROUNDS = 4, RSTEPS = 16, TSTEPS = ROUNDS * RSTEPS;
constexpr unsigned X_BYTES = MLOC * KDIM * 2u, W_BYTES = (unsigned)KDIM * NDIM * 2u, OUT_BYTES = 4096u * NSH * 2u;
constexpr unsigned long long DEADLINE = 100000000ull;
constexpr unsigned SPIN_CAP = 1u << 26;
constexpr unsigned WS_DONE = 0, WS_REG = 2048, WS_ARR = 4096, WS_ARR1 = 6144, WS_CNT = 8192, WS_TMO = 16384, WS_AGF = 32768, WS_DONE2 = 49152, WS_SCR = 196608, WS_DBG = 65536, WS_XF = 131072;
constexpr unsigned WS_XD = 1u << 20, XD_WG = 49152u, XD_BUF = NWG * XD_WG;
constexpr unsigned WS_G = 64u << 20, G_SRC = 1u << 20, G_BUF = NR * G_SRC;
constexpr unsigned WS_NEED = WS_G + 2 * G_BUF;
static_assert(WS_XD + ROUNDS * XD_BUF <= WS_G, "ws");
static_assert(LDS_BYTES <= 160 * 1024, "LDS");

typedef __attribute__((address_space(3))) char lds_char;
typedef short s16x4 __attribute__((ext_vector_type(4)));
typedef short s16x2 __attribute__((ext_vector_type(2)));
typedef __bf16 bf16x8 __attribute__((ext_vector_type(8)));
typedef __bf16 bf16x2 __attribute__((ext_vector_type(2)));
typedef float f32x4 __attribute__((ext_vector_type(4)));
typedef float f32x2 __attribute__((ext_vector_type(2)));
typedef unsigned u32x4 __attribute__((ext_vector_type(4)));
typedef __amdgpu_buffer_rsrc_t rsrc_t;
typedef __attribute__((address_space(3))) s16x4* lp4;
typedef const __attribute__((address_space(3))) bf16x8* lp8;
typedef volatile __attribute__((address_space(3))) u32x4* lpq;
typedef volatile __attribute__((address_space(3))) unsigned* lpu;
typedef __attribute__((address_space(1))) unsigned* gptr_u;

struct Params {
  const void* X; const void* W;
  char* out[NR]; char* ws[NR];
  int rank; int mode;
};

#define WAIT_V(n) asm volatile("s_waitcnt vmcnt(%0)" ::"n"(n) : "memory")
#define WAIT_L(n) asm volatile("s_waitcnt lgkmcnt(%0)" ::"n"(n) : "memory")
#define BARRIER() do { asm volatile("" ::: "memory"); __builtin_amdgcn_s_barrier(); asm volatile("" ::: "memory"); } while (0)
#define SCHED_FENCE() __builtin_amdgcn_sched_barrier(0)
#define CFENCE() asm volatile("" ::: "memory")
#define AINL __attribute__((always_inline))
#define PIN(x) asm volatile("" : "+v"(x))
constexpr int AUX_SYS = 17;
constexpr int AUX_OUT = CFG_OAUX;

__device__ __forceinline__ rsrc_t mkrs(const void* p, unsigned bytes) { return __builtin_amdgcn_make_buffer_rsrc((void*)p, 0, bytes, 0x00020000); }

struct Tile { int xsrc;   int xrow; int wcol; int peer; int orow; int ocol; int rot; int blk; int ph;   };
__device__ __forceinline__ int nth_bit(unsigned m, int k) { for (int i = 0; i < k; ++i) m &= m - 1; return __builtin_ctz(m); }
__device__ __forceinline__ Tile tile_of(int b, int r, int rank) {
  const int h = (b >> 2) & 1, idx = b >> 3;
  const int nt = r < 2 ? 10 : r == 2 ? 8 : 0;
  Tile t;
  t.xsrc = -1;
  if (idx < 3 * nt) {
    const int rt = idx % 3;
    const int tq = r * 20 + h * nt + idx / 3;
    t.peer = (rank + 1 + tq % 7) & 7; t.ocol = (tq / 7) * BN;
    t.xrow = rt * BM; t.orow = rank * MLOC + t.xrow; t.rot = rt * (CFG_RB / 3); t.blk = CFG_RB; t.ph = CFG_PH ? ((idx / 3 + h) % 3) * (CFG_RB / 3) : 0;
  } else {
    const int j = idx - 3 * nt;
    t.peer = rank; t.ph = 0;
    if (r < 2) { t.xrow = (2 * r + j) * BM; t.ocol = h * BN; t.rot = j; t.blk = 2; }
    else if (r == 2 || (h == 0 && j < 8)) { t.xrow = (j & 3) * BM; t.ocol = ((r == 2 ? 2 + 2 * h : 6) + (j >> 2)) * BN; t.rot = j & 3; t.blk = 4; }
    else { const int jj = h * 32 + j - 8; t.xsrc = (rank + 1 + jj % 7) & 7; t.xrow = 0; t.ocol = (jj / 7) * BN; t.rot = jj % 7; t.blk = 8; }
    t.orow = t.xsrc < 0 ? rank * MLOC + t.xrow : t.xsrc * MLOC + 3 * BM;
  }
  t.wcol = t.peer * NSH + t.ocol;
  return t;
}

__device__ __forceinline__ unsigned pack2(float a, float b) {
  f32x2 v = {a, b};
  return __builtin_bit_cast(unsigned, __builtin_convertvector(v, bf16x2));
}
__device__ __forceinline__ unsigned relu2(unsigned x) {
  s16x2 s = __builtin_bit_cast(s16x2, x), z = {0, 0};
  return __builtin_bit_cast(unsigned, __builtin_elementwise_max(s, z));
}
__device__ __forceinline__ float unpk(unsigned d, int hi) { return __builtin_bit_cast(float, hi ? (d & 0xffff0000u) : (d << 16)); }
__device__ __forceinline__ int vm_outstanding() { int v; asm volatile("s_getreg_b32 %0, hwreg(HW_REG_IB_STS, 0, 4)" : "=s"(v)); return v; }
template <int N> struct IC { static constexpr int v = N; };

__device__ __forceinline__ bool give_up(char* myws, unsigned call, unsigned long long t0) {
  if (__hip_atomic_load((unsigned*)(myws + WS_TMO), __ATOMIC_RELAXED, __HIP_MEMORY_SCOPE_AGENT) == call ||
      __builtin_amdgcn_s_memrealtime() - t0 > DEADLINE) {
    __hip_atomic_store((unsigned*)(myws + WS_TMO), call, __ATOMIC_RELAXED, __HIP_MEMORY_SCOPE_AGENT);
    return true;
  }
  return false;
}
__global__ void __launch_bounds__(512, 2) gemm_a2a(Params p) {
  __shared__ __attribute__((aligned(1024))) char smem[LDS_BYTES];
  lds_char* lds = (lds_char*)smem;
  const unsigned lbase = (unsigned)(size_t)lds;
  const int tid = threadIdx.x, lane = tid & 63;
  const int wid = __builtin_amdgcn_readfirstlane(tid >> 6);
  const int b = blockIdx.x, q = b & 3, rank = p.rank;
  char* myws = p.ws[rank];
  const unsigned call = *(const unsigned*)(myws + WS_CNT + b * 4) + 1u;
  const unsigned lane16 = lane * 16;

  if (wid >= 4 && wid < 7) {
    const bool isx = wid == 4;
    unsigned v[2];
#pragma unroll
    for (int e = 0; e < 2; ++e) {
      const unsigned vx = (unsigned)(lane >> 3) * (KDIM * 2) + (unsigned)(((lane & 7) ^ ((4 * e + (lane >> 4)) & 7)) * 16);
      const int c16 = lane & 15, f = (lane >> 4) | (e << 2);
      const unsigned vw = (unsigned)(lane >> 4) * (NDIM * 2) + (unsigned)((((c16 >> 1) ^ f) * 32) + (c16 & 1) * 16);
      v[e] = isx ? vx : vw;
    }
    rsrc_t rs = mkrs(p.W, W_BYTES);
    unsigned sbase = 0;
    int slot = 0, ks = 0, rnd = 0, kb = 0, ki = 0, blk = 1, cur = 1, rot = 0, nleft = 0;
    auto issue = [&]() AINL {
      if (ks == 0) {
        const Tile t = tile_of(b, rnd, rank);
        blk = t.blk; rot = t.rot; cur = t.ph ? t.ph : blk;
        kb = 0; nleft = cur; ki = rot * cur / blk;
        if (isx) {
          rs = t.xsrc < 0 ? mkrs(p.X, X_BYTES) : mkrs(myws + WS_G + (call & 1) * G_BUF + t.xsrc * G_SRC, G_SRC);
          sbase = (unsigned)t.xrow * (KDIM * 2) + (unsigned)q * 2048u;
        } else {
          sbase = (unsigned)(q * 1024) * (NDIM * 2) + (unsigned)(t.wcol + (wid - 5) * 128) * 2u;
        }
      }
      const int kse = kb + ki;
      if (++ki == cur) ki = 0;
      if (--nleft == 0) { kb += cur; cur = RSTEPS - kb < blk ? RSTEPS - kb : blk; nleft = cur; ki = cur == blk ? rot : rot * cur / blk; }
      lds_char* base = lds + slot * STAGE + (wid - 4) * XB;
      if (isx) {
        const unsigned s = sbase + (unsigned)kse * (BK * 2);
#pragma unroll
        for (int i = 0; i < 16; ++i) {
          __builtin_amdgcn_raw_ptr_buffer_load_lds(rs, base + i * 1024, 16, v[i & 1],
                                                   s + (unsigned)(32 * (((i >> 2) + q) & 3) + 8 * (i & 3)) * (KDIM * 2), 0, 0);
        }
      } else {
        const unsigned s = sbase + (unsigned)kse * (BK * NDIM * 2);
#pragma unroll
        for (int i = 0; i < 16; ++i) {
          __builtin_amdgcn_raw_ptr_buffer_load_lds(rs, base + i * 1024, 16, v[(i >> 1) & 1], s + i * 4 * (NDIM * 2), 0, CFG_WAUX);
        }
      }
      slot = (slot + 1 == NS) ? 0 : slot + 1;
      if (++ks == RSTEPS) { ks = 0; ++rnd; }
    };
#pragma unroll 1
    for (int s = 0; s < NS; ++s) issue();
    WAIT_V(32);
    BARRIER();
#pragma unroll 1
    for (int t = 0; t < TSTEPS - NS; ++t) {
      WAIT_V(16);
      BARRIER();
      issue();
    }
    WAIT_V(16); BARRIER();
    WAIT_V(0); BARRIER();
    BARRIER();
    return;
  }

  if (wid == 7) {
    int t = -1;
    auto step = [&]() AINL { if (t < TSTEPS - 1) { BARRIER(); ++t; } };
    const bool remote = !(p.mode & 1);
    const unsigned st_voff = (unsigned)(lane & 15) * (NSH * 2) + (unsigned)(((lane >> 4) & 1) * 2 + (lane >> 5)) * 16u;
    rsrc_t rxd = mkrs(myws + WS_XD, ROUNDS * XD_BUF), rxf = mkrs(myws + WS_XF + b * 64, 48);
    const unsigned goff = (unsigned)b * 4096u;
    size_t fptr = (size_t)((remote && lane < NR && lane != rank) ? p.ws[lane & 7] + (rank * NWG + b) * 4 : myws + WS_SCR - WS_AGF + b * 4);
    u32x4 d[4], ow[16], in[8][3], o, agf;
    unsigned xfv = 0;
    int agn = 0;
    auto ag_store = [&](const u32x4& dd, int j) AINL {
      const int peer = (rank + 1 + (agn + b) % 7) & 7;
      rsrc_t rd = mkrs(p.ws[peer] + WS_G + (call & 1) * G_BUF + rank * G_SRC, (remote && agn < 28) ? G_SRC : 0u);
      __builtin_amdgcn_raw_buffer_store_b128(dd, rd, lane16, goff + j * 1024, AUX_SYS);
      ++agn;
    };
    auto xf_load = [&]() AINL { CFENCE(); xfv = __builtin_amdgcn_raw_buffer_load_b32(rxf, lane < 12 ? lane * 4 : 0, 0, AUX_SYS); CFENCE(); };
    auto xf_check = [&](unsigned seq) AINL {
      if (__builtin_expect(!__all(xfv >= seq), 0)) {
        const unsigned long long t0 = __builtin_amdgcn_s_memrealtime();
        for (;;) {
          xf_load();
          if (__all(xfv >= seq)) break;
          if (give_up(myws, call, t0)) break;
        }
      }
      CFENCE();
    };
    auto in_load = [&](int r, int i) AINL {
#pragma unroll
      for (int k = 0; k < 3; ++k)
        in[i & 7][k] = __builtin_amdgcn_raw_buffer_load_b128(rxd, lane16, (r < ROUNDS - 1 ? (r & CFG_XM) * XD_BUF : 0x80000000u) + (unsigned)b * XD_WG + k * 16384u + i * 1024u, AUX_SYS);
    };
    auto ow_load = [&]() AINL {
#pragma unroll
      for (int i = 0; i < 16; ++i) ow[i] = *(lpq)(lbase + STG + i * 1024 + lane16);
      WAIT_L(0);
    };
    const unsigned seq0 = (call - 1u) * ROUNDS + 1u;
    {
      rsrc_t rx = mkrs(p.X, X_BYTES);
#pragma unroll
      for (int j = 0; j < 4; ++j) d[j] = __builtin_amdgcn_raw_buffer_load_b128(rx, lane16, 3u * BM * KDIM * 2 + goff + j * 1024, 0);
      PIN(fptr);
      WAIT_V(0);
      BARRIER();
#pragma unroll
      for (int j = 0; j < 3; ++j) {
#pragma unroll 1
        for (int k = 0; k < (j < 2 ? 7 : 6); ++k) { ag_store(d[j], j); step(); }
      }
      xf_load(); ag_store(d[2], 2); step();
      xf_check(seq0); in_load(0, 0); ag_store(d[3], 3); step();
      in_load(0, 1); ag_store(d[3], 3); ow_load(); step();
    }
    const int agsrc = remote ? tile_of(b, ROUNDS - 1, rank).xsrc : -1;
#pragma unroll 1
    for (int r = 0; r < ROUNDS - 1; ++r) {
      const Tile tl = tile_of(b, r, rank);
      rsrc_t rd = mkrs(p.out[tl.peer], OUT_BYTES);
      const unsigned so = (unsigned)(tl.orow + 32 * q) * (NSH * 2) + (unsigned)tl.ocol * 2u;
      const bool more = r < ROUNDS - 2;
      const bool agchk = r == 1 && agsrc >= 0;
      rsrc_t raf = mkrs(myws + WS_AGF + (agsrc & 7) * (NWG * 4), agchk ? NWG * 4 : 0);
#pragma unroll
      for (int i = 0; i < 16; ++i) {
        if (i == 13) xf_load();
        if (i == CFG_AGCK - 1) { agf = __builtin_amdgcn_raw_buffer_load_b128(raf, lane16, 0, AUX_SYS); CFENCE(); }
        {
          float v[8];
#define GAP() asm volatile("s_nop 0")
#pragma unroll
          for (int w = 0; w < 8; ++w) {
            float a_ = unpk(ow[i][w >> 1], w & 1); PIN(a_); GAP();
#pragma unroll
            for (int k = 0; k < 3; ++k) { float u_ = unpk(in[i & 7][k][w >> 1], w & 1); PIN(u_); GAP(); a_ += u_; PIN(a_); GAP(); }
            v[w] = a_;
          }
          unsigned d0 = pack2(v[0], v[1]); PIN(d0); GAP(); unsigned d1 = pack2(v[2], v[3]); PIN(d1); GAP();
          unsigned e0 = pack2(v[4], v[5]); PIN(e0); GAP(); unsigned e1 = pack2(v[6], v[7]); PIN(e1); GAP();
          d0 = relu2(d0); PIN(d0); GAP(); d1 = relu2(d1); PIN(d1); GAP(); e0 = relu2(e0); PIN(e0); GAP(); e1 = relu2(e1); PIN(e1); GAP();
          auto s0 = __builtin_amdgcn_permlane16_swap(d0, e0, false, false);
          auto s1 = __builtin_amdgcn_permlane16_swap(d1, e1, false, false);
          o = (u32x4){s0[0], s1[0], s0[1], s1[1]};
          PIN(o);
        }
        if (i == CFG_AGCK) {
          if (__builtin_expect(agchk && !__all(agf[0] >= call && agf[1] >= call && agf[2] >= call && agf[3] >= call), 0)) {
            const unsigned long long t0 = __builtin_amdgcn_s_memrealtime();
            for (;;) {
              CFENCE();
              agf = __builtin_amdgcn_raw_buffer_load_b128(raf, lane16, 0, AUX_SYS);
              if (__all(agf[0] >= call && agf[1] >= call && agf[2] >= call && agf[3] >= call)) break;
              if (give_up(myws, call, t0)) break;
            }
          }
          CFENCE();
        }
        if (i == 14) xf_check(more ? seq0 + r + 1u : 0u);
        if (i < 6) { in_load(r, 2 * i + 2); in_load(r, 2 * i + 3); }
        else if (i < 8) in_load(r, i + 8);
        else if (i >= 14) in_load(r + 1, i - 14);
        CFENCE();
        __builtin_amdgcn_raw_buffer_store_b128(o, rd, st_voff, so + (unsigned)((i >> 1) & 1) * (16 * NSH * 2) + (i >> 2) * 128u + (i & 1) * 64u, AUX_SYS);
        if (i < 5 * CFG_AGS && i % CFG_AGS == 0) ag_store(d[3], 3);
        if (i == 4 * CFG_AGS + 3 && r == 0) {
          WAIT_V(3);
          __hip_atomic_store((gptr_u)(fptr + WS_AGF), call, __ATOMIC_RELAXED, __HIP_MEMORY_SCOPE_SYSTEM);
        }
        if (i == 15 && more) ow_load();
        SCHED_FENCE(); asm volatile("s_nop 1" ::: "memory"); SCHED_FENCE();
        step();
      }
    }
    WAIT_V(0);
    __hip_atomic_store((gptr_u)(fptr + WS_DONE2), call, __ATOMIC_RELAXED, __HIP_MEMORY_SCOPE_SYSTEM);
    return;
  }

  const int c = lane & 15, g = lane >> 4;
  unsigned ax0 = lbase + c * 128 + ((g ^ (c >> 1)) * 16), ax1 = 0, aw[4];
  {
    const int qq = c >> 2, pp = c & 3;
#pragma unroll
    for (int jn = 0; jn < 4; ++jn)
      aw[jn] = lbase + XB * (1 + (wid >> 1)) + (8 * g + qq) * 256 + (((4 * (wid & 1) + jn) ^ (qq | ((g & 1) << 2))) * 32) + 8 * pp;
  }
  f32x4 acc[8][4];
  bf16x8 x0[8], x1[8], wa[4], wb[4];
  u32x4 rb[13];
  unsigned flagv = 0;
#pragma unroll
  for (int mi = 0; mi < 8; ++mi)
#pragma unroll
    for (int jn = 0; jn < 4; ++jn) acc[mi][jn] = (f32x4){0.f, 0.f, 0.f, 0.f};
#pragma unroll
  for (int j = 0; j < 13; ++j) rb[j] = (u32x4){0u, 0u, 0u, 0u};

  s16x4 wlo;
#define RD_WL(KK, jn) wlo = __builtin_amdgcn_ds_read_tr16_b64_v4i16((lp4)(aw[jn] + (KK) * 8192))
#define RD_WH(D, KK, jn) D[jn] = __builtin_bit_cast(bf16x8, __builtin_shufflevector(wlo, (s16x4)__builtin_amdgcn_ds_read_tr16_b64_v4i16((lp4)(aw[jn] + (KK) * 8192 + 1024)), 0, 1, 2, 3, 4, 5, 6, 7))
#define RD_X0(mi) x0[mi] = *(lp8)(ax0 + (mi) * 2048)
#define RD_X1(mi) x1[mi] = *(lp8)(ax1 + (mi) * 2048)
#define MFMA(ACC, A, B) asm volatile("v_mfma_f32_16x16x32_bf16 %0, %1, %2, %0" : "+v"(ACC) : "v"(A), "v"(B))
#define MFMA0(ACC, A, B) asm volatile("v_mfma_f32_16x16x32_bf16 %0, %1, %2, 0" : "+v"(ACC) : "v"(A), "v"(B))
#define SLOT(G, S, mi, XV, WV, ZZ, FILL) do { \
    if constexpr (ZZ) MFMA0(acc[mi][S], WV[S], XV[mi]); else MFMA(acc[mi][S], WV[S], XV[mi]); \
    SCHED_FENCE(); FILL; extra(KIND_, IC<G>{}, IC<S>{}); SCHED_FENCE(); } while (0)
#define GROUP(G, mi, XV, WV, ZZ, F0, F1, F2, F3) do { \
    SLOT(G, 0, mi, XV, WV, ZZ, F0); SLOT(G, 1, mi, XV, WV, ZZ, F1); SLOT(G, 2, mi, XV, WV, ZZ, F2); SLOT(G, 3, mi, XV, WV, ZZ, F3); } while (0)
#define NOP_ (void)0
#define STPAD() do { SCHED_FENCE(); asm volatile("s_nop 1"); SCHED_FENCE(); } while (0)

  const int gb = b & ~3;
  rsrc_t rxd = mkrs(myws + WS_XD, 0);
  rsrc_t rxf = mkrs(myws + WS_XF, 0);
  unsigned xd_buf = 0, seq = 0;
  const unsigned stg_addr = lbase + STG + wid * 4096 + lane16;
  auto opaque_l16 = [&]() AINL { unsigned l = lane16; PIN(l); return l; };
  auto pack_part = [&](auto N_) AINL -> u32x4 {
    constexpr int n = decltype(N_)::v, mi = n >> 1, h = n & 1;
    const f32x4 v0 = acc[mi][2 * h], v1 = acc[mi][2 * h + 1];
    return (u32x4){pack2(v0[0], v0[1]), pack2(v0[2], v0[3]), pack2(v1[0], v1[1]), pack2(v1[2], v1[3])};
  };
  auto store_part = [&](auto N_, u32x4 d) AINL {
    constexpr int n = decltype(N_)::v, u = n >> 2;
    const unsigned so = xd_buf + (unsigned)(gb + ((q + u) & 3)) * XD_WG + (unsigned)((u - 1) * 4 + wid) * 4096u + (n & 3) * 1024u;
    __builtin_amdgcn_raw_buffer_store_b128(d, rxd, lane16, so, AUX_SYS);
    SCHED_FENCE(); asm volatile("s_nop 0");
  };
  auto load_part = [&](int slot_, int j) AINL -> u32x4 {
    return __builtin_amdgcn_raw_buffer_load_b128(rxd, lane16, xd_buf + (unsigned)b * XD_WG + (unsigned)(slot_ * 4 + wid) * 4096u + j * 1024u, AUX_SYS);
  };
  auto stage_own = [&](auto J_) AINL { *(lpq)(stg_addr + decltype(J_)::v * 1024) = pack_part(J_); };
  auto send_flags = [&]() AINL {
    WAIT_V(0);
    const unsigned l = opaque_l16() >> 4;
    const unsigned vo = (l >= 1 && l < 4) ? (((q + l) & 3) * 16 + (l - 1) * 4) * 4 : 0x80000000u;
    __builtin_amdgcn_raw_buffer_store_b32(seq, rxf, vo, (unsigned)(gb * 16 + wid) * 4u, AUX_SYS);
  };
  auto load_flags = [&]() AINL {
    const unsigned l = opaque_l16();
    flagv = __builtin_amdgcn_raw_buffer_load_b32(rxf, l < 32u ? l : 32u, (unsigned)(b * 16 + wid) * 4u, AUX_SYS);
  };

  int slot = 0;
  auto extra = [&](auto KIND_, auto G_, auto S_) AINL {
    constexpr int KIND = decltype(KIND_)::v, G = decltype(G_)::v, S = decltype(S_)::v, SL = G * 4 + S;
#define PK(sl, n) if constexpr (SL == sl) { rb[n - 3] = pack_part(IC<n>{}); PIN(rb[n - 3]); }
#define ST(sl, n) if constexpr (SL == sl) store_part(IC<n>{}, rb[n - 3]);
    if constexpr (KIND == 5 && !(ABL & 1)) {
      PK(39, 4) ST(41, 4) PK(42, 5) PK(43, 6) PK(47, 7) ST(49, 5) ST(57, 6)
      if constexpr (SL == 62) stage_own(IC<0>{});
      if constexpr (SL == 63) stage_own(IC<1>{});
    }
    if constexpr (KIND == 1 && !(ABL & 1)) {
      if constexpr (SL == 2) stage_own(IC<2>{});
      if constexpr (SL == 3) stage_own(IC<3>{});
      ST(1, 7) PK(5, 8) PK(6, 9) PK(7, 10) ST(9, 8) PK(11, 11) PK(15, 12) ST(17, 9) PK(19, 13) PK(23, 14) ST(25, 10) PK(26, 15)
      ST(33, 11) ST(41, 12) ST(49, 13) ST(56, 14) ST(63, 15)
    }
  };
  auto kstep = [&](auto KIND_) AINL {
    constexpr int KIND = decltype(KIND_)::v;
    constexpr bool Z = KIND == 1;
    slot = (slot + 1 == NS) ? 0 : slot + 1;
    const unsigned dlt = slot == 0 ? (unsigned)(-(NS - 1) * STAGE) : (unsigned)STAGE;
    if constexpr (Z) asm volatile("s_nop 3");
    SCHED_FENCE();
    GROUP(0, 0, x0, wa, Z, RD_X0(3), NOP_, NOP_, NOP_);
    GROUP(1, 1, x0, wa, Z, RD_X0(4), NOP_, NOP_, NOP_);
    GROUP(2, 2, x0, wa, Z, RD_X0(5), RD_WL(1, 0), RD_WH(wb, 1, 0), NOP_);
    GROUP(3, 3, x0, wa, Z, RD_X0(6), RD_WL(1, 1), RD_WH(wb, 1, 1), NOP_);
    GROUP(4, 4, x0, wa, Z, RD_X0(7), RD_WL(1, 2), RD_WH(wb, 1, 2), ax1 = ax0 ^ 64u; PIN(ax1));
    GROUP(5, 5, x0, wa, Z, RD_X1(2), RD_WL(1, 3), RD_WH(wb, 1, 3), NOP_);
    GROUP(6, 6, x0, wa, Z, RD_X1(3), NOP_, NOP_, NOP_);
    GROUP(7, 7, x0, wa, Z, RD_X1(4), NOP_, RD_X1(5), NOP_);
    GROUP(8, 2, x1, wb, false, RD_X1(6), NOP_, NOP_, NOP_);
    GROUP(9, 3, x1, wb, false, RD_X1(7), NOP_, NOP_, NOP_);
    GROUP(10, 4, x1, wb, false, RD_X1(0), NOP_, NOP_, NOP_);
    GROUP(11, 5, x1, wb, false, RD_X1(1), NOP_, NOP_, ax0 += dlt; PIN(ax0));
    GROUP(12, 6, x1, wb, false, aw[0] += dlt; PIN(aw[0]), aw[1] += dlt; PIN(aw[1]), aw[2] += dlt; PIN(aw[2]), aw[3] += dlt; PIN(aw[3]));
    WAIT_L(0);
    BARRIER();
    RD_X0(0);
    SCHED_FENCE();
    GROUP(13, 7, x1, wb, false, RD_WL(0, 0), RD_WH(wa, 0, 0), RD_WL(0, 1), RD_WH(wa, 0, 1));
    GROUP(14, 0, x1, wb, false, RD_WL(0, 2), RD_WH(wa, 0, 2), RD_WL(0, 3), RD_WH(wa, 0, 3));
    GROUP(15, 1, x1, wb, false, RD_X0(1), RD_X0(2), NOP_, NOP_);
  };

#define FSTAMP()
#define STAMP()
  BARRIER();
  STAMP();
  FSTAMP();
  RD_X0(0);
#pragma unroll
  for (int jn = 0; jn < 4; ++jn) { RD_WL(0, jn); RD_WH(wa, 0, jn); }
  RD_X0(1); RD_X0(2);
  SCHED_FENCE();
#pragma unroll 1
  for (int r = 0; r < ROUNDS; ++r) {
    kstep(IC<1>{}); FSTAMP();
    kstep(IC<0>{}); FSTAMP();
    send_flags();
#define KS0() do { kstep(IC<0>{}); FSTAMP(); } while (0)
    KS0(); KS0(); KS0(); KS0(); KS0(); KS0(); KS0(); KS0(); KS0(); KS0(); KS0(); KS0(); KS0();
    rxd = mkrs(myws + WS_XD, ROUNDS * XD_BUF);
    rxf = mkrs(myws + WS_XF, NWG * 64);
    xd_buf = (r & CFG_XM) * XD_BUF;
    seq = (call - 1u) * ROUNDS + r + 1u;
    kstep(IC<5>{}); FSTAMP();
    STAMP();
  }
  {
    SCHED_FENCE();
    store_part(IC<7>{}, rb[4]); STPAD();
#define SP(n) store_part(IC<n>{}, pack_part(IC<n>{})); STPAD()
    SP(8); SP(9); SP(10); SP(11); SP(12); SP(13); SP(14); SP(15);
    FSTAMP();
    send_flags();
    FSTAMP();
    {
      const unsigned long long t0 = __builtin_amdgcn_s_memrealtime();
      for (;;) {
        CFENCE();
        load_flags();
        if (__all(flagv >= seq)) break;
        if (give_up(myws, call, t0)) break;
      }
      CFENCE();
    }
    FSTAMP();
    u32x4 r3[3][4];
#pragma unroll
    for (int s = 0; s < 3; ++s)
#pragma unroll
      for (int j = 0; j < 4; ++j) r3[s][j] = load_part(s, j);
    const bool bad = __hip_atomic_load((unsigned*)(myws + WS_TMO), __ATOMIC_RELAXED, __HIP_MEMORY_SCOPE_AGENT) == call;
    const Tile tl = tile_of(b, ROUNDS - 1, rank);
    rsrc_t rd = mkrs(p.out[tl.peer], OUT_BYTES);
    const unsigned so = (unsigned)(tl.orow + 32 * q) * (NSH * 2) + (unsigned)tl.ocol * 2u;
    const unsigned l = opaque_l16() >> 4, gg = l >> 4;
    const unsigned vo = (l & 15) * (NSH * 2) + (8 * wid + (gg & 1) * 2 + (gg >> 1)) * 16u;
#pragma unroll
    for (int j = 0; j < 4; ++j) {
      f32x4 v[2] = {acc[j >> 1][2 * (j & 1)], acc[j >> 1][2 * (j & 1) + 1]};
#pragma unroll
      for (int s = 0; s < 3; ++s)
#pragma unroll
        for (int w = 0; w < 8; ++w) v[w >> 2][w & 3] += unpk(r3[s][j][w >> 1], w & 1);
      const unsigned d0 = relu2(pack2(v[0][0], v[0][1])), d1 = relu2(pack2(v[0][2], v[0][3]));
      const unsigned e0 = relu2(pack2(v[1][0], v[1][1])), e1 = relu2(pack2(v[1][2], v[1][3]));
      auto s0 = __builtin_amdgcn_permlane16_swap(d0, e0, false, false);
      auto s1 = __builtin_amdgcn_permlane16_swap(d1, e1, false, false);
      u32x4 o = {s0[0], s1[0], s0[1], s1[1]};
      if (bad) o = (u32x4){0x7fc07fc0u, 0x7fc07fc0u, 0x7fc07fc0u, 0x7fc07fc0u};
      __builtin_amdgcn_raw_buffer_store_b128(o, rd, vo, so + (j >> 1) * (16 * NSH * 2) + (j & 1) * 64u, 0);
      STPAD();
    }
    FSTAMP();
  }
  STAMP();
  FSTAMP();
  FSTAMP();
  STAMP();
  STAMP();
  if (wid == 0) {
    if (lane == 0) __hip_atomic_store((unsigned*)(myws + WS_CNT) + b, call, __ATOMIC_RELAXED, __HIP_MEMORY_SCOPE_AGENT);
  } else if (wid == 1 && b < NR - 1 && !(p.mode & 1)) {
    rsrc_t frs2 = mkrs(myws + WS_DONE2 + ((rank + 1 + b) & 7) * (NWG * 4), NWG * 4);
    const unsigned long long t0 = __builtin_amdgcn_s_memrealtime();
    for (;;) {
      CFENCE();
      const u32x4 f = __builtin_amdgcn_raw_buffer_load_b128(frs2, lane16, 0, AUX_SYS);
      const bool ok = f[0] >= call && f[1] >= call && f[2] >= call && f[3] >= call;
      if (__all(ok)) break;
      if (give_up(myws, call, t0)) {
        if (lane == 0) *(volatile unsigned short*)(p.out[rank] + (size_t)rank * MLOC * NSH * 2) = 0x7fc0;
        break;
      }
    }
  }
}
}

extern "C" void kernel_launch(int rank, int nranks, ncclComm_t comm, hipStream_t stream,
                              void* const* d_in, const int* in_sizes, int n_in,
                              void* d_out, int out_size, const int* in_dtypes, int out_dtype,
                              void* const* d_out_all, void* const* d_ws_all, size_t ws_bytes) {
  if (nranks != ga::NR || ws_bytes < ga::WS_NEED) return;
  ga::Params p;
  p.X = d_in[0]; p.W = d_in[1];
  for (int i = 0; i < ga::NR; ++i) { p.out[i] = (char*)d_out_all[i]; p.ws[i] = (char*)d_ws_all[i]; }
  p.rank = rank; p.mode = 0;
  hipLaunchKernelGGL(ga::gemm_a2a, dim3(ga::NWG), dim3(512), 0, stream, p);
}
```

```cpp
#include <hip/hip_runtime.h>
#include <rccl/rccl.h>
#include <cstdio>
#pragma clang diagnostic ignored "-Wint-to-pointer-cast"

namespace ga {
constexpr int MLOC = 512, KDIM = 4096, NDIM = 16384, NR = 8, NSH = 2048;
constexpr int BM = 128, BN = 256, BK = 64, NS = 3;
constexpr int XB = 16384, STAGE = 3 * XB, STG = NS * STAGE;
constexpr int LDS_BYTES = STG + 16384;
constexpr int NWG = 256, ROUNDS = 4, RSTEPS = 16, TSTEPS = ROUNDS * RSTEPS;
constexpr unsigned X_BYTES = MLOC * KDIM * 2u, W_BYTES = (unsigned)KDIM * NDIM * 2u, OUT_BYTES = 4096u * NSH * 2u;
constexpr unsigned long long DEADLINE = 100000000ull;
constexpr unsigned long long PACE = 0x006068686000ull;
constexpr unsigned WS_DONE = 0;
constexpr unsigned WS_REG = 2048;
constexpr unsigned WS_ARR = 4096;
constexpr unsigned WS_CNT = 8192;
constexpr unsigned WS_TMO = 16384;
constexpr unsigned WS_AGF = 32768;
constexpr unsigned WS_DONE2 = 49152;
constexpr unsigned WS_XF = 131072;
constexpr unsigned WS_XD = 1u << 20, XD_WG = 49152u, XD_BUF = NWG * XD_WG;
constexpr unsigned WS_G = 64u << 20, G_SRC = 1u << 20, G_BUF = NR * G_SRC;
constexpr unsigned WS_NEED = WS_G + 2 * G_BUF;
static_assert(WS_XD + ROUNDS * XD_BUF <= WS_G, "ws");
static_assert(LDS_BYTES <= 160 * 1024, "LDS");

typedef __attribute__((address_space(3))) char lds_char;
typedef short s16x4 __attribute__((ext_vector_type(4)));
typedef short s16x2 __attribute__((ext_vector_type(2)));
typedef __bf16 bf16x8 __attribute__((ext_vector_type(8)));
typedef __bf16 bf16x2 __attribute__((ext_vector_type(2)));
typedef float f32x4 __attribute__((ext_vector_type(4)));
typedef float f32x2 __attribute__((ext_vector_type(2)));
typedef unsigned u32x4 __attribute__((ext_vector_type(4)));
typedef __amdgpu_buffer_rsrc_t rsrc_t;
typedef __attribute__((address_space(3))) s16x4* lp4;
typedef const __attribute__((address_space(3))) bf16x8* lp8;
typedef volatile __attribute__((address_space(3))) u32x4* lpq;
typedef __attribute__((address_space(1))) unsigned* gptr_u;

struct Params {
  const void* X; const void* W;
  char* out[NR]; char* ws[NR];
  int rank; int pad; unsigned long long pace;
  unsigned tab[ROUNDS * 64];
};

#define WAIT_V(n) asm volatile("s_waitcnt vmcnt(%0)" ::"n"(n) : "memory")
#define WAIT_L(n) asm volatile("s_waitcnt lgkmcnt(%0)" ::"n"(n) : "memory")
#define BARRIER() do { asm volatile("" ::: "memory"); __builtin_amdgcn_s_barrier(); asm volatile("" ::: "memory"); } while (0)
#define SCHED_FENCE() __builtin_amdgcn_sched_barrier(0)
#define CFENCE() asm volatile("" ::: "memory")
#define AINL __attribute__((always_inline))
#define PIN(x) asm volatile("" : "+v"(x))
#define ST_SYS(ptr, val) __hip_atomic_store((ptr), (val), __ATOMIC_RELAXED, __HIP_MEMORY_SCOPE_SYSTEM)
constexpr int AUX_SYS = 17;
constexpr int AUX_NT = 2;

__device__ __forceinline__ rsrc_t mkrs(const void* p, unsigned bytes) { return __builtin_amdgcn_make_buffer_rsrc((void*)p, 0, bytes, 0x00020000); }

struct Tile { int xsrc;   int xrow, wcol, peer, orow, ocol, rot, blk, ph; };
__device__ __forceinline__ Tile tile_dec(unsigned e, int rank) {
  Tile t;
  const int src = (e >> 8) & 7, rt = (e >> 6) & 3;
  t.peer = e & 7; t.ocol = ((e >> 3) & 7) * BN;
  t.xsrc = src == rank ? -1 : src; t.xrow = src == rank ? rt * BM : 0; t.orow = src * MLOC + rt * BM;
  t.rot = (e >> 12) & 15; t.blk = (e >> 16) & 15; t.ph = (e >> 20) & 15;
  t.wcol = t.peer * NSH + t.ocol;
  return t;
}
#define TILE_OF(r) tile_dec(p.tab[((r) & 3) * 64 + ((b >> 2) & 1) * 32 + (b >> 3)], rank)

__device__ __forceinline__ unsigned pack2(float a, float b) {
  f32x2 v = {a, b};
  return __builtin_bit_cast(unsigned, __builtin_convertvector(v, bf16x2));
}
__device__ __forceinline__ unsigned relu2(unsigned x) {
  s16x2 s = __builtin_bit_cast(s16x2, x), z = {0, 0};
  return __builtin_bit_cast(unsigned, __builtin_elementwise_max(s, z));
}
__device__ __forceinline__ float unpk(unsigned d, int hi) { return __builtin_bit_cast(float, hi ? (d & 0xffff0000u) : (d << 16)); }
__device__ __forceinline__ int vm_outstanding() { int v; asm volatile("s_getreg_b32 %0, hwreg(HW_REG_IB_STS, 0, 4)" : "=s"(v)); return v; }
__device__ __forceinline__ unsigned xcc_id() { unsigned v; asm volatile("s_getreg_b32 %0, hwreg(HW_REG_XCC_ID, 0, 3)" : "=s"(v)); return v; }
__device__ __forceinline__ bool all_ge(const u32x4& f, unsigned v) { return __all(f[0] >= v && f[1] >= v && f[2] >= v && f[3] >= v); }
template <int N> struct IC { static constexpr int v = N; };

__device__ __forceinline__ bool give_up(char* myws, unsigned call, unsigned long long t0) {
  if (__hip_atomic_load((unsigned*)(myws + WS_TMO), __ATOMIC_RELAXED, __HIP_MEMORY_SCOPE_AGENT) == call ||
      __builtin_amdgcn_s_memrealtime() - t0 > DEADLINE) {
    __hip_atomic_store((unsigned*)(myws + WS_TMO), call, __ATOMIC_RELAXED, __HIP_MEMORY_SCOPE_AGENT);
    return true;
  }
  return false;
}
__device__ __forceinline__ bool src_done(unsigned v, unsigned call, int lane) {
  const unsigned long long ge = __ballot((v >> 8) >= call), gt = __ballot((v >> 8) > call);
  unsigned m = (v >> 8) == call ? (v & 255u) : 0u;
  m |= __shfl_xor(m, 1); m |= __shfl_xor(m, 2); m |= __shfl_xor(m, 4);
  const unsigned geb = (unsigned)(ge >> (lane & 56)) & 255u, gtb = (unsigned)(gt >> (lane & 56)) & 255u;
  return gtb != 0u || (m != 0u && (m & ~geb) == 0u);
}

__global__ void __launch_bounds__(512, 2) gemm_a2a(Params p) {
  __shared__ __attribute__((aligned(1024))) char smem[LDS_BYTES];
  lds_char* lds = (lds_char*)smem;
  const unsigned lbase = (unsigned)(size_t)lds;
  const int tid = threadIdx.x, lane = tid & 63;
  const int wid = __builtin_amdgcn_readfirstlane(tid >> 6);
  const int b = blockIdx.x, q = b & 3, rank = p.rank & 7;
  char* myws = p.ws[rank];
  unsigned call = *(const unsigned*)(myws + WS_CNT + b * 4) + 1u;
  const unsigned lane16 = lane * 16;

  if (wid >= 4 && wid < 7) {
    const bool isx = wid == 4;
    unsigned v[2];
#pragma unroll
    for (int e = 0; e < 2; ++e) {
      const unsigned vx = (unsigned)(lane >> 3) * (KDIM * 2) + (unsigned)(((lane & 7) ^ ((4 * e + (lane >> 4)) & 7)) * 16);
      const int c16 = lane & 15, f = (lane >> 4) | (e << 2);
      const unsigned vw = (unsigned)(lane >> 4) * (NDIM * 2) + (unsigned)((((c16 >> 1) ^ f) * 32) + (c16 & 1) * 16);
      v[e] = isx ? vx : vw;
    }
    rsrc_t rs = mkrs(p.W, W_BYTES);
    unsigned sbase = 0;
    int slot = 0, ks = 0, rnd = 0, kb = 0, ki = 0, blk = 1, cur = 1, rot = 0, nleft = 0;
    auto issue = [&]() AINL {
      if (ks == 0) {
        const Tile t = TILE_OF(rnd);
        blk = t.blk; rot = t.rot; cur = t.ph ? t.ph : blk;
        kb = 0; nleft = cur; ki = rot * cur / blk;
        if (isx) {
          rs = t.xsrc < 0 ? mkrs(p.X, X_BYTES) : mkrs(myws + WS_G + (call & 1) * G_BUF + t.xsrc * G_SRC, G_SRC);
          sbase = (unsigned)t.xrow * (KDIM * 2) + (unsigned)q * 2048u;
        } else {
          sbase = (unsigned)(q * 1024) * (NDIM * 2) + (unsigned)(t.wcol + (wid - 5) * 128) * 2u;
        }
      }
      const int kse = kb + ki;
      if (++ki == cur) ki = 0;
      if (--nleft == 0) { kb += cur; cur = RSTEPS - kb < blk ? RSTEPS - kb : blk; nleft = cur; ki = cur == blk ? rot : rot * cur / blk; }
      lds_char* base = lds + slot * STAGE + (wid - 4) * XB;
      if (isx) {
        const unsigned s = sbase + (unsigned)kse * (BK * 2);
        if (rnd == ROUNDS - 1) {
#pragma unroll
          for (int i = 0; i < 16; ++i)
            __builtin_amdgcn_raw_ptr_buffer_load_lds(rs, base + i * 1024, 16, v[i & 1],
                                                     s + (unsigned)(32 * (((i >> 2) + q) & 3) + 8 * (i & 3)) * (KDIM * 2), 0, AUX_SYS);
        } else {
#pragma unroll
          for (int i = 0; i < 16; ++i)
            __builtin_amdgcn_raw_ptr_buffer_load_lds(rs, base + i * 1024, 16, v[i & 1],
                                                     s + (unsigned)(32 * (((i >> 2) + q) & 3) + 8 * (i & 3)) * (KDIM * 2), 0, 0);
        }
      } else {
        const unsigned s = sbase + (unsigned)kse * (BK * NDIM * 2);
#pragma unroll
        for (int i = 0; i < 16; ++i)
          __builtin_amdgcn_raw_ptr_buffer_load_lds(rs, base + i * 1024, 16, v[(i >> 1) & 1], s + i * 4 * (NDIM * 2), 0, 0);
      }
      slot = (slot + 1 == NS) ? 0 : slot + 1;
      if (++ks == RSTEPS) { ks = 0; ++rnd; }
    };
#pragma unroll 1
    for (int s = 0; s < NS; ++s) issue();
    asm volatile("" : "+s"(call));
    WAIT_V(32);
    BARRIER();
    unsigned tnext = (unsigned)__builtin_amdgcn_s_memrealtime();
#pragma unroll 1
    for (int t = 0; t < TSTEPS - NS; ++t) {
      WAIT_V(16);
      if (isx) {
        tnext += (unsigned)(p.pace >> (t & 56)) & 255u;
        unsigned now = (unsigned)__builtin_amdgcn_s_memrealtime();
        while ((int)(now - tnext) < 0) now = (unsigned)__builtin_amdgcn_s_memrealtime();
        tnext = now;
      }
      BARRIER();
      issue();
    }
    WAIT_V(16); BARRIER();
    WAIT_V(0); BARRIER();
    if (!isx) { BARRIER(); return; }
    const unsigned xcc = xcc_id();
    rsrc_t rreg = mkrs(myws + WS_REG, NWG * 4), rarr = mkrs(myws + WS_ARR, NWG * 4);
    u32x4 rg = __builtin_amdgcn_raw_buffer_load_b128(rreg, lane16, 0, AUX_SYS);
    u32x4 ar = __builtin_amdgcn_raw_buffer_load_b128(rarr, lane16, 0, AUX_SYS);
    char* pw = p.ws[lane & 7];
    BARRIER();
    const unsigned long long t0 = __builtin_amdgcn_s_memrealtime();
    while (!__all((rg[0] >> 4) == call && (rg[1] >> 4) == call && (rg[2] >> 4) == call && (rg[3] >> 4) == call)) {
      if (give_up(myws, call, t0)) return;
      CFENCE();
      rg = __builtin_amdgcn_raw_buffer_load_b128(rreg, lane16, 0, AUX_SYS);
    }
    bool lower = false; unsigned mask = 0;
#pragma unroll
    for (int k = 0; k < 4; ++k) lower |= (rg[k] & 7u) == xcc && lane * 4 + k < b;
    if (__any(lower)) return;
#pragma unroll
    for (unsigned x = 0; x < 8; ++x)
      if (__any((rg[0] & 7u) == x || (rg[1] & 7u) == x || (rg[2] & 7u) == x || (rg[3] & 7u) == x)) mask |= 1u << x;
    for (;;) {
      bool ok = true;
#pragma unroll
      for (int k = 0; k < 4; ++k) ok &= (rg[k] & 7u) != xcc || ar[k] >= call;
      if (__all(ok)) break;
      if (give_up(myws, call, t0)) return;
      CFENCE();
      ar = __builtin_amdgcn_raw_buffer_load_b128(rarr, lane16, 0, AUX_SYS);
    }
    __builtin_amdgcn_fence(__ATOMIC_RELEASE, "");
    WAIT_V(0);
    if (lane < NR && lane != rank) ST_SYS((unsigned*)(pw + WS_DONE + rank * 128) + xcc, (call << 8) | mask);
    return;
  }

  if (wid == 7) {
    int t = -1;
    auto step = [&]() AINL { if (t < TSTEPS - 1) { BARRIER(); ++t; } };
    const unsigned st_voff = (unsigned)(lane & 7) * (NSH * 2) + (unsigned)(((lane >> 4) & 1) * 2 + (lane >> 5)) * 16u + (unsigned)(lane & 8) * 8u;
    rsrc_t rxd = mkrs(myws + WS_XD, (ROUNDS - 1) * XD_BUF), rxf = mkrs(myws + WS_XF + b * 64, 48);
    const unsigned goff = (unsigned)b * 4096u;
    const bool flane = lane < NR && lane != rank;
    size_t fptr = (size_t)(p.ws[lane & 7] + (rank * NWG + b) * 4);
    u32x4 d[4], ow[16], in[8][3], o, oprev, ob, agf;
    unsigned xfv = 0;
    int agn = 0;
    auto ag_store = [&](const u32x4& dd, int j) AINL {
      const int peer = (rank + 1 + (agn + b) % 7) & 7;
      rsrc_t rd = mkrs(p.ws[peer] + WS_G + (call & 1) * G_BUF + rank * G_SRC, agn < 28 ? G_SRC : 0u);
      __builtin_amdgcn_raw_buffer_store_b128(dd, rd, lane16, goff + j * 1024, AUX_SYS);
      ++agn;
    };
    auto xf_load = [&]() AINL { CFENCE(); xfv = __builtin_amdgcn_raw_buffer_load_b32(rxf, lane < 12 ? lane * 4 : 0, 0, AUX_SYS); CFENCE(); };
    auto xf_check = [&](unsigned seq) AINL {
      if (__builtin_expect(!__all(xfv >= seq), 0)) {
        const unsigned long long t0 = __builtin_amdgcn_s_memrealtime();
        for (;;) {
          xf_load();
          if (__all(xfv >= seq)) break;
          if (give_up(myws, call, t0)) break;
        }
      }
      CFENCE();
    };
    auto in_load = [&](int r, int i) AINL {
#pragma unroll
      for (int k = 0; k < 3; ++k)
        in[i & 7][k] = __builtin_amdgcn_raw_buffer_load_b128(rxd, lane16, r * XD_BUF + (unsigned)b * XD_WG + k * 16384u + i * 1024u, AUX_SYS);
    };
    auto ow_load = [&]() AINL {
#pragma unroll
      for (int i = 0; i < 16; ++i) ow[i] = *(lpq)(lbase + STG + i * 1024 + lane16);
      WAIT_L(0);
    };
    const unsigned seq0 = (call - 1u) * ROUNDS + 1u;
    {
      rsrc_t rx = mkrs(p.X, X_BYTES);
#pragma unroll
      for (int j = 0; j < 4; ++j) d[j] = __builtin_amdgcn_raw_buffer_load_b128(rx, lane16, 3u * BM * KDIM * 2 + goff + j * 1024, 0);
      if (lane == 0) ST_SYS((unsigned*)(myws + WS_REG) + b, (call << 4) | xcc_id());
      BARRIER();
      PIN(fptr);
      WAIT_V(0);
#pragma unroll
      for (int j = 0; j < 3; ++j) {
#pragma unroll 1
        for (int k = 0; k < (j < 2 ? 7 : 6); ++k) { ag_store(d[j], j); step(); }
      }
      xf_load(); ag_store(d[2], 2); step();
      xf_check(seq0); in_load(0, 0); ag_store(d[3], 3); step();
      in_load(0, 1); ag_store(d[3], 3); ow_load(); step();
    }
    const int agsrc = TILE_OF(ROUNDS - 1).xsrc;
#pragma unroll 1
    for (int r = 0; r < ROUNDS - 1; ++r) {
      const Tile tl = TILE_OF(r);
      const bool more = r < ROUNDS - 2;
      rsrc_t rd = mkrs(p.out[tl.peer], more ? 0u : OUT_BYTES), rdn = mkrs(p.out[tl.peer], more ? OUT_BYTES : 0u);
      auto ostore = [&](const u32x4& dd, unsigned so_) AINL {
        __builtin_amdgcn_raw_buffer_store_b128(dd, rd, st_voff, so_, AUX_SYS);
        __builtin_amdgcn_raw_buffer_store_b128(dd, rdn, st_voff, so_, AUX_NT);
      };
      const unsigned so = (unsigned)(tl.orow + 32 * q) * (NSH * 2) + (unsigned)tl.ocol * 2u;
      const bool agchk = r == 1 && agsrc >= 0;
      rsrc_t raf = mkrs(myws + WS_AGF + (agsrc & 7) * (NWG * 4), agchk ? NWG * 4 : 0);
#pragma unroll
      for (int i = 0; i < 16; ++i) {
        if (i == 0 && r == 2) {
          WAIT_V(0);
          if (lane == 0) ST_SYS((unsigned*)(myws + WS_ARR) + b, call);
        }
        if (i == 12 && r == 0) {
          WAIT_V(0);
          if (flane) ST_SYS((gptr_u)(fptr + WS_AGF), call);
        }
        if (i == 13) xf_load();
        if (i == 5) { agf = __builtin_amdgcn_raw_buffer_load_b128(raf, lane16, 0, AUX_SYS); CFENCE(); }
        {
          float v[8];
#define GAP() asm volatile("s_nop 0")
#pragma unroll
          for (int w = 0; w < 8; ++w) {
            float a_ = unpk(ow[i][w >> 1], w & 1); PIN(a_); GAP();
#pragma unroll
            for (int k = 0; k < 3; ++k) { float u_ = unpk(in[i & 7][k][w >> 1], w & 1); PIN(u_); GAP(); a_ += u_; PIN(a_); GAP(); }
            v[w] = a_;
          }
          unsigned d0 = pack2(v[0], v[1]); PIN(d0); GAP(); unsigned d1 = pack2(v[2], v[3]); PIN(d1); GAP();
          unsigned e0 = pack2(v[4], v[5]); PIN(e0); GAP(); unsigned e1 = pack2(v[6], v[7]); PIN(e1); GAP();
          d0 = relu2(d0); PIN(d0); GAP(); d1 = relu2(d1); PIN(d1); GAP(); e0 = relu2(e0); PIN(e0); GAP(); e1 = relu2(e1); PIN(e1); GAP();
          auto s0 = __builtin_amdgcn_permlane16_swap(d0, e0, false, false);
          auto s1 = __builtin_amdgcn_permlane16_swap(d1, e1, false, false);
          o = (u32x4){s0[0], s1[0], s0[1], s1[1]};
          PIN(o);
        }
        if (i == 6) {
          if (__builtin_expect(agchk && !all_ge(agf, call), 0)) {
            const unsigned long long t0 = __builtin_amdgcn_s_memrealtime();
            for (;;) {
              CFENCE();
              agf = __builtin_amdgcn_raw_buffer_load_b128(raf, lane16, 0, AUX_SYS);
              if (all_ge(agf, call)) break;
              if (give_up(myws, call, t0)) break;
            }
          }
          CFENCE();
        }
        if (i == 14) xf_check(more ? seq0 + r + 1u : 0u);
        if (i < 6) { in_load(r, 2 * i + 2); in_load(r, 2 * i + 3); }
        else if (i < 8) in_load(r, i + 8);
        else if (i >= 14) in_load(r + 1, i - 14);
        CFENCE();
        if (i & 1) {
          u32x4 oa;
#pragma unroll
          for (int k = 0; k < 4; ++k) {
            oa[k] = __builtin_amdgcn_update_dpp(oprev[k], o[k], 0x128  , 0xf, 0xc, false); PIN(oa); GAP();
            ob[k] = __builtin_amdgcn_update_dpp(o[k], oprev[k], 0x128, 0xf, 0x3, false); PIN(ob); GAP();
          }
          ostore(oa, so + (unsigned)((i >> 1) & 1) * (16 * NSH * 2) + (i >> 2) * 128u);
          if (i == 15) ostore(ob, so + (unsigned)(16 + 8) * (NSH * 2) + 3 * 128u);
        } else {
          oprev = o; PIN(oprev);
          if (i > 0) ostore(ob, so + (unsigned)((((i - 1) >> 1) & 1) * 16 + 8) * (NSH * 2) + ((i - 1) >> 2) * 128u);
        }
        if (i < 10 && !(i & 1)) ag_store(d[3], 3);
        if (i == 15 && more) ow_load();
        SCHED_FENCE(); asm volatile("s_nop 1" ::: "memory"); SCHED_FENCE();
        if (more || i < 4 || (i & 1)) step();
      }
    }
    while (t < TSTEPS - 1 && vm_outstanding() > 0) step();
    WAIT_V(0);
    if (flane) ST_SYS((gptr_u)(fptr + WS_DONE2), call);
    while (t < TSTEPS - 1) step();
    return;
  }

  const int c = lane & 15, g = lane >> 4;
  unsigned ax0 = lbase + c * 128 + ((g ^ (c >> 1)) * 16), ax1 = 0, aw[4];
  {
    const int qq = c >> 2, pp = c & 3;
#pragma unroll
    for (int jn = 0; jn < 4; ++jn)
      aw[jn] = lbase + XB * (1 + (wid >> 1)) + (8 * g + qq) * 256 + (((4 * (wid & 1) + jn) ^ (qq | ((g & 1) << 2))) * 32) + 8 * pp;
  }
  f32x4 acc[8][4];
  bf16x8 x0[8], x1[8], wa[4], wb[4];
  u32x4 rb[12];
#pragma unroll
  for (int mi = 0; mi < 8; ++mi)
#pragma unroll
    for (int jn = 0; jn < 4; ++jn) acc[mi][jn] = (f32x4){0.f, 0.f, 0.f, 0.f};
#pragma unroll
  for (int j = 0; j < 12; ++j) rb[j] = (u32x4){0u, 0u, 0u, 0u};

  s16x4 wlo;
#define RD_WL(KK, jn) wlo = __builtin_amdgcn_ds_read_tr16_b64_v4i16((lp4)(aw[jn] + (KK) * 8192))
#define RD_WH(D, KK, jn) D[jn] = __builtin_bit_cast(bf16x8, __builtin_shufflevector(wlo, (s16x4)__builtin_amdgcn_ds_read_tr16_b64_v4i16((lp4)(aw[jn] + (KK) * 8192 + 1024)), 0, 1, 2, 3, 4, 5, 6, 7))
#define RD_X0(mi) x0[mi] = *(lp8)(ax0 + (mi) * 2048)
#define RD_X1(mi) x1[mi] = *(lp8)(ax1 + (mi) * 2048)
#define MFMA(ACC, A, B) asm volatile("v_mfma_f32_16x16x32_bf16 %0, %1, %2, %0" : "+v"(ACC) : "v"(A), "v"(B))
#define MFMA0(ACC, A, B) asm volatile("v_mfma_f32_16x16x32_bf16 %0, %1, %2, 0" : "+v"(ACC) : "v"(A), "v"(B))
#define SLOT(G, S, mi, XV, WV, ZZ, FILL) do { \
    if constexpr (ZZ) MFMA0(acc[mi][S], WV[S], XV[mi]); else MFMA(acc[mi][S], WV[S], XV[mi]); \
    SCHED_FENCE(); FILL; extra(KIND_, IC<G>{}, IC<S>{}); SCHED_FENCE(); } while (0)
#define GROUP(G, mi, XV, WV, ZZ, F0, F1, F2, F3) do { \
    SLOT(G, 0, mi, XV, WV, ZZ, F0); SLOT(G, 1, mi, XV, WV, ZZ, F1); SLOT(G, 2, mi, XV, WV, ZZ, F2); SLOT(G, 3, mi, XV, WV, ZZ, F3); } while (0)
#define NOP_ (void)0
#define STPAD() do { SCHED_FENCE(); asm volatile("s_nop 1"); SCHED_FENCE(); } while (0)

  const int gb = b & ~3;
  rsrc_t rxd = mkrs(myws + WS_XD, 0);
  rsrc_t rxf = mkrs(myws + WS_XF, 0);
  unsigned xd_buf = 0, seq = 0;
  const unsigned stg_addr = lbase + STG + wid * 4096 + lane16;
  auto opaque_l16 = [&]() AINL { unsigned l = lane16; PIN(l); return l; };
  auto pack_part = [&](auto N_) AINL -> u32x4 {
    constexpr int n = decltype(N_)::v, mi = n >> 1, h = n & 1;
    const f32x4 v0 = acc[mi][2 * h], v1 = acc[mi][2 * h + 1];
    return (u32x4){pack2(v0[0], v0[1]), pack2(v0[2], v0[3]), pack2(v1[0], v1[1]), pack2(v1[2], v1[3])};
  };
  auto store_part = [&](auto N_, u32x4 dd) AINL {
    constexpr int n = decltype(N_)::v, u = n >> 2;
    const unsigned so = xd_buf + (unsigned)(gb + ((q + u) & 3)) * XD_WG + (unsigned)((u - 1) * 4 + wid) * 4096u + (n & 3) * 1024u;
    __builtin_amdgcn_raw_buffer_store_b128(dd, rxd, lane16, so, AUX_SYS);
    SCHED_FENCE(); asm volatile("s_nop 0");
  };
  auto stage_own = [&](auto J_) AINL { *(lpq)(stg_addr + decltype(J_)::v * 1024) = pack_part(J_); };
  auto send_flags = [&]() AINL {
    WAIT_V(0);
    const unsigned l = opaque_l16() >> 4;
    const unsigned vo = (l >= 1 && l < 4) ? (((q + l) & 3) * 16 + (l - 1) * 4) * 4 : 0x80000000u;
    __builtin_amdgcn_raw_buffer_store_b32(seq, rxf, vo, (unsigned)(gb * 16 + wid) * 4u, AUX_SYS);
  };

  int slot = 0;
  auto extra = [&](auto KIND_, auto G_, auto S_) AINL {
    constexpr int KIND = decltype(KIND_)::v, SL = decltype(G_)::v * 4 + decltype(S_)::v;
#define PK(sl, n) if constexpr (SL == sl) { rb[n - 4] = pack_part(IC<n>{}); PIN(rb[n - 4]); }
#define ST(sl, n) if constexpr (SL == sl) store_part(IC<n>{}, rb[n - 4]);
    if constexpr (KIND == 5) {
      PK(39, 4) ST(41, 4) PK(42, 5) PK(43, 6) PK(47, 7) ST(49, 5) ST(57, 6)
      if constexpr (SL == 62) stage_own(IC<0>{});
      if constexpr (SL == 63) stage_own(IC<1>{});
    }
    if constexpr (KIND == 1) {
      if constexpr (SL == 2) stage_own(IC<2>{});
      if constexpr (SL == 3) stage_own(IC<3>{});
      ST(1, 7) PK(5, 8) PK(6, 9) PK(7, 10) ST(9, 8) PK(11, 11) PK(15, 12) ST(17, 9) PK(19, 13) PK(23, 14) ST(25, 10) PK(26, 15)
      ST(33, 11) ST(41, 12) ST(49, 13) ST(56, 14) ST(63, 15)
    }
  };
  auto kstep = [&](auto KIND_) AINL {
    constexpr bool Z = decltype(KIND_)::v == 1;
    slot = (slot + 1 == NS) ? 0 : slot + 1;
    const unsigned dlt = slot == 0 ? (unsigned)(-(NS - 1) * STAGE) : (unsigned)STAGE;
    if constexpr (Z) asm volatile("s_nop 3");
    SCHED_FENCE();
    GROUP(0, 0, x0, wa, Z, RD_X0(3), NOP_, NOP_, NOP_);
    GROUP(1, 1, x0, wa, Z, RD_X0(4), NOP_, NOP_, NOP_);
    GROUP(2, 2, x0, wa, Z, RD_X0(5), RD_WL(1, 0), RD_WH(wb, 1, 0), NOP_);
    GROUP(3, 3, x0, wa, Z, RD_X0(6), RD_WL(1, 1), RD_WH(wb, 1, 1), NOP_);
    GROUP(4, 4, x0, wa, Z, RD_X0(7), RD_WL(1, 2), RD_WH(wb, 1, 2), ax1 = ax0 ^ 64u; PIN(ax1));
    GROUP(5, 5, x0, wa, Z, RD_X1(2), RD_WL(1, 3), RD_WH(wb, 1, 3), NOP_);
    GROUP(6, 6, x0, wa, Z, RD_X1(3), NOP_, NOP_, NOP_);
    GROUP(7, 7, x0, wa, Z, RD_X1(4), NOP_, RD_X1(5), NOP_);
    GROUP(8, 2, x1, wb, false, RD_X1(6), NOP_, NOP_, NOP_);
    GROUP(9, 3, x1, wb, false, RD_X1(7), NOP_, NOP_, NOP_);
    GROUP(10, 4, x1, wb, false, RD_X1(0), NOP_, NOP_, NOP_);
    GROUP(11, 5, x1, wb, false, RD_X1(1), NOP_, NOP_, ax0 += dlt; PIN(ax0));
    GROUP(12, 6, x1, wb, false, aw[0] += dlt; PIN(aw[0]), aw[1] += dlt; PIN(aw[1]), aw[2] += dlt; PIN(aw[2]), aw[3] += dlt; PIN(aw[3]));
    WAIT_L(0);
    BARRIER();
    RD_X0(0);
    SCHED_FENCE();
    GROUP(13, 7, x1, wb, false, RD_WL(0, 0), RD_WH(wa, 0, 0), RD_WL(0, 1), RD_WH(wa, 0, 1));
    GROUP(14, 0, x1, wb, false, RD_WL(0, 2), RD_WH(wa, 0, 2), RD_WL(0, 3), RD_WH(wa, 0, 3));
    GROUP(15, 1, x1, wb, false, RD_X0(1), RD_X0(2), NOP_, NOP_);
  };

  asm volatile("" : "+s"(call));
  BARRIER();
  RD_X0(0);
#pragma unroll
  for (int jn = 0; jn < 4; ++jn) { RD_WL(0, jn); RD_WH(wa, 0, jn); }
  RD_X0(1); RD_X0(2);
  SCHED_FENCE();
#define KS0() kstep(IC<0>{})
#pragma unroll 1
  for (int r = 0; r < ROUNDS; ++r) {
    kstep(IC<1>{});
    KS0();
    send_flags();
    KS0(); KS0(); KS0(); KS0(); KS0(); KS0(); KS0(); KS0(); KS0(); KS0(); KS0(); KS0(); KS0();
    rxd = mkrs(myws + WS_XD, ROUNDS * XD_BUF);
    rxf = mkrs(myws + WS_XF, NWG * 64);
    xd_buf = r * XD_BUF;
    seq = (call - 1u) * ROUNDS + r + 1u;
    kstep(IC<5>{});
  }
  {
    SCHED_FENCE();
    store_part(IC<7>{}, rb[3]); STPAD();
#define SP(n) store_part(IC<n>{}, pack_part(IC<n>{})); STPAD()
    SP(8); SP(9); SP(10); SP(11); SP(12); SP(13); SP(14); SP(15);
    send_flags();
    {
      const unsigned long long t0 = __builtin_amdgcn_s_memrealtime();
      const unsigned l = opaque_l16();
      for (;;) {
        CFENCE();
        const unsigned f = __builtin_amdgcn_raw_buffer_load_b32(rxf, l < 32u ? l : 32u, (unsigned)(b * 16 + wid) * 4u, AUX_SYS);
        if (__all(f >= seq)) break;
        if (give_up(myws, call, t0)) break;
      }
      CFENCE();
    }
    u32x4 r3[3][4];
#pragma unroll
    for (int s = 0; s < 3; ++s)
#pragma unroll
      for (int j = 0; j < 4; ++j)
        r3[s][j] = __builtin_amdgcn_raw_buffer_load_b128(rxd, lane16, xd_buf + (unsigned)b * XD_WG + (unsigned)(s * 4 + wid) * 4096u + j * 1024u, AUX_SYS);
    const bool bad = __hip_atomic_load((unsigned*)(myws + WS_TMO), __ATOMIC_RELAXED, __HIP_MEMORY_SCOPE_AGENT) == call;
    const Tile tl = TILE_OF(ROUNDS - 1);
    rsrc_t rd = mkrs(p.out[tl.peer], OUT_BYTES);
    const unsigned so = (unsigned)(tl.orow + 32 * q) * (NSH * 2) + (unsigned)tl.ocol * 2u;
    const unsigned l = opaque_l16() >> 4, gg = l >> 4;
    const unsigned vo = (l & 15) * (NSH * 2) + (8 * wid + (gg & 1) * 2 + (gg >> 1)) * 16u;
#pragma unroll
    for (int j = 0; j < 4; ++j) {
      f32x4 v[2] = {acc[j >> 1][2 * (j & 1)], acc[j >> 1][2 * (j & 1) + 1]};
#pragma unroll
      for (int s = 0; s < 3; ++s)
#pragma unroll
        for (int w = 0; w < 8; ++w) v[w >> 2][w & 3] += unpk(r3[s][j][w >> 1], w & 1);
      const unsigned d0 = relu2(pack2(v[0][0], v[0][1])), d1 = relu2(pack2(v[0][2], v[0][3]));
      const unsigned e0 = relu2(pack2(v[1][0], v[1][1])), e1 = relu2(pack2(v[1][2], v[1][3]));
      auto s0 = __builtin_amdgcn_permlane16_swap(d0, e0, false, false);
      auto s1 = __builtin_amdgcn_permlane16_swap(d1, e1, false, false);
      u32x4 o = {s0[0], s1[0], s0[1], s1[1]};
      if (bad) o = (u32x4){0x7fc07fc0u, 0x7fc07fc0u, 0x7fc07fc0u, 0x7fc07fc0u};
      __builtin_amdgcn_raw_buffer_store_b128(o, rd, vo, so + (j >> 1) * (16 * NSH * 2) + (j & 1) * 64u, AUX_SYS);
      STPAD();
    }
  }
  if (wid == 0) {
    if (lane == 0) __hip_atomic_store((unsigned*)(myws + WS_CNT) + b, call, __ATOMIC_RELAXED, __HIP_MEMORY_SCOPE_AGENT);
  } else if (wid == 1 && b < NR) {
    rsrc_t frs = mkrs(myws + WS_DONE, NR * 128), frs2 = mkrs(myws + WS_DONE2 + ((rank + b) & 7) * (NWG * 4), NWG * 4);
    const unsigned long long t0 = __builtin_amdgcn_s_memrealtime();
    for (;;) {
      CFENCE();
      bool ok;
      if (b == 0) {
        const unsigned v = __builtin_amdgcn_raw_buffer_load_b32(frs, (lane >> 3) * 128 + (lane & 7) * 4, 0, AUX_SYS);
        ok = __all(src_done(v, call, lane) || (lane >> 3) == rank);
      } else {
        ok = all_ge(__builtin_amdgcn_raw_buffer_load_b128(frs2, lane16, 0, AUX_SYS), call);
      }
      if (ok) break;
      if (give_up(myws, call, t0)) {
        if (lane == 0) *(volatile unsigned short*)(p.out[rank] + (size_t)rank * MLOC * NSH * 2) = 0x7fc0;
        break;
      }
    }
  }
}

static void place(unsigned* tab, int h, int& pos, int peer, int col, int n, const int* src, const int* rt, int gi) {
  int k = 0;
  while (k < n) {
    const int room = 32 - (pos & 31), m = n - k < room ? n - k : room;
    for (int j = 0; j < m; ++j, ++pos) {
      const int ph = m > 1 && m <= 4 ? gi % m : 0;
      tab[(pos >> 5) * 64 + h * 32 + (pos & 31)] = (unsigned)(peer | col << 3 | rt[k + j] << 6 | src[k + j] << 8 | j << 12 | m << 16 | ph << 20);
    }
    k += m;
  }
}
static void build_table(unsigned* tab, int rank) {
  for (int h = 0; h < 2; ++h) {
    int pos = 0, src[8], rt[8];
    for (int u = 0; u < 32; ++u) {
      for (int j = 0; j < 3; ++j) { src[j] = rank; rt[j] = j; }
      place(tab, h, pos, u < 28 ? (rank + 1 + (u + 4 * h) % 7) & 7 : rank, h * 4 + (u < 28 ? u / 7 : u - 28), 3, src, rt, u + h);
    }
    for (int c = 0; c < 4; ++c) {
      for (int j = 0; j < 8; ++j) { src[j] = (rank + j) & 7; rt[j] = 3; }
      place(tab, h, pos, rank, h * 4 + c, 8, src, rt, 0);
    }
  }
}
}

extern "C" void kernel_launch(int rank, int nranks, ncclComm_t comm, hipStream_t stream,
                              void* const* d_in, const int* in_sizes, int n_in,
                              void* d_out, int out_size, const int* in_dtypes, int out_dtype,
                              void* const* d_out_all, void* const* d_ws_all, size_t ws_bytes) {
  if (nranks != ga::NR || (unsigned)rank >= (unsigned)ga::NR || n_in < 2 || ws_bytes < ga::WS_NEED ||
      in_sizes[0] != ga::MLOC * ga::KDIM || in_sizes[1] != ga::KDIM * ga::NDIM || out_size != 4096 * ga::NSH) {
    fprintf(stderr, "kernel_launch: unexpected problem shape\n");
    return;
  }
  ga::Params p;
  p.X = d_in[0]; p.W = d_in[1];
  for (int i = 0; i < ga::NR; ++i) { p.out[i] = (char*)d_out_all[i]; p.ws[i] = (char*)d_ws_all[i]; }
  p.rank = rank; p.pad = 0; p.pace = ga::PACE;
  ga::build_table(p.tab, rank);
  hipLaunchKernelGGL(ga::gemm_a2a, dim3(ga::NWG), dim3(512), 0, stream, p);
}
```

```cpp
#include <hip/hip_runtime.h>
#include <rccl/rccl.h>
#include <cstdio>
#pragma clang diagnostic ignored "-Wint-to-pointer-cast"

namespace ga {
constexpr int MLOC = 512, KDIM = 4096, NDIM = 16384, NR = 8, NSH = 2048;
constexpr int BM = 128, BN = 256, BK = 64, NS = 3;
constexpr int XB = 16384, STAGE = 3 * XB, STG = NS * STAGE;
constexpr int LDS_BYTES = STG + 16384;
constexpr int NWG = 256, ROUNDS = 4, RSTEPS = 16, TSTEPS = ROUNDS * RSTEPS;
constexpr unsigned X_BYTES = MLOC * KDIM * 2u, W_BYTES = (unsigned)KDIM * NDIM * 2u, OUT_BYTES = 4096u * NSH * 2u;
constexpr unsigned long long DEADLINE = 100000000ull;
constexpr unsigned long long PACE = 0x006068686000ull;
constexpr unsigned WS_DONE = 0;
constexpr unsigned WS_REG = 2048;
constexpr unsigned WS_ARR = 4096;
constexpr unsigned WS_CNT = 8192;
constexpr unsigned WS_TMO = 16384;
constexpr unsigned WS_AGF = 32768;
constexpr unsigned WS_DONE2 = 49152;
constexpr unsigned WS_XF = 131072;
constexpr unsigned WS_XD = 1u << 20, XD_WG = 49152u, XD_BUF = NWG * XD_WG;
constexpr unsigned WS_G = 64u << 20, G_SRC = 1u << 20, G_BUF = NR * G_SRC;
constexpr unsigned WS_NEED = WS_G + 2 * G_BUF;
static_assert(WS_XD + ROUNDS * XD_BUF <= WS_G, "ws");
static_assert(LDS_BYTES <= 160 * 1024, "LDS");

typedef __attribute__((address_space(3))) char lds_char;
typedef short s16x4 __attribute__((ext_vector_type(4)));
typedef short s16x2 __attribute__((ext_vector_type(2)));
typedef __bf16 bf16x8 __attribute__((ext_vector_type(8)));
typedef __bf16 bf16x2 __attribute__((ext_vector_type(2)));
typedef float f32x4 __attribute__((ext_vector_type(4)));
typedef float f32x2 __attribute__((ext_vector_type(2)));
typedef unsigned u32x4 __attribute__((ext_vector_type(4)));
typedef __amdgpu_buffer_rsrc_t rsrc_t;
typedef __attribute__((address_space(3))) s16x4* lp4;
typedef const __attribute__((address_space(3))) bf16x8* lp8;
typedef volatile __attribute__((address_space(3))) u32x4* lpq;
typedef __attribute__((address_space(1))) unsigned* gptr_u;

struct Params {
  const void* X; const void* W; char* myws;
  int rank; int pad; unsigned long long pace, pad2;
  char* out[NR]; char* ws[NR];
  u32x4 tab[64];
};

#define WAIT_V(n) asm volatile("s_waitcnt vmcnt(%0)" ::"n"(n) : "memory")
#define WAIT_L(n) asm volatile("s_waitcnt lgkmcnt(%0)" ::"n"(n) : "memory")
#define BARRIER() do { asm volatile("" ::: "memory"); __builtin_amdgcn_s_barrier(); asm volatile("" ::: "memory"); } while (0)
#define SCHED_FENCE() __builtin_amdgcn_sched_barrier(0)
#define CFENCE() asm volatile("" ::: "memory")
#define AINL __attribute__((always_inline))
#define PIN(x) asm volatile("" : "+v"(x))
#define ST_SYS(ptr, val) __hip_atomic_store((ptr), (val), __ATOMIC_RELAXED, __HIP_MEMORY_SCOPE_SYSTEM)
constexpr int AUX_SYS = 17;
constexpr int AUX_NT = 2;

__device__ __forceinline__ rsrc_t mkrs(const void* p, unsigned bytes) { return __builtin_amdgcn_make_buffer_rsrc((void*)p, 0, bytes, 0x00020000); }

struct Tile { int xsrc;   int xrow, wcol, peer, orow, ocol, rot, blk, ph; };
__device__ __forceinline__ Tile tile_dec(unsigned e, int rank) {
  Tile t;
  const int src = (e >> 8) & 7, rt = (e >> 6) & 3;
  t.peer = e & 7; t.ocol = ((e >> 3) & 7) * BN;
  t.xsrc = src == rank ? -1 : src; t.xrow = src == rank ? rt * BM : 0; t.orow = src * MLOC + rt * BM;
  t.rot = (e >> 12) & 15; t.blk = (e >> 16) & 15; t.ph = (e >> 20) & 15;
  t.wcol = t.peer * NSH + t.ocol;
  return t;
}
#define TILE_OF(r) tile_dec((r) == 0 ? te[0] : (r) == 1 ? te[1] : (r) == 2 ? te[2] : te[3], rank)

__device__ __forceinline__ unsigned pack2(float a, float b) {
  f32x2 v = {a, b};
  return __builtin_bit_cast(unsigned, __builtin_convertvector(v, bf16x2));
}
__device__ __forceinline__ unsigned relu2(unsigned x) {
  s16x2 s = __builtin_bit_cast(s16x2, x), z = {0, 0};
  return __builtin_bit_cast(unsigned, __builtin_elementwise_max(s, z));
}
__device__ __forceinline__ float unpk(unsigned d, int hi) { return __builtin_bit_cast(float, hi ? (d & 0xffff0000u) : (d << 16)); }
__device__ __forceinline__ int vm_outstanding() { int v; asm volatile("s_getreg_b32 %0, hwreg(HW_REG_IB_STS, 0, 4)" : "=s"(v)); return v; }
__device__ __forceinline__ unsigned xcc_id() { unsigned v; asm volatile("s_getreg_b32 %0, hwreg(HW_REG_XCC_ID, 0, 3)" : "=s"(v)); return v; }
__device__ __forceinline__ bool all_ge(const u32x4& f, unsigned v) { return __all(f[0] >= v && f[1] >= v && f[2] >= v && f[3] >= v); }
template <int N> struct IC { static constexpr int v = N; };

__device__ __forceinline__ bool give_up(char* myws, unsigned call, unsigned long long t0) {
  if (__hip_atomic_load((unsigned*)(myws + WS_TMO), __ATOMIC_RELAXED, __HIP_MEMORY_SCOPE_AGENT) == call ||
      __builtin_amdgcn_s_memrealtime() - t0 > DEADLINE) {
    __hip_atomic_store((unsigned*)(myws + WS_TMO), call, __ATOMIC_RELAXED, __HIP_MEMORY_SCOPE_AGENT);
    return true;
  }
  return false;
}
__device__ __forceinline__ bool src_done(unsigned v, unsigned call, int lane) {
  const unsigned long long ge = __ballot((v >> 8) >= call), gt = __ballot((v >> 8) > call);
  unsigned m = (v >> 8) == call ? (v & 255u) : 0u;
  m |= __shfl_xor(m, 1); m |= __shfl_xor(m, 2); m |= __shfl_xor(m, 4);
  const unsigned geb = (unsigned)(ge >> (lane & 56)) & 255u, gtb = (unsigned)(gt >> (lane & 56)) & 255u;
  return gtb != 0u || (m != 0u && (m & ~geb) == 0u);
}

__global__ void __launch_bounds__(512, 2) gemm_a2a(Params p) {
  __shared__ __attribute__((aligned(1024))) char smem[LDS_BYTES];
  lds_char* lds = (lds_char*)smem;
  const unsigned lbase = (unsigned)(size_t)lds;
  const int tid = threadIdx.x, lane = tid & 63;
  const int wid = __builtin_amdgcn_readfirstlane(tid >> 6);
  const int b = blockIdx.x, q = b & 3, rank = p.rank & 7;
  char* myws = p.myws;
  const u32x4 te = p.tab[((b >> 2) & 1) * 32 + ((b >> 3) & 31)];
  auto load_call = [&]() AINL {
    unsigned n;
    asm volatile("s_load_dword %0, %1, 0x0\n\ts_waitcnt lgkmcnt(0)" : "=s"(n) : "s"(myws + WS_CNT + b * 4) : "memory");
    return n + 1u;
  };
  const unsigned lane16 = lane * 16;

  if (__builtin_expect(wid >= 4 && wid < 7, 1)) {
    const bool isx = wid == 4;
    unsigned v[2];
#pragma unroll
    for (int e = 0; e < 2; ++e) {
      const unsigned vx = (unsigned)(lane >> 3) * (KDIM * 2) + (unsigned)(((lane & 7) ^ ((4 * e + (lane >> 4)) & 7)) * 16);
      const int c16 = lane & 15, f = (lane >> 4) | (e << 2);
      const unsigned vw = (unsigned)(lane >> 4) * (NDIM * 2) + (unsigned)((((c16 >> 1) ^ f) * 32) + (c16 & 1) * 16);
      v[e] = isx ? vx : vw;
    }
    rsrc_t rs = mkrs(p.W, W_BYTES);
    unsigned sbase = 0, call = 0;
    int slot = 0, ks = 0, rnd = 0, kb = 0, ki = 0, blk = 1, cur = 1, rot = 0, nleft = 0;
    auto issue = [&]() AINL {
      if (ks == 0) {
        const Tile t = TILE_OF(rnd);
        blk = t.blk; rot = t.rot; cur = t.ph ? t.ph : blk;
        kb = 0; nleft = cur; ki = rot < cur ? rot : 0;
        if (isx) {
          rs = t.xsrc < 0 ? mkrs(p.X, X_BYTES) : mkrs(myws + WS_G + (call & 1) * G_BUF + t.xsrc * G_SRC, G_SRC);
          sbase = (unsigned)t.xrow * (KDIM * 2) + (unsigned)q * 2048u;
        } else {
          sbase = (unsigned)(q * 1024) * (NDIM * 2) + (unsigned)(t.wcol + (wid - 5) * 128) * 2u;
        }
      }
      const int kse = kb + ki;
      if (++ki == cur) ki = 0;
      if (--nleft == 0) { kb += cur; cur = RSTEPS - kb < blk ? RSTEPS - kb : blk; nleft = cur; ki = rot < cur ? rot : 0; }
      lds_char* base = lds + slot * STAGE + (wid - 4) * XB;
      if (isx) {
        const unsigned s = sbase + (unsigned)kse * (BK * 2);
        if (rnd == ROUNDS - 1) {
#pragma unroll
          for (int i = 0; i < 16; ++i)
            __builtin_amdgcn_raw_ptr_buffer_load_lds(rs, base + i * 1024, 16, v[i & 1],
                                                     s + (unsigned)(32 * (((i >> 2) + q) & 3) + 8 * (i & 3)) * (KDIM * 2), 0, AUX_SYS);
        } else {
#pragma unroll
          for (int i = 0; i < 16; ++i)
            __builtin_amdgcn_raw_ptr_buffer_load_lds(rs, base + i * 1024, 16, v[i & 1],
                                                     s + (unsigned)(32 * (((i >> 2) + q) & 3) + 8 * (i & 3)) * (KDIM * 2), 0, 0);
        }
      } else {
        const unsigned s = sbase + (unsigned)kse * (BK * NDIM * 2);
#pragma unroll
        for (int i = 0; i < 16; ++i)
          __builtin_amdgcn_raw_ptr_buffer_load_lds(rs, base + i * 1024, 16, v[(i >> 1) & 1], s + i * 4 * (NDIM * 2), 0, 0);
      }
      slot = (slot + 1 == NS) ? 0 : slot + 1;
      if (++ks == RSTEPS) { ks = 0; ++rnd; }
    };
#pragma unroll 1
    for (int s = 0; s < NS; ++s) {
      issue();
      if (s == 0) { WAIT_V(0); BARRIER(); }
    }
    if (isx) call = load_call();
    unsigned tnext = (unsigned)__builtin_amdgcn_s_memrealtime();
#pragma unroll 1
    for (int t = 0; t < TSTEPS - NS; ++t) {
      WAIT_V(16);
      if (isx) {
        tnext += (unsigned)(p.pace >> (t & 56)) & 255u;
        unsigned now = (unsigned)__builtin_amdgcn_s_memrealtime();
        while ((int)(now - tnext) < 0) now = (unsigned)__builtin_amdgcn_s_memrealtime();
        tnext = now;
      }
      BARRIER();
      issue();
    }
    WAIT_V(16); BARRIER();
    WAIT_V(0); BARRIER();
    if (!isx) { BARRIER(); return; }
    const unsigned xcc = xcc_id();
    rsrc_t rreg = mkrs(myws + WS_REG, NWG * 4), rarr = mkrs(myws + WS_ARR, NWG * 4);
    u32x4 rg = __builtin_amdgcn_raw_buffer_load_b128(rreg, lane16, 0, AUX_SYS);
    u32x4 ar = __builtin_amdgcn_raw_buffer_load_b128(rarr, lane16, 0, AUX_SYS);
    char* pw = p.ws[lane & 7];
    BARRIER();
    const unsigned long long t0 = __builtin_amdgcn_s_memrealtime();
    while (!__all((rg[0] >> 4) == call && (rg[1] >> 4) == call && (rg[2] >> 4) == call && (rg[3] >> 4) == call)) {
      if (give_up(myws, call, t0)) return;
      CFENCE();
      rg = __builtin_amdgcn_raw_buffer_load_b128(rreg, lane16, 0, AUX_SYS);
    }
    bool lower = false; unsigned mask = 0;
#pragma unroll
    for (int k = 0; k < 4; ++k) lower |= (rg[k] & 7u) == xcc && lane * 4 + k < b;
    if (__any(lower)) return;
#pragma unroll
    for (unsigned x = 0; x < 8; ++x)
      if (__any((rg[0] & 7u) == x || (rg[1] & 7u) == x || (rg[2] & 7u) == x || (rg[3] & 7u) == x)) mask |= 1u << x;
    for (;;) {
      bool ok = true;
#pragma unroll
      for (int k = 0; k < 4; ++k) ok &= (rg[k] & 7u) != xcc || ar[k] >= call;
      if (__all(ok)) break;
      if (give_up(myws, call, t0)) return;
      CFENCE();
      ar = __builtin_amdgcn_raw_buffer_load_b128(rarr, lane16, 0, AUX_SYS);
    }
    __builtin_amdgcn_fence(__ATOMIC_RELEASE, "");
    WAIT_V(0);
    if (lane < NR && lane != rank) ST_SYS((unsigned*)(pw + WS_DONE + rank * 128) + xcc, (call << 8) | mask);
    return;
  }

  if (wid == 7) {
    int t = -1;
    const unsigned call = load_call();
    auto step = [&]() AINL { if (t < TSTEPS - 1) { BARRIER(); ++t; } };
    const unsigned st_voff = (unsigned)(lane & 7) * (NSH * 2) + (unsigned)(((lane >> 4) & 1) * 2 + (lane >> 5)) * 16u + (unsigned)(lane & 8) * 8u;
    rsrc_t rxd = mkrs(myws + WS_XD, (ROUNDS - 1) * XD_BUF), rxf = mkrs(myws + WS_XF + b * 64, 48);
    const unsigned goff = (unsigned)b * 4096u;
    const bool flane = lane < NR && lane != rank;
    size_t fptr = (size_t)(p.ws[lane & 7] + (rank * NWG + b) * 4);
    u32x4 d[4], ow[16], in[8][3], o, oprev, ob, agf;
    unsigned xfv = 0;
    int agn = 0;
    auto ag_store = [&](const u32x4& dd, int j) AINL {
      const int peer = (rank + 1 + (agn + b) % 7) & 7;
      rsrc_t rd = mkrs(p.ws[peer] + WS_G + (call & 1) * G_BUF + rank * G_SRC, agn < 28 ? G_SRC : 0u);
      __builtin_amdgcn_raw_buffer_store_b128(dd, rd, lane16, goff + j * 1024, AUX_SYS);
      ++agn;
    };
    auto xf_load = [&]() AINL { CFENCE(); xfv = __builtin_amdgcn_raw_buffer_load_b32(rxf, lane < 12 ? lane * 4 : 0, 0, AUX_SYS); CFENCE(); };
    auto xf_check = [&](unsigned seq) AINL {
      if (__builtin_expect(!__all(xfv >= seq), 0)) {
        const unsigned long long t0 = __builtin_amdgcn_s_memrealtime();
        for (;;) {
          xf_load();
          if (__all(xfv >= seq)) break;
          if (give_up(myws, call, t0)) break;
        }
      }
      CFENCE();
    };
    auto in_load = [&](int r, int i) AINL {
#pragma unroll
      for (int k = 0; k < 3; ++k)
        in[i & 7][k] = __builtin_amdgcn_raw_buffer_load_b128(rxd, lane16, r * XD_BUF + (unsigned)b * XD_WG + k * 16384u + i * 1024u, AUX_SYS);
    };
    auto ow_load = [&]() AINL {
#pragma unroll
      for (int i = 0; i < 16; ++i) ow[i] = *(lpq)(lbase + STG + i * 1024 + lane16);
      WAIT_L(0);
    };
    const unsigned seq0 = (call - 1u) * ROUNDS + 1u;
    {
      rsrc_t rx = mkrs(p.X, X_BYTES);
#pragma unroll
      for (int j = 0; j < 4; ++j) d[j] = __builtin_amdgcn_raw_buffer_load_b128(rx, lane16, 3u * BM * KDIM * 2 + goff + j * 1024, 0);
      BARRIER();
      if (lane == 0) ST_SYS((unsigned*)(myws + WS_REG) + b, (call << 4) | xcc_id());
      PIN(fptr);
      WAIT_V(0);
#pragma unroll
      for (int j = 0; j < 3; ++j) {
#pragma unroll 1
        for (int k = 0; k < (j < 2 ? 7 : 6); ++k) { ag_store(d[j], j); step(); }
      }
      xf_load(); ag_store(d[2], 2); step();
      xf_check(seq0); in_load(0, 0); ag_store(d[3], 3); step();
      in_load(0, 1); ag_store(d[3], 3); ow_load(); step();
    }
    const int agsrc = TILE_OF(ROUNDS - 1).xsrc;
#pragma unroll 1
    for (int r = 0; r < ROUNDS - 1; ++r) {
      const Tile tl = TILE_OF(r);
      const bool more = r < ROUNDS - 2;
      rsrc_t rd = mkrs(p.out[tl.peer], more ? 0u : OUT_BYTES), rdn = mkrs(p.out[tl.peer], more ? OUT_BYTES : 0u);
      auto ostore = [&](const u32x4& dd, unsigned so_) AINL {
        __builtin_amdgcn_raw_buffer_store_b128(dd, rd, st_voff, so_, AUX_SYS);
        __builtin_amdgcn_raw_buffer_store_b128(dd, rdn, st_voff, so_, AUX_NT);
      };
      const unsigned so = (unsigned)(tl.orow + 32 * q) * (NSH * 2) + (unsigned)tl.ocol * 2u;
      const bool agchk = r == 1 && agsrc >= 0;
      rsrc_t raf = mkrs(myws + WS_AGF + (agsrc & 7) * (NWG * 4), agchk ? NWG * 4 : 0);
#pragma unroll
      for (int i = 0; i < 16; ++i) {
        if (i == 0 && r == 2) {
          WAIT_V(0);
          if (lane == 0) ST_SYS((unsigned*)(myws + WS_ARR) + b, call);
        }
        if (i == 12 && r == 0) {
          WAIT_V(0);
          if (flane) ST_SYS((gptr_u)(fptr + WS_AGF), call);
        }
        if (i == 13) xf_load();
        if (i == 5) { agf = __builtin_amdgcn_raw_buffer_load_b128(raf, lane16, 0, AUX_SYS); CFENCE(); }
        {
          float v[8];
#pragma unroll
          for (int w = 0; w < 8; ++w) {
            float a_ = unpk(ow[i][w >> 1], w & 1); PIN(a_);
#pragma unroll
            for (int k = 0; k < 3; ++k) { float u_ = unpk(in[i & 7][k][w >> 1], w & 1); PIN(u_); a_ += u_; PIN(a_); }
            v[w] = a_;
          }
          unsigned d0 = pack2(v[0], v[1]); PIN(d0); unsigned d1 = pack2(v[2], v[3]); PIN(d1);
          unsigned e0 = pack2(v[4], v[5]); PIN(e0); unsigned e1 = pack2(v[6], v[7]); PIN(e1);
          d0 = relu2(d0); PIN(d0); d1 = relu2(d1); PIN(d1); e0 = relu2(e0); PIN(e0); e1 = relu2(e1); PIN(e1);
          auto s0 = __builtin_amdgcn_permlane16_swap(d0, e0, false, false);
          auto s1 = __builtin_amdgcn_permlane16_swap(d1, e1, false, false);
          o = (u32x4){s0[0], s1[0], s0[1], s1[1]};
          PIN(o);
        }
        if (i == 6) {
          if (__builtin_expect(agchk && !all_ge(agf, call), 0)) {
            const unsigned long long t0 = __builtin_amdgcn_s_memrealtime();
            for (;;) {
              CFENCE();
              agf = __builtin_amdgcn_raw_buffer_load_b128(raf, lane16, 0, AUX_SYS);
              if (all_ge(agf, call)) break;
              if (give_up(myws, call, t0)) break;
            }
          }
          CFENCE();
        }
        if (i == 14) xf_check(more ? seq0 + r + 1u : 0u);
        if (i < 6) { in_load(r, 2 * i + 2); in_load(r, 2 * i + 3); }
        else if (i < 8) in_load(r, i + 8);
        else if (i >= 14) in_load(r + 1, i - 14);
        CFENCE();
        if (i & 1) {
          u32x4 oa;
#pragma unroll
          for (int k = 0; k < 4; ++k) {
            oa[k] = __builtin_amdgcn_update_dpp(oprev[k], o[k], 0x128  , 0xf, 0xc, false); PIN(oa);
            ob[k] = __builtin_amdgcn_update_dpp(o[k], oprev[k], 0x128, 0xf, 0x3, false); PIN(ob);
          }
          ostore(oa, so + (unsigned)((i >> 1) & 1) * (16 * NSH * 2) + (i >> 2) * 128u);
          if (i == 15) ostore(ob, so + (unsigned)(16 + 8) * (NSH * 2) + 3 * 128u);
        } else {
          oprev = o; PIN(oprev);
          if (i > 0) ostore(ob, so + (unsigned)((((i - 1) >> 1) & 1) * 16 + 8) * (NSH * 2) + ((i - 1) >> 2) * 128u);
        }
        if (i < 10 && !(i & 1)) ag_store(d[3], 3);
        if (i == 15 && more) ow_load();
        SCHED_FENCE(); asm volatile("s_nop 1" ::: "memory"); SCHED_FENCE();
        if (more || i < 4 || (i & 1)) step();
      }
    }
    while (t < TSTEPS - 1 && vm_outstanding() > 0) step();
    WAIT_V(0);
    if (flane) ST_SYS((gptr_u)(fptr + WS_DONE2), call);
    while (t < TSTEPS - 1) step();
    return;
  }

  const unsigned call = load_call();
  const int c = lane & 15, g = lane >> 4;
  unsigned ax0 = lbase + c * 128 + ((g ^ (c >> 1)) * 16), ax1 = 0, aw[4];
  {
    const int qq = c >> 2, pp = c & 3;
#pragma unroll
    for (int jn = 0; jn < 4; ++jn)
      aw[jn] = lbase + XB * (1 + (wid >> 1)) + (8 * g + qq) * 256 + (((4 * (wid & 1) + jn) ^ (qq | ((g & 1) << 2))) * 32) + 8 * pp;
  }
  f32x4 acc[8][4];
  bf16x8 x0[8], x1[8], wa[4], wb[4];
  u32x4 rb[12];
#pragma unroll
  for (int mi = 0; mi < 8; ++mi)
#pragma unroll
    for (int jn = 0; jn < 4; ++jn) acc[mi][jn] = (f32x4){0.f, 0.f, 0.f, 0.f};

  s16x4 wlo;
#define RD_WL(KK, jn) wlo = __builtin_amdgcn_ds_read_tr16_b64_v4i16((lp4)(aw[jn] + (KK) * 8192))
#define RD_WH(D, KK, jn) D[jn] = __builtin_bit_cast(bf16x8, __builtin_shufflevector(wlo, (s16x4)__builtin_amdgcn_ds_read_tr16_b64_v4i16((lp4)(aw[jn] + (KK) * 8192 + 1024)), 0, 1, 2, 3, 4, 5, 6, 7))
#define RD_X0(mi) x0[mi] = *(lp8)(ax0 + (mi) * 2048)
#define RD_X1(mi) x1[mi] = *(lp8)(ax1 + (mi) * 2048)
#define MFMA(ACC, A, B) asm volatile("v_mfma_f32_16x16x32_bf16 %0, %1, %2, %0" : "+v"(ACC) : "v"(A), "v"(B))
#define MFMA0(ACC, A, B) asm volatile("v_mfma_f32_16x16x32_bf16 %0, %1, %2, 0" : "+v"(ACC) : "v"(A), "v"(B))
#define SLOT(G, S, mi, XV, WV, ZZ, FILL) do { \
    if constexpr (ZZ) MFMA0(acc[mi][S], WV[S], XV[mi]); else MFMA(acc[mi][S], WV[S], XV[mi]); \
    SCHED_FENCE(); FILL; extra(KIND_, IC<G>{}, IC<S>{}); SCHED_FENCE(); } while (0)
#define GROUP(G, mi, XV, WV, ZZ, F0, F1, F2, F3) do { \
    SLOT(G, 0, mi, XV, WV, ZZ, F0); SLOT(G, 1, mi, XV, WV, ZZ, F1); SLOT(G, 2, mi, XV, WV, ZZ, F2); SLOT(G, 3, mi, XV, WV, ZZ, F3); } while (0)
#define NOP_ (void)0
#define STPAD() do { SCHED_FENCE(); asm volatile("s_nop 1"); SCHED_FENCE(); } while (0)

  const int gb = b & ~3;
  const rsrc_t rxd = mkrs(myws + WS_XD, ROUNDS * XD_BUF);
  rsrc_t rxf = mkrs(myws + WS_XF, 0);
  unsigned xd_buf = 0, seq = 0;
  const unsigned stg_addr = lbase + STG + wid * 4096 + lane16;
  auto opaque_l16 = [&]() AINL { unsigned l = lane16; PIN(l); return l; };
  auto pack_part = [&](auto N_) AINL -> u32x4 {
    constexpr int n = decltype(N_)::v, mi = n >> 1, h = n & 1;
    const f32x4 v0 = acc[mi][2 * h], v1 = acc[mi][2 * h + 1];
    return (u32x4){pack2(v0[0], v0[1]), pack2(v0[2], v0[3]), pack2(v1[0], v1[1]), pack2(v1[2], v1[3])};
  };
  auto store_part = [&](auto N_, u32x4 dd) AINL {
    constexpr int n = decltype(N_)::v, u = n >> 2;
    const unsigned so = xd_buf + (unsigned)(gb + ((q + u) & 3)) * XD_WG + (unsigned)((u - 1) * 4 + wid) * 4096u + (n & 3) * 1024u;
    __builtin_amdgcn_raw_buffer_store_b128(dd, rxd, lane16, so, AUX_SYS);
    SCHED_FENCE(); asm volatile("s_nop 0");
  };
  auto stage_own = [&](auto J_) AINL { *(lpq)(stg_addr + decltype(J_)::v * 1024) = pack_part(J_); };
  auto send_flags = [&]() AINL {
    WAIT_V(0);
    const unsigned l = opaque_l16() >> 4;
    const unsigned vo = (l >= 1 && l < 4) ? (((q + l) & 3) * 16 + (l - 1) * 4) * 4 : 0x80000000u;
    __builtin_amdgcn_raw_buffer_store_b32(seq, rxf, vo, (unsigned)(gb * 16 + wid) * 4u, AUX_SYS);
  };

  int slot = 0;
  auto extra = [&](auto KIND_, auto G_, auto S_) AINL {
    constexpr int KIND = decltype(KIND_)::v, SL = decltype(G_)::v * 4 + decltype(S_)::v;
#define PK(sl, n) if constexpr (SL == sl) { rb[n - 4] = pack_part(IC<n>{}); PIN(rb[n - 4]); }
#define ST(sl, n) if constexpr (SL == sl) store_part(IC<n>{}, rb[n - 4]);
    if constexpr (KIND == 5) {
      PK(39, 4) ST(41, 4) PK(42, 5) PK(43, 6) PK(47, 7) ST(49, 5) ST(57, 6)
      if constexpr (SL == 62) stage_own(IC<0>{});
      if constexpr (SL == 63) stage_own(IC<1>{});
    }
    if constexpr (KIND == 6) {
      PK(38, 4) ST(39, 4) PK(40, 5) ST(41, 5) PK(42, 6) ST(43, 6) PK(44, 7) ST(45, 7) PK(46, 8) ST(47, 8) PK(48, 9) ST(49, 9)
      PK(50, 10) ST(51, 10) PK(52, 11) ST(53, 11) PK(54, 12) ST(55, 12) PK(56, 13) ST(57, 13) PK(58, 14) ST(59, 14) PK(60, 15) ST(61, 15)
    }
    if constexpr (KIND == 1) {
      if constexpr (SL == 2) stage_own(IC<2>{});
      if constexpr (SL == 3) stage_own(IC<3>{});
      ST(1, 7) PK(5, 8) PK(6, 9) PK(7, 10) ST(9, 8) PK(11, 11) PK(15, 12) ST(17, 9) PK(19, 13) PK(23, 14) ST(25, 10) PK(26, 15)
      ST(33, 11) ST(41, 12) ST(49, 13) ST(56, 14) ST(63, 15)
    }
  };
  auto kstep = [&](auto KIND_) AINL {
    constexpr bool Z = decltype(KIND_)::v == 1 || decltype(KIND_)::v == 2;
    slot = (slot + 1 == NS) ? 0 : slot + 1;
    const unsigned dlt = slot == 0 ? (unsigned)(-(NS - 1) * STAGE) : (unsigned)STAGE;
    if constexpr (Z) asm volatile("s_nop 3");
    SCHED_FENCE();
    GROUP(0, 0, x0, wa, Z, RD_X0(3), NOP_, NOP_, NOP_);
    GROUP(1, 1, x0, wa, Z, RD_X0(4), NOP_, NOP_, NOP_);
    GROUP(2, 2, x0, wa, Z, RD_X0(5), RD_WL(1, 0), RD_WH(wb, 1, 0), NOP_);
    GROUP(3, 3, x0, wa, Z, RD_X0(6), RD_WL(1, 1), RD_WH(wb, 1, 1), NOP_);
    GROUP(4, 4, x0, wa, Z, RD_X0(7), RD_WL(1, 2), RD_WH(wb, 1, 2), ax1 = ax0 ^ 64u; PIN(ax1));
    GROUP(5, 5, x0, wa, Z, RD_X1(2), RD_WL(1, 3), RD_WH(wb, 1, 3), NOP_);
    GROUP(6, 6, x0, wa, Z, RD_X1(3), NOP_, NOP_, NOP_);
    GROUP(7, 7, x0, wa, Z, RD_X1(4), NOP_, RD_X1(5), NOP_);
    GROUP(8, 2, x1, wb, false, RD_X1(6), NOP_, NOP_, NOP_);
    GROUP(9, 3, x1, wb, false, RD_X1(7), NOP_, NOP_, NOP_);
    GROUP(10, 4, x1, wb, false, RD_X1(0), NOP_, NOP_, NOP_);
    GROUP(11, 5, x1, wb, false, RD_X1(1), NOP_, NOP_, ax0 += dlt; PIN(ax0));
    GROUP(12, 6, x1, wb, false, aw[0] += dlt; PIN(aw[0]), aw[1] += dlt; PIN(aw[1]), aw[2] += dlt; PIN(aw[2]), aw[3] += dlt; PIN(aw[3]));
    WAIT_L(0);
    BARRIER();
    RD_X0(0);
    SCHED_FENCE();
    GROUP(13, 7, x1, wb, false, RD_WL(0, 0), RD_WH(wa, 0, 0), RD_WL(0, 1), RD_WH(wa, 0, 1));
    GROUP(14, 0, x1, wb, false, RD_WL(0, 2), RD_WH(wa, 0, 2), RD_WL(0, 3), RD_WH(wa, 0, 3));
    GROUP(15, 1, x1, wb, false, RD_X0(1), RD_X0(2), NOP_, NOP_);
  };

  BARRIER();
  RD_X0(0);
#pragma unroll
  for (int jn = 0; jn < 4; ++jn) { RD_WL(0, jn); RD_WH(wa, 0, jn); }
  RD_X0(1); RD_X0(2);
  SCHED_FENCE();
#define KS0() kstep(IC<0>{})
  kstep(IC<2>{});
#pragma unroll 1
  for (int r = 0; r < ROUNDS; ++r) {
    if (__builtin_expect(r > 0, 1)) {
      kstep(IC<5>{});
      kstep(IC<1>{});
    }
    KS0();
    send_flags();
    KS0(); KS0(); KS0(); KS0(); KS0(); KS0(); KS0(); KS0(); KS0(); KS0(); KS0(); KS0(); KS0();
    rxf = mkrs(myws + WS_XF, NWG * 64);
    xd_buf = r * XD_BUF;
    seq = (call - 1u) * ROUNDS + r + 1u;
  }
  kstep(IC<6>{});
  {
    SCHED_FENCE();
    send_flags();
    {
      const unsigned long long t0 = __builtin_amdgcn_s_memrealtime();
      const unsigned l = opaque_l16();
      for (;;) {
        CFENCE();
        const unsigned f = __builtin_amdgcn_raw_buffer_load_b32(rxf, l < 32u ? l : 32u, (unsigned)(b * 16 + wid) * 4u, AUX_SYS);
        if (__all(f >= seq)) break;
        if (give_up(myws, call, t0)) break;
      }
      CFENCE();
    }
    u32x4 r3[3][4];
#pragma unroll
    for (int s = 0; s < 3; ++s)
#pragma unroll
      for (int j = 0; j < 4; ++j)
        r3[s][j] = __builtin_amdgcn_raw_buffer_load_b128(rxd, lane16, xd_buf + (unsigned)b * XD_WG + (unsigned)(s * 4 + wid) * 4096u + j * 1024u, AUX_SYS);
    const bool bad = __hip_atomic_load((unsigned*)(myws + WS_TMO), __ATOMIC_RELAXED, __HIP_MEMORY_SCOPE_AGENT) == call;
    const Tile tl = TILE_OF(ROUNDS - 1);
    rsrc_t rd = mkrs(p.out[tl.peer], OUT_BYTES);
    const unsigned so = (unsigned)(tl.orow + 32 * q) * (NSH * 2) + (unsigned)tl.ocol * 2u;
    const unsigned l = opaque_l16() >> 4, gg = l >> 4;
    const unsigned vo = (l & 15) * (NSH * 2) + (8 * wid + (gg & 1) * 2 + (gg >> 1)) * 16u;
#pragma unroll
    for (int j = 0; j < 4; ++j) {
      f32x4 v[2] = {acc[j >> 1][2 * (j & 1)], acc[j >> 1][2 * (j & 1) + 1]};
#pragma unroll
      for (int s = 0; s < 3; ++s)
#pragma unroll
        for (int w = 0; w < 8; ++w) v[w >> 2][w & 3] += unpk(r3[s][j][w >> 1], w & 1);
      const unsigned d0 = relu2(pack2(v[0][0], v[0][1])), d1 = relu2(pack2(v[0][2], v[0][3]));
      const unsigned e0 = relu2(pack2(v[1][0], v[1][1])), e1 = relu2(pack2(v[1][2], v[1][3]));
      auto s0 = __builtin_amdgcn_permlane16_swap(d0, e0, false, false);
      auto s1 = __builtin_amdgcn_permlane16_swap(d1, e1, false, false);
      u32x4 o = {s0[0], s1[0], s0[1], s1[1]};
      if (bad) o = (u32x4){0x7fc07fc0u, 0x7fc07fc0u, 0x7fc07fc0u, 0x7fc07fc0u};
      __builtin_amdgcn_raw_buffer_store_b128(o, rd, vo, so + (j >> 1) * (16 * NSH * 2) + (j & 1) * 64u, AUX_SYS);
      STPAD();
    }
  }
  if (wid == 0) {
    if (lane == 0) __hip_atomic_store((unsigned*)(myws + WS_CNT) + b, call, __ATOMIC_RELAXED, __HIP_MEMORY_SCOPE_AGENT);
  } else if (wid == 1 && b < NR) {
    rsrc_t frs = mkrs(myws + WS_DONE, NR * 128), frs2 = mkrs(myws + WS_DONE2 + ((rank + b) & 7) * (NWG * 4), NWG * 4);
    const unsigned long long t0 = __builtin_amdgcn_s_memrealtime();
    for (;;) {
      CFENCE();
      bool ok;
      if (b == 0) {
        const unsigned v = __builtin_amdgcn_raw_buffer_load_b32(frs, (lane >> 3) * 128 + (lane & 7) * 4, 0, AUX_SYS);
        ok = __all(src_done(v, call, lane) || (lane >> 3) == rank);
      } else {
        ok = all_ge(__builtin_amdgcn_raw_buffer_load_b128(frs2, lane16, 0, AUX_SYS), call);
      }
      if (ok) break;
      if (give_up(myws, call, t0)) {
        if (lane == 0) *(volatile unsigned short*)(p.out[rank] + (size_t)rank * MLOC * NSH * 2) = 0x7fc0;
        break;
      }
    }
  }
}

static void place(unsigned* tab, int h, int& pos, int peer, int col, int n, const int* src, const int* rt, int gi) {
  int k = 0;
  while (k < n) {
    const int room = 32 - (pos & 31), m = n - k < room ? n - k : room;
    for (int j = 0; j < m; ++j, ++pos) {
      const int ph = m > 1 && m <= 4 ? gi % m : 0;
      tab[(h * 32 + (pos & 31)) * 4 + (pos >> 5)] = (unsigned)(peer | col << 3 | rt[k + j] << 6 | src[k + j] << 8 | j << 12 | m << 16 | ph << 20);
    }
    k += m;
  }
}
static void build_table(unsigned* tab, int rank) {
  for (int h = 0; h < 2; ++h) {
    int pos = 0, src[8], rt[8];
    for (int u = 0; u < 32; ++u) {
      for (int j = 0; j < 3; ++j) { src[j] = rank; rt[j] = j; }
      place(tab, h, pos, u < 28 ? (rank + 1 + (u + 4 * h) % 7) & 7 : rank, h * 4 + (u < 28 ? u / 7 : u - 28), 3, src, rt, u + h);
    }
    for (int c = 0; c < 4; ++c) {
      for (int j = 0; j < 8; ++j) { src[j] = (rank + j) & 7; rt[j] = 3; }
      place(tab, h, pos, rank, h * 4 + c, 8, src, rt, 0);
    }
  }
}
}

extern "C" void kernel_launch(int rank, int nranks, ncclComm_t comm, hipStream_t stream,
                              void* const* d_in, const int* in_sizes, int n_in,
                              void* d_out, int out_size, const int* in_dtypes, int out_dtype,
                              void* const* d_out_all, void* const* d_ws_all, size_t ws_bytes) {
  if (nranks != ga::NR || (unsigned)rank >= (unsigned)ga::NR || n_in < 2 || ws_bytes < ga::WS_NEED ||
      in_sizes[0] != ga::MLOC * ga::KDIM || in_sizes[1] != ga::KDIM * ga::NDIM || out_size != 4096 * ga::NSH) {
    fprintf(stderr, "kernel_launch: unexpected problem shape\n");
    return;
  }
  ga::Params p;
  p.X = d_in[0]; p.W = d_in[1]; p.myws = (char*)d_ws_all[rank];
  for (int i = 0; i < ga::NR; ++i) { p.out[i] = (char*)d_out_all[i]; p.ws[i] = (char*)d_ws_all[i]; }
  p.rank = rank; p.pad = 0; p.pad2 = 0; p.pace = ga::PACE;
  ga::build_table((unsigned*)p.tab, rank);
  hipLaunchKernelGGL(ga::gemm_a2a, dim3(ga::NWG), dim3(512), 0, stream, p);
}
```

```cpp
#include <hip/hip_runtime.h>
#include <rccl/rccl.h>
#include <cstdio>
#pragma clang diagnostic ignored "-Wint-to-pointer-cast"

namespace ga {
constexpr int MLOC = 512, KDIM = 4096, NDIM = 16384, NR = 8, NSH = 2048;
constexpr int BM = 128, BN = 256, BK = 64, NS = 3;
constexpr int XB = 16384, STAGE = 3 * XB, STG = NS * STAGE;
constexpr int LDS_BYTES = STG + 16384;
constexpr int NWG = 256, ROUNDS = 4, RSTEPS = 16, TSTEPS = ROUNDS * RSTEPS;
constexpr unsigned X_BYTES = MLOC * KDIM * 2u, W_BYTES = (unsigned)KDIM * NDIM * 2u, OUT_BYTES = 4096u * NSH * 2u;
constexpr unsigned long long DEADLINE = 100000000ull;
constexpr unsigned long long PACE = 0x005860605800ull;
constexpr unsigned WS_DONE = 0;
constexpr unsigned WS_REG = 2048;
constexpr unsigned WS_ARR = 4096;
constexpr unsigned WS_CNT = 8192;
constexpr unsigned WS_TMO = 16384;
constexpr unsigned WS_AGF = 32768;
constexpr unsigned WS_DONE2 = 49152;
constexpr unsigned WS_XF = 131072;
constexpr unsigned WS_XD = 1u << 20, XD_WG = 49152u, XD_BUF = NWG * XD_WG;
constexpr unsigned WS_G = 64u << 20, G_SRC = 1u << 20, G_BUF = NR * G_SRC;
constexpr unsigned WS_NEED = WS_G + 2 * G_BUF;
static_assert(WS_XD + ROUNDS * XD_BUF <= WS_G, "ws");
static_assert(LDS_BYTES <= 160 * 1024, "LDS");

typedef __attribute__((address_space(3))) char lds_char;
typedef short s16x4 __attribute__((ext_vector_type(4)));
typedef short s16x2 __attribute__((ext_vector_type(2)));
typedef __bf16 bf16x8 __attribute__((ext_vector_type(8)));
typedef __bf16 bf16x2 __attribute__((ext_vector_type(2)));
typedef float f32x4 __attribute__((ext_vector_type(4)));
typedef float f32x2 __attribute__((ext_vector_type(2)));
typedef unsigned u32x4 __attribute__((ext_vector_type(4)));
typedef __amdgpu_buffer_rsrc_t rsrc_t;
typedef __attribute__((address_space(3))) s16x4* lp4;
typedef const __attribute__((address_space(3))) bf16x8* lp8;
typedef volatile __attribute__((address_space(3))) u32x4* lpq;
typedef __attribute__((address_space(1))) unsigned* gptr_u;

struct Params {
  const void* X; const void* W; char* myws;
  int rank; int pad; unsigned long long pace, pad2;
  char* out[NR]; char* ws[NR];
  u32x4 tab[64];
};

#define WAIT_V(n) asm volatile("s_waitcnt vmcnt(%0)" ::"n"(n) : "memory")
#define WAIT_L(n) asm volatile("s_waitcnt lgkmcnt(%0)" ::"n"(n) : "memory")
#define BARRIER() do { asm volatile("" ::: "memory"); __builtin_amdgcn_s_barrier(); asm volatile("" ::: "memory"); } while (0)
#define SCHED_FENCE() __builtin_amdgcn_sched_barrier(0)
#define CFENCE() asm volatile("" ::: "memory")
#define AINL __attribute__((always_inline))
#define PIN(x) asm volatile("" : "+v"(x))
#define ST_SYS(ptr, val) __hip_atomic_store((ptr), (val), __ATOMIC_RELAXED, __HIP_MEMORY_SCOPE_SYSTEM)
constexpr int AUX_SYS = 17;
constexpr int AUX_NT = 2;

__device__ __forceinline__ rsrc_t mkrs(const void* p, unsigned bytes) { return __builtin_amdgcn_make_buffer_rsrc((void*)p, 0, bytes, 0x00020000); }

struct Tile { int xsrc;   int xrow, wcol, peer, orow, ocol, rot, blk, ph; };
__device__ __forceinline__ Tile tile_dec(unsigned e, int rank) {
  Tile t;
  const int src = (e >> 8) & 7, rt = (e >> 6) & 3;
  t.peer = e & 7; t.ocol = ((e >> 3) & 7) * BN;
  t.xsrc = src == rank ? -1 : src; t.xrow = src == rank ? rt * BM : 0; t.orow = src * MLOC + rt * BM;
  t.rot = (e >> 12) & 15; t.blk = (e >> 16) & 15; t.ph = (e >> 20) & 15;
  t.wcol = t.peer * NSH + t.ocol;
  return t;
}
#define TILE_OF(r) tile_dec((r) == 0 ? te[0] : (r) == 1 ? te[1] : (r) == 2 ? te[2] : te[3], rank)

__device__ __forceinline__ unsigned pack2(float a, float b) {
  f32x2 v = {a, b};
  return __builtin_bit_cast(unsigned, __builtin_convertvector(v, bf16x2));
}
__device__ __forceinline__ unsigned relu2(unsigned x) {
  s16x2 s = __builtin_bit_cast(s16x2, x), z = {0, 0};
  return __builtin_bit_cast(unsigned, __builtin_elementwise_max(s, z));
}
__device__ __forceinline__ float unpk(unsigned d, int hi) { return __builtin_bit_cast(float, hi ? (d & 0xffff0000u) : (d << 16)); }
__device__ __forceinline__ int vm_outstanding() { int v; asm volatile("s_getreg_b32 %0, hwreg(HW_REG_IB_STS, 0, 4)" : "=s"(v)); return v; }
__device__ __forceinline__ unsigned xcc_id() { unsigned v; asm volatile("s_getreg_b32 %0, hwreg(HW_REG_XCC_ID, 0, 3)" : "=s"(v)); return v; }
__device__ __forceinline__ bool all_ge(const u32x4& f, unsigned v) { return __all(f[0] >= v && f[1] >= v && f[2] >= v && f[3] >= v); }
template <int N> struct IC { static constexpr int v = N; };

__device__ __forceinline__ bool give_up(char* myws, unsigned call, unsigned long long t0) {
  if (__hip_atomic_load((unsigned*)(myws + WS_TMO), __ATOMIC_RELAXED, __HIP_MEMORY_SCOPE_AGENT) == call ||
      __builtin_amdgcn_s_memrealtime() - t0 > DEADLINE) {
    __hip_atomic_store((unsigned*)(myws + WS_TMO), call, __ATOMIC_RELAXED, __HIP_MEMORY_SCOPE_AGENT);
    return true;
  }
  return false;
}
__device__ __forceinline__ bool src_done(unsigned v, unsigned call, int lane) {
  const unsigned long long ge = __ballot((v >> 8) >= call), gt = __ballot((v >> 8) > call);
  unsigned m = (v >> 8) == call ? (v & 255u) : 0u;
  m |= __shfl_xor(m, 1); m |= __shfl_xor(m, 2); m |= __shfl_xor(m, 4);
  const unsigned geb = (unsigned)(ge >> (lane & 56)) & 255u, gtb = (unsigned)(gt >> (lane & 56)) & 255u;
  return gtb != 0u || (m != 0u && (m & ~geb) == 0u);
}

__global__ void __launch_bounds__(512, 2) gemm_a2a(Params p) {
  __shared__ __attribute__((aligned(1024))) char smem[LDS_BYTES];
  lds_char* lds = (lds_char*)smem;
  const unsigned lbase = (unsigned)(size_t)lds;
  const int tid = threadIdx.x, lane = tid & 63;
  const int wid = __builtin_amdgcn_readfirstlane(tid >> 6);
  const int b = blockIdx.x, q = b & 3, rank = p.rank & 7;
  char* myws = p.myws;
  const u32x4 te = p.tab[((b >> 2) & 1) * 32 + ((b >> 3) & 31)];
  auto load_call = [&]() AINL {
    unsigned n;
    asm volatile("s_load_dword %0, %1, 0x0\n\ts_waitcnt lgkmcnt(0)" : "=s"(n) : "s"(myws + WS_CNT + b * 4) : "memory");
    return n + 1u;
  };
  const unsigned lane16 = lane * 16;

  if (__builtin_expect(wid >= 4 && wid < 7, 1)) {
    const bool isx = wid == 4;
    unsigned v[2];
#pragma unroll
    for (int e = 0; e < 2; ++e) {
      const unsigned vx = (unsigned)(lane >> 3) * (KDIM * 2) + (unsigned)(((lane & 7) ^ ((4 * e + (lane >> 4)) & 7)) * 16);
      const int c16 = lane & 15, f = (lane >> 4) | (e << 2);
      const unsigned vw = (unsigned)(lane >> 4) * (NDIM * 2) + (unsigned)((((c16 >> 1) ^ f) * 32) + (c16 & 1) * 16);
      v[e] = isx ? vx : vw;
    }
    rsrc_t rs = mkrs(p.W, W_BYTES);
    unsigned sbase = 0, call = 0;
    int slot = 0, ks = 0, rnd = 0, kb = 0, ki = 0, blk = 1, cur = 1, rot = 0, nleft = 0;
    auto issue = [&]() AINL {
      if (ks == 0) {
        const Tile t = TILE_OF(rnd);
        blk = t.blk; rot = t.rot; cur = t.ph ? t.ph : blk;
        kb = 0; nleft = cur; ki = rot < cur ? rot : 0;
        if (isx) {
          rs = t.xsrc < 0 ? mkrs(p.X, X_BYTES) : mkrs(myws + WS_G + (call & 1) * G_BUF + t.xsrc * G_SRC, G_SRC);
          sbase = (unsigned)t.xrow * (KDIM * 2) + (unsigned)q * 2048u;
        } else {
          sbase = (unsigned)(q * 1024) * (NDIM * 2) + (unsigned)(t.wcol + (wid - 5) * 128) * 2u;
        }
      }
      const int kse = kb + ki;
      if (++ki == cur) ki = 0;
      if (--nleft == 0) { kb += cur; cur = RSTEPS - kb < blk ? RSTEPS - kb : blk; nleft = cur; ki = rot < cur ? rot : 0; }
      lds_char* base = lds + slot * STAGE + (wid - 4) * XB;
      if (isx) {
        const unsigned s = sbase + (unsigned)kse * (BK * 2);
        if (rnd == ROUNDS - 1) {
#pragma unroll
          for (int i = 0; i < 16; ++i)
            __builtin_amdgcn_raw_ptr_buffer_load_lds(rs, base + i * 1024, 16, v[i & 1],
                                                     s + (unsigned)(32 * (((i >> 2) + q) & 3) + 8 * (i & 3)) * (KDIM * 2), 0, AUX_SYS);
        } else {
#pragma unroll
          for (int i = 0; i < 16; ++i)
            __builtin_amdgcn_raw_ptr_buffer_load_lds(rs, base + i * 1024, 16, v[i & 1],
                                                     s + (unsigned)(32 * (((i >> 2) + q) & 3) + 8 * (i & 3)) * (KDIM * 2), 0, 0);
        }
      } else {
        const unsigned s = sbase + (unsigned)kse * (BK * NDIM * 2);
#pragma unroll
        for (int i = 0; i < 16; ++i)
          __builtin_amdgcn_raw_ptr_buffer_load_lds(rs, base + i * 1024, 16, v[(i >> 1) & 1], s + i * 4 * (NDIM * 2), 0, 0);
      }
      slot = (slot + 1 == NS) ? 0 : slot + 1;
      if (++ks == RSTEPS) { ks = 0; ++rnd; }
    };
#pragma unroll 1
    for (int s = 0; s < NS; ++s) {
      issue();
      if (s == 0) { WAIT_V(0); BARRIER(); }
    }
    if (isx) call = load_call();
    unsigned tnext = (unsigned)__builtin_amdgcn_s_memrealtime();
#pragma unroll 1
    for (int t = 0; t < TSTEPS - NS; ++t) {
      WAIT_V(16);
      if (isx) {
        tnext += (unsigned)(p.pace >> (t & 56)) & 255u;
        unsigned now = (unsigned)__builtin_amdgcn_s_memrealtime();
        while ((int)(now - tnext) < 0) now = (unsigned)__builtin_amdgcn_s_memrealtime();
        tnext = now;
      }
      BARRIER();
      issue();
    }
    WAIT_V(16); BARRIER();
    WAIT_V(0); BARRIER();
    if (!isx) { BARRIER(); return; }
    const unsigned xcc = xcc_id();
    rsrc_t rreg = mkrs(myws + WS_REG, NWG * 4), rarr = mkrs(myws + WS_ARR, NWG * 4);
    u32x4 rg = __builtin_amdgcn_raw_buffer_load_b128(rreg, lane16, 0, AUX_SYS);
    u32x4 ar = __builtin_amdgcn_raw_buffer_load_b128(rarr, lane16, 0, AUX_SYS);
    char* pw = p.ws[lane & 7];
    BARRIER();
    const unsigned long long t0 = __builtin_amdgcn_s_memrealtime();
    while (!__all((rg[0] >> 4) == call && (rg[1] >> 4) == call && (rg[2] >> 4) == call && (rg[3] >> 4) == call)) {
      if (give_up(myws, call, t0)) return;
      CFENCE();
      rg = __builtin_amdgcn_raw_buffer_load_b128(rreg, lane16, 0, AUX_SYS);
    }
    bool lower = false; unsigned mask = 0;
#pragma unroll
    for (int k = 0; k < 4; ++k) lower |= (rg[k] & 7u) == xcc && lane * 4 + k < b;
    if (__any(lower)) return;
#pragma unroll
    for (unsigned x = 0; x < 8; ++x)
      if (__any((rg[0] & 7u) == x || (rg[1] & 7u) == x || (rg[2] & 7u) == x || (rg[3] & 7u) == x)) mask |= 1u << x;
    for (;;) {
      bool ok = true;
#pragma unroll
      for (int k = 0; k < 4; ++k) ok &= (rg[k] & 7u) != xcc || ar[k] >= call;
      if (__all(ok)) break;
      if (give_up(myws, call, t0)) return;
      CFENCE();
      ar = __builtin_amdgcn_raw_buffer_load_b128(rarr, lane16, 0, AUX_SYS);
    }
    __builtin_amdgcn_fence(__ATOMIC_RELEASE, "");
    WAIT_V(0);
    if (lane < NR && lane != rank) ST_SYS((unsigned*)(pw + WS_DONE + rank * 128) + xcc, (call << 8) | mask);
    return;
  }

  if (wid == 7) {
    int t = -1;
    const unsigned call = load_call();
    auto step = [&]() AINL { if (t < TSTEPS - 1) { BARRIER(); ++t; } };
    const unsigned st_voff = (unsigned)(lane & 7) * (NSH * 2) + (unsigned)(((lane >> 4) & 1) * 2 + (lane >> 5)) * 16u + (unsigned)(lane & 8) * 8u;
    rsrc_t rxd = mkrs(myws + WS_XD, (ROUNDS - 1) * XD_BUF), rxf = mkrs(myws + WS_XF + b * 64, 48);
    const unsigned goff = (unsigned)b * 4096u;
    const bool flane = lane < NR && lane != rank;
    size_t fptr = (size_t)(p.ws[lane & 7] + (rank * NWG + b) * 4);
    u32x4 d[4], ow[16], in[8][3], o, oprev, ob, agf;
    unsigned xfv = 0;
    int agn = 0;
    auto ag_store = [&](const u32x4& dd, int j) AINL {
      const int peer = (rank + 1 + (agn + b) % 7) & 7;
      rsrc_t rd = mkrs(p.ws[peer] + WS_G + (call & 1) * G_BUF + rank * G_SRC, agn < 28 ? G_SRC : 0u);
      __builtin_amdgcn_raw_buffer_store_b128(dd, rd, lane16, goff + j * 1024, AUX_SYS);
      ++agn;
    };
    auto xf_load = [&]() AINL { CFENCE(); xfv = __builtin_amdgcn_raw_buffer_load_b32(rxf, lane < 12 ? lane * 4 : 0, 0, AUX_SYS); CFENCE(); };
    auto xf_check = [&](unsigned seq) AINL {
      if (__builtin_expect(!__all(xfv >= seq), 0)) {
        const unsigned long long t0 = __builtin_amdgcn_s_memrealtime();
        for (;;) {
          xf_load();
          if (__all(xfv >= seq)) break;
          if (give_up(myws, call, t0)) break;
        }
      }
      CFENCE();
    };
    auto in_load = [&](int r, int i) AINL {
#pragma unroll
      for (int k = 0; k < 3; ++k)
        in[i & 7][k] = __builtin_amdgcn_raw_buffer_load_b128(rxd, lane16, r * XD_BUF + (unsigned)b * XD_WG + k * 16384u + i * 1024u, AUX_SYS);
    };
    auto ow_load = [&]() AINL {
#pragma unroll
      for (int i = 0; i < 16; ++i) ow[i] = *(lpq)(lbase + STG + i * 1024 + lane16);
      WAIT_L(0);
    };
    const unsigned seq0 = (call - 1u) * ROUNDS + 1u;
    {
      rsrc_t rx = mkrs(p.X, X_BYTES);
#pragma unroll
      for (int j = 0; j < 4; ++j) d[j] = __builtin_amdgcn_raw_buffer_load_b128(rx, lane16, 3u * BM * KDIM * 2 + goff + j * 1024, 0);
      BARRIER();
      if (lane == 0) ST_SYS((unsigned*)(myws + WS_REG) + b, (call << 4) | xcc_id());
      PIN(fptr);
      WAIT_V(0);
#pragma unroll
      for (int j = 0; j < 3; ++j) {
#pragma unroll 1
        for (int k = 0; k < (j < 2 ? 7 : 6); ++k) { ag_store(d[j], j); step(); }
      }
      xf_load(); ag_store(d[2], 2); step();
      xf_check(seq0); in_load(0, 0); ag_store(d[3], 3); step();
      in_load(0, 1); ag_store(d[3], 3); ow_load(); step();
    }
    const int agsrc = TILE_OF(ROUNDS - 1).xsrc;
#pragma unroll 1
    for (int r = 0; r < ROUNDS - 1; ++r) {
      const Tile tl = TILE_OF(r);
      const bool more = r < ROUNDS - 2;
      rsrc_t rd = mkrs(p.out[tl.peer], more ? 0u : OUT_BYTES), rdn = mkrs(p.out[tl.peer], more ? OUT_BYTES : 0u);
      auto ostore = [&](const u32x4& dd, unsigned so_) AINL {
        __builtin_amdgcn_raw_buffer_store_b128(dd, rd, st_voff, so_, AUX_SYS);
        __builtin_amdgcn_raw_buffer_store_b128(dd, rdn, st_voff, so_, AUX_NT);
      };
      const unsigned so = (unsigned)(tl.orow + 32 * q) * (NSH * 2) + (unsigned)tl.ocol * 2u;
      const bool agchk = r == 1 && agsrc >= 0;
      rsrc_t raf = mkrs(myws + WS_AGF + (agsrc & 7) * (NWG * 4), agchk ? NWG * 4 : 0);
#pragma unroll
      for (int i = 0; i < 16; ++i) {
        if (i == 0 && r == 2) {
          WAIT_V(0);
          if (lane == 0) ST_SYS((unsigned*)(myws + WS_ARR) + b, call);
        }
        if (i == 12 && r == 0) {
          WAIT_V(0);
          if (flane) ST_SYS((gptr_u)(fptr + WS_AGF), call);
        }
        if (i == 13) xf_load();
        if (i == 5) { agf = __builtin_amdgcn_raw_buffer_load_b128(raf, lane16, 0, AUX_SYS); CFENCE(); }
        {
          float v[8];
#pragma unroll
          for (int w = 0; w < 8; ++w) {
            float a_ = unpk(ow[i][w >> 1], w & 1); PIN(a_);
#pragma unroll
            for (int k = 0; k < 3; ++k) { float u_ = unpk(in[i & 7][k][w >> 1], w & 1); PIN(u_); a_ += u_; PIN(a_); }
            v[w] = a_;
          }
          unsigned d0 = pack2(v[0], v[1]); PIN(d0); unsigned d1 = pack2(v[2], v[3]); PIN(d1);
          unsigned e0 = pack2(v[4], v[5]); PIN(e0); unsigned e1 = pack2(v[6], v[7]); PIN(e1);
          d0 = relu2(d0); PIN(d0); d1 = relu2(d1); PIN(d1); e0 = relu2(e0); PIN(e0); e1 = relu2(e1); PIN(e1);
          auto s0 = __builtin_amdgcn_permlane16_swap(d0, e0, false, false);
          auto s1 = __builtin_amdgcn_permlane16_swap(d1, e1, false, false);
          o = (u32x4){s0[0], s1[0], s0[1], s1[1]};
          PIN(o);
        }
        if (i == 6) {
          if (__builtin_expect(agchk && !all_ge(agf, call), 0)) {
            const unsigned long long t0 = __builtin_amdgcn_s_memrealtime();
            for (;;) {
              CFENCE();
              agf = __builtin_amdgcn_raw_buffer_load_b128(raf, lane16, 0, AUX_SYS);
              if (all_ge(agf, call)) break;
              if (give_up(myws, call, t0)) break;
            }
          }
          CFENCE();
        }
        if (i == 14) xf_check(more ? seq0 + r + 1u : 0u);
        if (i < 6) { in_load(r, 2 * i + 2); in_load(r, 2 * i + 3); }
        else if (i < 8) in_load(r, i + 8);
        else if (i >= 14) in_load(r + 1, i - 14);
        CFENCE();
        if (i & 1) {
          u32x4 oa;
#pragma unroll
          for (int k = 0; k < 4; ++k) {
            oa[k] = __builtin_amdgcn_update_dpp(oprev[k], o[k], 0x128  , 0xf, 0xc, false); PIN(oa);
            ob[k] = __builtin_amdgcn_update_dpp(o[k], oprev[k], 0x128, 0xf, 0x3, false); PIN(ob);
          }
          ostore(oa, so + (unsigned)((i >> 1) & 1) * (16 * NSH * 2) + (i >> 2) * 128u);
          if (i == 15) ostore(ob, so + (unsigned)(16 + 8) * (NSH * 2) + 3 * 128u);
        } else {
          oprev = o; PIN(oprev);
          if (i > 0) ostore(ob, so + (unsigned)((((i - 1) >> 1) & 1) * 16 + 8) * (NSH * 2) + ((i - 1) >> 2) * 128u);
        }
        if (i < 10 && !(i & 1)) ag_store(d[3], 3);
        if (i == 15 && more) ow_load();
        SCHED_FENCE(); asm volatile("s_nop 1" ::: "memory"); SCHED_FENCE();
        if (more || i < 4 || (i & 1)) step();
      }
    }
    while (t < TSTEPS - 1 && vm_outstanding() > 0) step();
    WAIT_V(0);
    if (flane) ST_SYS((gptr_u)(fptr + WS_DONE2), call);
    while (t < TSTEPS - 1) step();
    return;
  }

  const unsigned call = load_call();
  const int c = lane & 15, g = lane >> 4;
  unsigned ax0 = lbase + c * 128 + ((g ^ (c >> 1)) * 16), ax1 = ax0 ^ 64u, aw[4];
  {
    const int qq = c >> 2, pp = c & 3;
#pragma unroll
    for (int jn = 0; jn < 4; ++jn)
      aw[jn] = lbase + XB * (1 + (wid >> 1)) + (8 * g + qq) * 256 + (((4 * (wid & 1) + jn) ^ (qq | ((g & 1) << 2))) * 32) + 8 * pp;
  }
  f32x4 acc[8][4];
  bf16x8 xf[2][8], wc[2][4], wn[2][4];
  u32x4 pk;
#pragma unroll
  for (int mi = 0; mi < 8; ++mi)
#pragma unroll
    for (int jn = 0; jn < 4; ++jn) acc[mi][jn] = (f32x4){0.f, 0.f, 0.f, 0.f};

  s16x4 wlo;
#define RD_WL(KK, jn) wlo = __builtin_amdgcn_ds_read_tr16_b64_v4i16((lp4)(aw[jn] + (KK) * 8192))
#define RD_WH(KK, jn) wn[KK][jn] = __builtin_bit_cast(bf16x8, __builtin_shufflevector(wlo, (s16x4)__builtin_amdgcn_ds_read_tr16_b64_v4i16((lp4)(aw[jn] + (KK) * 8192 + 1024)), 0, 1, 2, 3, 4, 5, 6, 7))
#define RD_X0(G) xf[0][G] = *(lp8)(ax0 + (((G) + 2) & 7) * 2048)
#define RD_X1(G) xf[1][G] = *(lp8)(ax1 + (((G) + 2) & 7) * 2048)
#define MFMA(ACC, A, B) asm volatile("v_mfma_f32_16x16x32_bf16 %0, %1, %2, %0" : "+v"(ACC) : "v"(A), "v"(B))
#define MFMA0(ACC, A, B) asm volatile("v_mfma_f32_16x16x32_bf16 %0, %1, %2, 0" : "+v"(ACC) : "v"(A), "v"(B))
#define SLOT(G, S, FILL) do { \
    constexpr int mi_ = ((G) + 2) & 7, jn_ = ((G) & 1) ? 3 - ((S) >> 1) : ((S) >> 1), kk_ = (((S) >> 1) ^ (S)) & 1; \
    if constexpr (Z && !((S) & 1)) MFMA0(acc[mi_][jn_], wc[kk_][jn_], xf[kk_][G]); else MFMA(acc[mi_][jn_], wc[kk_][jn_], xf[kk_][G]); \
    SCHED_FENCE(); FILL; extra(KIND_, IC<(G) * 8 + (S)>{}); SCHED_FENCE(); } while (0)
#define GROUP(G, F0, F1, F2, F3, F4, F5, F6, F7) do { \
    SLOT(G, 0, F0); SLOT(G, 1, F1); SLOT(G, 2, F2); SLOT(G, 3, F3); SLOT(G, 4, F4); SLOT(G, 5, F5); SLOT(G, 6, F6); SLOT(G, 7, F7); } while (0)
#define NOP_ (void)0
#define STPAD() do { SCHED_FENCE(); asm volatile("s_nop 1"); SCHED_FENCE(); } while (0)

  const int gb = b & ~3;
  const rsrc_t rxd = mkrs(myws + WS_XD, ROUNDS * XD_BUF);
  rsrc_t rxf = mkrs(myws + WS_XF, 0);
  unsigned xd_buf = 0, seq = 0;
  const unsigned stg_addr = lbase + STG + wid * 4096 + lane16;
  auto opaque_l16 = [&]() AINL { unsigned l = lane16; PIN(l); return l; };
  auto pack_part = [&](auto N_) AINL -> u32x4 {
    constexpr int n = decltype(N_)::v, mi = n >> 1, h = n & 1;
    const f32x4 v0 = acc[mi][2 * h], v1 = acc[mi][2 * h + 1];
    return (u32x4){pack2(v0[0], v0[1]), pack2(v0[2], v0[3]), pack2(v1[0], v1[1]), pack2(v1[2], v1[3])};
  };
  auto store_part = [&](auto N_, u32x4 dd) AINL {
    constexpr int n = decltype(N_)::v, u = n >> 2;
    const unsigned so = xd_buf + (unsigned)(gb + ((q + u) & 3)) * XD_WG + (unsigned)((u - 1) * 4 + wid) * 4096u + (n & 3) * 1024u;
    __builtin_amdgcn_raw_buffer_store_b128(dd, rxd, lane16, so, AUX_SYS);
    SCHED_FENCE(); asm volatile("s_nop 0");
  };
  auto stage_own = [&](auto J_) AINL { *(lpq)(stg_addr + decltype(J_)::v * 1024) = pack_part(J_); };
  auto send_flags = [&]() AINL {
    WAIT_V(0);
    const unsigned l = opaque_l16() >> 4;
    const unsigned vo = (l >= 1 && l < 4) ? (((q + l) & 3) * 16 + (l - 1) * 4) * 4 : 0x80000000u;
    __builtin_amdgcn_raw_buffer_store_b32(seq, rxf, vo, (unsigned)(gb * 16 + wid) * 4u, AUX_SYS);
  };

  int slot = 0;
  auto extra = [&](auto KIND_, auto SL_) AINL {
    constexpr int KIND = decltype(KIND_)::v, SL = decltype(SL_)::v;
#define PS(sl, n) if constexpr (SL == sl) { pk = pack_part(IC<n>{}); PIN(pk); } if constexpr (SL == sl + 1) store_part(IC<n>{}, pk);
    if constexpr (KIND == 5) { PS(13, 4) PS(21, 5) PS(29, 6) PS(37, 7) PS(45, 8) PS(53, 9) PS(61, 10) }
    if constexpr (KIND == 1) {
      PS(5, 11) PS(13, 12) PS(21, 13) PS(29, 14) PS(36, 15)
      if constexpr (SL == 1) stage_own(IC<0>{});
      if constexpr (SL == 9) stage_own(IC<1>{});
      if constexpr (SL == 17) stage_own(IC<2>{});
      if constexpr (SL == 25) stage_own(IC<3>{});
    }
    if constexpr (KIND == 6) {
      PS(13, 4) PS(16, 5) PS(21, 6) PS(24, 7) PS(29, 8) PS(32, 9) PS(37, 10) PS(40, 11) PS(45, 12) PS(48, 13) PS(53, 14) PS(56, 15)
    }
  };
  auto kstep = [&](auto KIND_) AINL {
    constexpr bool Z = decltype(KIND_)::v == 1 || decltype(KIND_)::v == 2;
    slot = (slot + 1 == NS) ? 0 : slot + 1;
    const unsigned dlt = slot == 0 ? (unsigned)(-(NS - 1) * STAGE) : (unsigned)STAGE;
#pragma unroll
    for (int jn = 0; jn < 4; ++jn) { wc[0][jn] = wn[0][jn]; wc[1][jn] = wn[1][jn]; }
    SCHED_FENCE();
    GROUP(0, RD_X0(2), NOP_, NOP_, NOP_, RD_X1(2), NOP_, NOP_, NOP_);
    GROUP(1, RD_X0(3), NOP_, NOP_, NOP_, RD_X1(3), NOP_, NOP_, NOP_);
    GROUP(2, RD_X0(4), NOP_, NOP_, NOP_, RD_X1(4), NOP_, NOP_, aw[0] += dlt; PIN(aw[0]));
    GROUP(3, RD_X0(5), NOP_, RD_X1(5), aw[1] += dlt; PIN(aw[1]), RD_X0(6), NOP_, NOP_, aw[2] += dlt; PIN(aw[2]));
    GROUP(4, RD_X1(6), NOP_, RD_X0(7), RD_X1(7), NOP_, NOP_, aw[3] += dlt; PIN(aw[3]), ax0 += dlt; PIN(ax0));
    SLOT(5, 0, ax1 += dlt; PIN(ax1));
    SLOT(5, 1, NOP_);
    WAIT_L(0);
    BARRIER();
    RD_X0(0);
    SCHED_FENCE();
    SLOT(5, 2, RD_WL(0, 0)); SLOT(5, 3, RD_WH(0, 0)); SLOT(5, 4, RD_X1(0)); SLOT(5, 5, RD_WL(1, 0)); SLOT(5, 6, RD_WH(1, 0)); SLOT(5, 7, RD_WL(1, 1));
    GROUP(6, RD_WH(1, 1), RD_WL(0, 1), RD_WH(0, 1), RD_WL(0, 2), RD_WH(0, 2), RD_WL(1, 2), RD_WH(1, 2), RD_WL(1, 3));
    GROUP(7, RD_WH(1, 3), RD_WL(0, 3), RD_WH(0, 3), RD_X0(1), RD_X1(1), NOP_, NOP_, NOP_);
  };

  BARRIER();
  RD_X0(0); RD_X1(0);
#pragma unroll
  for (int jn = 0; jn < 4; ++jn) { RD_WL(0, jn); RD_WH(0, jn); RD_WL(1, jn); RD_WH(1, jn); }
  RD_X0(1); RD_X1(1);
  SCHED_FENCE();
#define KS0() kstep(IC<0>{})
  kstep(IC<2>{});
#pragma unroll 1
  for (int r = 0; r < ROUNDS; ++r) {
    if (__builtin_expect(r > 0, 1)) {
      kstep(IC<5>{});
      kstep(IC<1>{});
    }
    KS0();
    send_flags();
    KS0(); KS0(); KS0(); KS0(); KS0(); KS0(); KS0(); KS0(); KS0(); KS0(); KS0(); KS0(); KS0();
    rxf = mkrs(myws + WS_XF, NWG * 64);
    xd_buf = r * XD_BUF;
    seq = (call - 1u) * ROUNDS + r + 1u;
  }
  kstep(IC<6>{});
  {
    SCHED_FENCE();
    send_flags();
    {
      const unsigned long long t0 = __builtin_amdgcn_s_memrealtime();
      const unsigned l = opaque_l16();
      for (;;) {
        CFENCE();
        const unsigned f = __builtin_amdgcn_raw_buffer_load_b32(rxf, l < 32u ? l : 32u, (unsigned)(b * 16 + wid) * 4u, AUX_SYS);
        if (__all(f >= seq)) break;
        if (give_up(myws, call, t0)) break;
      }
      CFENCE();
    }
    u32x4 r3[3][4];
#pragma unroll
    for (int s = 0; s < 3; ++s)
#pragma unroll
      for (int j = 0; j < 4; ++j)
        r3[s][j] = __builtin_amdgcn_raw_buffer_load_b128(rxd, lane16, xd_buf + (unsigned)b * XD_WG + (unsigned)(s * 4 + wid) * 4096u + j * 1024u, AUX_SYS);
    const bool bad = __hip_atomic_load((unsigned*)(myws + WS_TMO), __ATOMIC_RELAXED, __HIP_MEMORY_SCOPE_AGENT) == call;
    const Tile tl = TILE_OF(ROUNDS - 1);
    rsrc_t rd = mkrs(p.out[tl.peer], OUT_BYTES);
    const unsigned so = (unsigned)(tl.orow + 32 * q) * (NSH * 2) + (unsigned)tl.ocol * 2u;
    const unsigned l = opaque_l16() >> 4, gg = l >> 4;
    const unsigned vo = (l & 15) * (NSH * 2) + (8 * wid + (gg & 1) * 2 + (gg >> 1)) * 16u;
#pragma unroll
    for (int j = 0; j < 4; ++j) {
      f32x4 v[2] = {acc[j >> 1][2 * (j & 1)], acc[j >> 1][2 * (j & 1) + 1]};
#pragma unroll
      for (int s = 0; s < 3; ++s)
#pragma unroll
        for (int w = 0; w < 8; ++w) v[w >> 2][w & 3] += unpk(r3[s][j][w >> 1], w & 1);
      const unsigned d0 = relu2(pack2(v[0][0], v[0][1])), d1 = relu2(pack2(v[0][2], v[0][3]));
      const unsigned e0 = relu2(pack2(v[1][0], v[1][1])), e1 = relu2(pack2(v[1][2], v[1][3]));
      auto s0 = __builtin_amdgcn_permlane16_swap(d0, e0, false, false);
      auto s1 = __builtin_amdgcn_permlane16_swap(d1, e1, false, false);
      u32x4 o = {s0[0], s1[0], s0[1], s1[1]};
      if (bad) o = (u32x4){0x7fc07fc0u, 0x7fc07fc0u, 0x7fc07fc0u, 0x7fc07fc0u};
      __builtin_amdgcn_raw_buffer_store_b128(o, rd, vo, so + (j >> 1) * (16 * NSH * 2) + (j & 1) * 64u, AUX_SYS);
      STPAD();
    }
  }
  if (wid == 0) {
    if (lane == 0) __hip_atomic_store((unsigned*)(myws + WS_CNT) + b, call, __ATOMIC_RELAXED, __HIP_MEMORY_SCOPE_AGENT);
  } else if (wid == 1 && b < NR) {
    rsrc_t frs = mkrs(myws + WS_DONE, NR * 128), frs2 = mkrs(myws + WS_DONE2 + ((rank + b) & 7) * (NWG * 4), NWG * 4);
    const unsigned long long t0 = __builtin_amdgcn_s_memrealtime();
    for (;;) {
      CFENCE();
      bool ok;
      if (b == 0) {
        const unsigned v = __builtin_amdgcn_raw_buffer_load_b32(frs, (lane >> 3) * 128 + (lane & 7) * 4, 0, AUX_SYS);
        ok = __all(src_done(v, call, lane) || (lane >> 3) == rank);
      } else {
        ok = all_ge(__builtin_amdgcn_raw_buffer_load_b128(frs2, lane16, 0, AUX_SYS), call);
      }
      if (ok) break;
      if (give_up(myws, call, t0)) {
        if (lane == 0) *(volatile unsigned short*)(p.out[rank] + (size_t)rank * MLOC * NSH * 2) = 0x7fc0;
        break;
      }
    }
  }
}

static void place(unsigned* tab, int h, int& pos, int peer, int col, int n, const int* src, const int* rt, int gi) {
  int k = 0;
  while (k < n) {
    const int room = 32 - (pos & 31), m = n - k < room ? n - k : room;
    for (int j = 0; j < m; ++j, ++pos) {
      const int ph = m > 1 && m <= 4 ? gi % m : 0;
      tab[(h * 32 + (pos & 31)) * 4 + (pos >> 5)] = (unsigned)(peer | col << 3 | rt[k + j] << 6 | src[k + j] << 8 | j << 12 | m << 16 | ph << 20);
    }
    k += m;
  }
}
static void build_table(unsigned* tab, int rank) {
  for (int h = 0; h < 2; ++h) {
    int pos = 0, src[8], rt[8];
    for (int u = 0; u < 32; ++u) {
      for (int j = 0; j < 3; ++j) { src[j] = rank; rt[j] = j; }
      place(tab, h, pos, u < 28 ? (rank + 1 + (u + 4 * h) % 7) & 7 : rank, h * 4 + (u < 28 ? u / 7 : u - 28), 3, src, rt, u + h);
    }
    for (int c = 0; c < 4; ++c) {
      for (int j = 0; j < 8; ++j) { src[j] = (rank + j) & 7; rt[j] = 3; }
      place(tab, h, pos, rank, h * 4 + c, 8, src, rt, 0);
    }
  }
}
}

extern "C" void kernel_launch(int rank, int nranks, ncclComm_t comm, hipStream_t stream,
                              void* const* d_in, const int* in_sizes, int n_in,
                              void* d_out, int out_size, const int* in_dtypes, int out_dtype,
                              void* const* d_out_all, void* const* d_ws_all, size_t ws_bytes) {
  if (nranks != ga::NR || (unsigned)rank >= (unsigned)ga::NR || n_in < 2 || ws_bytes < ga::WS_NEED ||
      in_sizes[0] != ga::MLOC * ga::KDIM || in_sizes[1] != ga::KDIM * ga::NDIM || out_size != 4096 * ga::NSH) {
    fprintf(stderr, "kernel_launch: unexpected problem shape\n");
    return;
  }
  ga::Params p;
  p.X = d_in[0]; p.W = d_in[1]; p.myws = (char*)d_ws_all[rank];
  for (int i = 0; i < ga::NR; ++i) { p.out[i] = (char*)d_out_all[i]; p.ws[i] = (char*)d_ws_all[i]; }
  p.rank = rank; p.pad = 0; p.pad2 = 0; p.pace = ga::PACE;
  ga::build_table((unsigned*)p.tab, rank);
  hipLaunchKernelGGL(ga::gemm_a2a, dim3(ga::NWG), dim3(512), 0, stream, p);
}
```

```cpp
#include <hip/hip_runtime.h>
#include <rccl/rccl.h>
#include <cstdio>
#pragma clang diagnostic ignored "-Wint-to-pointer-cast"

namespace ga {
constexpr int MLOC = 512, KDIM = 4096, NDIM = 16384, NR = 8, NSH = 2048;
constexpr int BM = 128, BN = 256, BK = 64, NS = 3;
constexpr int XB = 16384, STAGE = 3 * XB, STG = NS * STAGE;
constexpr int LDS_BYTES = STG + 16384;
constexpr int NWG = 256, ROUNDS = 4, RSTEPS = 16, TSTEPS = ROUNDS * RSTEPS;
constexpr unsigned X_BYTES = MLOC * KDIM * 2u, W_BYTES = (unsigned)KDIM * NDIM * 2u, OUT_BYTES = 4096u * NSH * 2u;
constexpr unsigned long long DEADLINE = 100000000ull;
constexpr unsigned long long PACE = 0x006064646000ull;
constexpr unsigned WS_DONE = 0;
constexpr unsigned WS_REG = 2048;
constexpr unsigned WS_ARR = 4096;
constexpr unsigned WS_CNT = 8192;
constexpr unsigned WS_TMO = 16384;
constexpr unsigned WS_AGF = 32768;
constexpr unsigned WS_DONE2 = 49152;
constexpr unsigned WS_XF = 131072;
constexpr unsigned WS_XD = 1u << 20, XD_WG = 49152u, XD_BUF = NWG * XD_WG;
constexpr unsigned WS_G = 64u << 20, G_SRC = 1u << 20, G_BUF = NR * G_SRC;
constexpr unsigned WS_NEED = WS_G + 2 * G_BUF;
static_assert(WS_XD + ROUNDS * XD_BUF <= WS_G, "ws");
static_assert(LDS_BYTES <= 160 * 1024, "LDS");

typedef __attribute__((address_space(3))) char lds_char;
typedef short s16x4 __attribute__((ext_vector_type(4)));
typedef short s16x2 __attribute__((ext_vector_type(2)));
typedef __bf16 bf16x8 __attribute__((ext_vector_type(8)));
typedef __bf16 bf16x2 __attribute__((ext_vector_type(2)));
typedef float f32x4 __attribute__((ext_vector_type(4)));
typedef float f32x2 __attribute__((ext_vector_type(2)));
typedef unsigned u32x4 __attribute__((ext_vector_type(4)));
typedef __amdgpu_buffer_rsrc_t rsrc_t;
typedef __attribute__((address_space(3))) s16x4* lp4;
typedef const __attribute__((address_space(3))) bf16x8* lp8;
typedef volatile __attribute__((address_space(3))) u32x4* lpq;
typedef __attribute__((address_space(1))) unsigned* gptr_u;

struct Params {
  const void* X; const void* W; char* myws;
  int rank; int pad; unsigned long long pace, pad2;
  char* out[NR]; char* ws[NR];
  u32x4 tab[64];
};

#define WAIT_V(n) asm volatile("s_waitcnt vmcnt(%0)" ::"n"(n) : "memory")
#define WAIT_L(n) asm volatile("s_waitcnt lgkmcnt(%0)" ::"n"(n) : "memory")
#define BARRIER() do { asm volatile("" ::: "memory"); __builtin_amdgcn_s_barrier(); asm volatile("" ::: "memory"); } while (0)
#define SCHED_FENCE() __builtin_amdgcn_sched_barrier(0)
#define CFENCE() asm volatile("" ::: "memory")
#define AINL __attribute__((always_inline))
#define PIN(x) asm volatile("" : "+v"(x))
#define ST_SYS(ptr, val) __hip_atomic_store((ptr), (val), __ATOMIC_RELAXED, __HIP_MEMORY_SCOPE_SYSTEM)
constexpr int AUX_SYS = 17;
constexpr int AUX_NT = 2;

__device__ __forceinline__ rsrc_t mkrs(const void* p, unsigned bytes) { return __builtin_amdgcn_make_buffer_rsrc((void*)p, 0, bytes, 0x00020000); }

struct Tile { int xsrc;   int xrow, wcol, peer, orow, ocol, rot, blk, ph; };
__device__ __forceinline__ Tile tile_dec(unsigned e, int rank) {
  Tile t;
  const int src = (e >> 8) & 7, rt = (e >> 6) & 3;
  t.peer = e & 7; t.ocol = ((e >> 3) & 7) * BN;
  t.xsrc = src == rank ? -1 : src; t.xrow = src == rank ? rt * BM : 0; t.orow = src * MLOC + rt * BM;
  t.rot = (e >> 12) & 15; t.blk = (e >> 16) & 15; t.ph = (e >> 20) & 15;
  t.wcol = t.peer * NSH + t.ocol;
  return t;
}
#define TILE_OF(r) tile_dec((r) == 0 ? te[0] : (r) == 1 ? te[1] : (r) == 2 ? te[2] : te[3], rank)

__device__ __forceinline__ unsigned pack2(float a, float b) {
  f32x2 v = {a, b};
  return __builtin_bit_cast(unsigned, __builtin_convertvector(v, bf16x2));
}
__device__ __forceinline__ unsigned relu2(unsigned x) {
  s16x2 s = __builtin_bit_cast(s16x2, x), z = {0, 0};
  return __builtin_bit_cast(unsigned, __builtin_elementwise_max(s, z));
}
__device__ __forceinline__ float unpk(unsigned d, int hi) { return __builtin_bit_cast(float, hi ? (d & 0xffff0000u) : (d << 16)); }
__device__ __forceinline__ int vm_outstanding() { int v; asm volatile("s_getreg_b32 %0, hwreg(HW_REG_IB_STS, 0, 4)" : "=s"(v)); return v; }
__device__ __forceinline__ unsigned xcc_id() { unsigned v; asm volatile("s_getreg_b32 %0, hwreg(HW_REG_XCC_ID, 0, 3)" : "=s"(v)); return v; }
__device__ __forceinline__ bool all_ge(const u32x4& f, unsigned v) { return __all(f[0] >= v && f[1] >= v && f[2] >= v && f[3] >= v); }
template <int N> struct IC { static constexpr int v = N; };

__device__ __forceinline__ bool give_up(char* myws, unsigned call, unsigned long long t0) {
  if (__hip_atomic_load((unsigned*)(myws + WS_TMO), __ATOMIC_RELAXED, __HIP_MEMORY_SCOPE_AGENT) == call ||
      __builtin_amdgcn_s_memrealtime() - t0 > DEADLINE) {
    __hip_atomic_store((unsigned*)(myws + WS_TMO), call, __ATOMIC_RELAXED, __HIP_MEMORY_SCOPE_AGENT);
    return true;
  }
  return false;
}
__device__ __forceinline__ bool src_done(unsigned v, unsigned call, int lane) {
  const unsigned long long ge = __ballot((v >> 8) >= call), gt = __ballot((v >> 8) > call);
  unsigned m = (v >> 8) == call ? (v & 255u) : 0u;
  m |= __shfl_xor(m, 1); m |= __shfl_xor(m, 2); m |= __shfl_xor(m, 4);
  const unsigned geb = (unsigned)(ge >> (lane & 56)) & 255u, gtb = (unsigned)(gt >> (lane & 56)) & 255u;
  return gtb != 0u || (m != 0u && (m & ~geb) == 0u);
}

__global__ void __launch_bounds__(512, 2) gemm_a2a(Params p) {
  __shared__ __attribute__((aligned(1024))) char smem[LDS_BYTES];
  lds_char* lds = (lds_char*)smem;
  const unsigned lbase = (unsigned)(size_t)lds;
  const int tid = threadIdx.x, lane = tid & 63;
  const int wid = __builtin_amdgcn_readfirstlane(tid >> 6);
  const int b = blockIdx.x, q = b & 3, rank = p.rank & 7;
  char* myws = p.myws;
  const u32x4 te = p.tab[((b >> 2) & 1) * 32 + ((b >> 3) & 31)];
  auto load_call = [&]() AINL {
    unsigned n;
    asm volatile("s_load_dword %0, %1, 0x0\n\ts_waitcnt lgkmcnt(0)" : "=s"(n) : "s"(myws + WS_CNT + b * 4) : "memory");
    return n + 1u;
  };
  const unsigned lane16 = lane * 16;

  if (__builtin_expect(wid >= 4 && wid < 7, 1)) {
    const bool isx = wid == 4;
    unsigned v[2];
#pragma unroll
    for (int e = 0; e < 2; ++e) {
      const unsigned vx = (unsigned)(lane >> 3) * (KDIM * 2) + (unsigned)(((lane & 7) ^ ((4 * e + (lane >> 4)) & 7)) * 16);
      const int c16 = lane & 15, f = (lane >> 4) | (e << 2);
      const unsigned vw = (unsigned)(lane >> 4) * (NDIM * 2) + (unsigned)((((c16 >> 1) ^ f) * 32) + (c16 & 1) * 16);
      v[e] = isx ? vx : vw;
    }
    rsrc_t rs = mkrs(p.W, W_BYTES);
    unsigned sbase = 0, call = 0;
    int slot = 0, ks = 0, rnd = 0, kb = 0, ki = 0, blk = 1, cur = 1, rot = 0, nleft = 0;
    auto issue = [&]() AINL {
      if (ks == 0) {
        const Tile t = TILE_OF(rnd);
        blk = t.blk; rot = t.rot; cur = t.ph ? t.ph : blk;
        kb = 0; nleft = cur; ki = rot < cur ? rot : 0;
        if (isx) {
          rs = t.xsrc < 0 ? mkrs(p.X, X_BYTES) : mkrs(myws + WS_G + (call & 1) * G_BUF + t.xsrc * G_SRC, G_SRC);
          sbase = (unsigned)t.xrow * (KDIM * 2) + (unsigned)q * 2048u;
        } else {
          sbase = (unsigned)(q * 1024) * (NDIM * 2) + (unsigned)(t.wcol + (wid - 5) * 128) * 2u;
        }
      }
      const int kse = kb + ki;
      if (++ki == cur) ki = 0;
      if (--nleft == 0) { kb += cur; cur = RSTEPS - kb < blk ? RSTEPS - kb : blk; nleft = cur; ki = rot < cur ? rot : 0; }
      lds_char* base = lds + slot * STAGE + (wid - 4) * XB;
      if (isx) {
        const unsigned s = sbase + (unsigned)kse * (BK * 2);
        if (rnd == ROUNDS - 1) {
#pragma unroll
          for (int i = 0; i < 16; ++i)
            __builtin_amdgcn_raw_ptr_buffer_load_lds(rs, base + i * 1024, 16, v[i & 1],
                                                     s + (unsigned)(32 * (((i >> 2) + q) & 3) + 8 * (i & 3)) * (KDIM * 2), 0, AUX_SYS);
        } else {
#pragma unroll
          for (int i = 0; i < 16; ++i)
            __builtin_amdgcn_raw_ptr_buffer_load_lds(rs, base + i * 1024, 16, v[i & 1],
                                                     s + (unsigned)(32 * (((i >> 2) + q) & 3) + 8 * (i & 3)) * (KDIM * 2), 0, 0);
        }
      } else {
        const unsigned s = sbase + (unsigned)kse * (BK * NDIM * 2);
#pragma unroll
        for (int i = 0; i < 16; ++i)
          __builtin_amdgcn_raw_ptr_buffer_load_lds(rs, base + i * 1024, 16, v[(i >> 1) & 1], s + i * 4 * (NDIM * 2), 0, 0);
      }
      slot = (slot + 1 == NS) ? 0 : slot + 1;
      if (++ks == RSTEPS) { ks = 0; ++rnd; }
    };
#pragma unroll 1
    for (int s = 0; s < NS; ++s) {
      issue();
      if (s == 0) { WAIT_V(0); BARRIER(); }
    }
    if (isx) call = load_call();
    unsigned tnext = (unsigned)__builtin_amdgcn_s_memrealtime();
#pragma unroll 1
    for (int t = 0; t < TSTEPS - NS; ++t) {
      WAIT_V(16);
      if (isx) {
        tnext += (unsigned)(p.pace >> (t & 56)) & 255u;
        unsigned now = (unsigned)__builtin_amdgcn_s_memrealtime();
        while ((int)(now - tnext) < 0) now = (unsigned)__builtin_amdgcn_s_memrealtime();
        tnext = now;
      }
      BARRIER();
      issue();
    }
    WAIT_V(16); BARRIER();
    WAIT_V(0); BARRIER();
    BARRIER();
    return;
  }

  if (wid == 7) {
    int t = -1;
    const unsigned call = load_call();
    auto step = [&]() AINL { if (t < TSTEPS - 1) { BARRIER(); ++t; } };
    const unsigned st_voff = (unsigned)(lane & 7) * (NSH * 2) + (unsigned)(((lane >> 4) & 1) * 2 + (lane >> 5)) * 16u + (unsigned)(lane & 8) * 8u;
    rsrc_t rxd = mkrs(myws + WS_XD, (ROUNDS - 1) * XD_BUF), rxf = mkrs(myws + WS_XF + b * 64, 48);
    const unsigned goff = (unsigned)b * 4096u;
    const bool flane = lane < NR && lane != rank;
    size_t fptr = (size_t)(p.ws[lane & 7] + (rank * NWG + b) * 4);
    u32x4 d[4], ow[16], in[8][3], o, oprev, ob, agf;
    unsigned xfv = 0;
    int agn = 0;
    auto ag_store = [&](const u32x4& dd, int j) AINL {
      const int peer = (rank + 1 + (agn + b) % 7) & 7;
      rsrc_t rd = mkrs(p.ws[peer] + WS_G + (call & 1) * G_BUF + rank * G_SRC, agn < 28 ? G_SRC : 0u);
      __builtin_amdgcn_raw_buffer_store_b128(dd, rd, lane16, goff + j * 1024, AUX_SYS);
      ++agn;
    };
    auto xf_load = [&]() AINL { CFENCE(); xfv = __builtin_amdgcn_raw_buffer_load_b32(rxf, lane < 12 ? lane * 4 : 0, 0, AUX_SYS); CFENCE(); };
    auto xf_check = [&](unsigned seq) AINL {
      if (__builtin_expect(!__all(xfv >= seq), 0)) {
        const unsigned long long t0 = __builtin_amdgcn_s_memrealtime();
        for (;;) {
          xf_load();
          if (__all(xfv >= seq)) break;
          if (give_up(myws, call, t0)) break;
        }
      }
      CFENCE();
    };
    auto in_load = [&](int r, int i) AINL {
#pragma unroll
      for (int k = 0; k < 3; ++k)
        in[i & 7][k] = __builtin_amdgcn_raw_buffer_load_b128(rxd, lane16, r * XD_BUF + (unsigned)b * XD_WG + k * 16384u + i * 1024u, AUX_SYS);
    };
    auto ow_load = [&]() AINL {
#pragma unroll
      for (int i = 0; i < 16; ++i) ow[i] = *(lpq)(lbase + STG + i * 1024 + lane16);
      WAIT_L(0);
    };
    const unsigned seq0 = (call - 1u) * ROUNDS + 1u;
    {
      rsrc_t rx = mkrs(p.X, X_BYTES);
#pragma unroll
      for (int j = 0; j < 4; ++j) d[j] = __builtin_amdgcn_raw_buffer_load_b128(rx, lane16, 3u * BM * KDIM * 2 + goff + j * 1024, 0);
      BARRIER();
      if (lane == 0) ST_SYS((unsigned*)(myws + WS_REG) + b, (call << 4) | xcc_id());
      PIN(fptr);
      WAIT_V(0);
#pragma unroll
      for (int j = 0; j < 3; ++j) {
#pragma unroll 1
        for (int k = 0; k < (j < 2 ? 7 : 5); ++k) { ag_store(d[j], j); step(); }
      }
      xf_load(); ag_store(d[2], 2); step();
      xf_check(seq0); in_load(0, 0); in_load(0, 1); ag_store(d[2], 2); step();
      in_load(0, 2); in_load(0, 3); ag_store(d[3], 3); ow_load(); step();
    }
    const unsigned xcc = xcc_id();
    rsrc_t rreg = mkrs(myws + WS_REG, NWG * 4), rarr = mkrs(myws + WS_ARR, NWG * 4);
    const int agsrc = TILE_OF(ROUNDS - 1).xsrc;
#pragma unroll 1
    for (int r = 0; r < ROUNDS - 1; ++r) {
      const Tile tl = TILE_OF(r);
      const bool more = r < ROUNDS - 2;
      rsrc_t rd = mkrs(p.out[tl.peer], more ? 0u : OUT_BYTES), rdn = mkrs(p.out[tl.peer], more ? OUT_BYTES : 0u);
      auto ostore = [&](const u32x4& dd, unsigned so_) AINL {
        __builtin_amdgcn_raw_buffer_store_b128(dd, rd, st_voff, so_, AUX_SYS);
        __builtin_amdgcn_raw_buffer_store_b128(dd, rdn, st_voff, so_, AUX_NT);
      };
      const unsigned so = (unsigned)(tl.orow + 32 * q) * (NSH * 2) + (unsigned)tl.ocol * 2u;
      const bool agchk = r == 1 && agsrc >= 0;
      rsrc_t raf = mkrs(myws + WS_AGF + (agsrc & 7) * (NWG * 4), agchk ? NWG * 4 : 0);
#pragma unroll
      for (int i = 0; i < 16; ++i) {
        if (i == 0 && r == 2) {
          WAIT_V(0);
          if (lane == 0) ST_SYS((unsigned*)(myws + WS_ARR) + b, call);
        }
        if (i == 13 && r == 0) {
          WAIT_V(0);
          if (flane) ST_SYS((gptr_u)(fptr + WS_AGF), call);
        }
        if (i == 13) xf_load();
        if (i == 6) { agf = __builtin_amdgcn_raw_buffer_load_b128(raf, lane16, 0, AUX_SYS); CFENCE(); }
        {
          float v[8];
#pragma unroll
          for (int w = 0; w < 8; ++w) {
            float a_ = unpk(ow[i][w >> 1], w & 1); PIN(a_);
#pragma unroll
            for (int k = 0; k < 3; ++k) { float u_ = unpk(in[i & 7][k][w >> 1], w & 1); PIN(u_); a_ += u_; PIN(a_); }
            v[w] = a_;
          }
          unsigned d0 = pack2(v[0], v[1]); PIN(d0); unsigned d1 = pack2(v[2], v[3]); PIN(d1);
          unsigned e0 = pack2(v[4], v[5]); PIN(e0); unsigned e1 = pack2(v[6], v[7]); PIN(e1);
          d0 = relu2(d0); PIN(d0); d1 = relu2(d1); PIN(d1); e0 = relu2(e0); PIN(e0); e1 = relu2(e1); PIN(e1);
          auto s0 = __builtin_amdgcn_permlane16_swap(d0, e0, false, false);
          auto s1 = __builtin_amdgcn_permlane16_swap(d1, e1, false, false);
          o = (u32x4){s0[0], s1[0], s0[1], s1[1]};
          PIN(o);
        }
        if (i == 7) {
          if (__builtin_expect(agchk && !all_ge(agf, call), 0)) {
            const unsigned long long t0 = __builtin_amdgcn_s_memrealtime();
            for (;;) {
              CFENCE();
              agf = __builtin_amdgcn_raw_buffer_load_b128(raf, lane16, 0, AUX_SYS);
              if (all_ge(agf, call)) break;
              if (give_up(myws, call, t0)) break;
            }
          }
          CFENCE();
        }
        if (i == 14) xf_check(more ? seq0 + r + 1u : 0u);
        if (i < 12) in_load(r, i + 4);
        else if (i >= 14) { in_load(r + 1, 2 * i - 28); in_load(r + 1, 2 * i - 27); }
        CFENCE();
        if (i & 1) {
          u32x4 oa;
#pragma unroll
          for (int k = 0; k < 4; ++k) {
            oa[k] = __builtin_amdgcn_update_dpp(oprev[k], o[k], 0x128  , 0xf, 0xc, false); PIN(oa);
            ob[k] = __builtin_amdgcn_update_dpp(o[k], oprev[k], 0x128, 0xf, 0x3, false); PIN(ob);
          }
          ostore(oa, so + (unsigned)((i >> 1) & 1) * (16 * NSH * 2) + (i >> 2) * 128u);
          if (i == 15) ostore(ob, so + (unsigned)(16 + 8) * (NSH * 2) + 3 * 128u);
        } else {
          oprev = o; PIN(oprev);
          if (i > 0) ostore(ob, so + (unsigned)((((i - 1) >> 1) & 1) * 16 + 8) * (NSH * 2) + ((i - 1) >> 2) * 128u);
        }
        if (i < 12 && !(i & 1)) ag_store(d[3], 3);
        if (i == 15 && more) ow_load();
        SCHED_FENCE(); asm volatile("s_nop 1" ::: "memory"); SCHED_FENCE();
        if (more || (i & 1)) step();
      }
    }
    u32x4 rg = __builtin_amdgcn_raw_buffer_load_b128(rreg, lane16, 0, AUX_SYS);
    u32x4 ar = __builtin_amdgcn_raw_buffer_load_b128(rarr, lane16, 0, AUX_SYS);
    while (t < TSTEPS - 1 && vm_outstanding() > 0) step();
    WAIT_V(0);
    if (flane) ST_SYS((gptr_u)(fptr + WS_DONE2), call);
    const unsigned long long t0 = __builtin_amdgcn_s_memrealtime();
    bool lead = true;
    while (!__all((rg[0] >> 4) == call && (rg[1] >> 4) == call && (rg[2] >> 4) == call && (rg[3] >> 4) == call)) {
      if (give_up(myws, call, t0)) { lead = false; break; }
      CFENCE();
      rg = __builtin_amdgcn_raw_buffer_load_b128(rreg, lane16, 0, AUX_SYS);
    }
    bool lower = false;
#pragma unroll
    for (int k = 0; k < 4; ++k) lower |= (rg[k] & 7u) == xcc && lane * 4 + k < b;
    if (lead && !__any(lower)) {
      unsigned mask = 0;
#pragma unroll
      for (unsigned x = 0; x < 8; ++x)
        if (__any((rg[0] & 7u) == x || (rg[1] & 7u) == x || (rg[2] & 7u) == x || (rg[3] & 7u) == x)) mask |= 1u << x;
      for (;;) {
        bool ok = true;
#pragma unroll
        for (int k = 0; k < 4; ++k) ok &= (rg[k] & 7u) != xcc || ar[k] >= call;
        if (__all(ok)) break;
        step();
        if (give_up(myws, call, t0)) { lead = false; break; }
        CFENCE();
        ar = __builtin_amdgcn_raw_buffer_load_b128(rarr, lane16, 0, AUX_SYS);
      }
      if (lead) {
        asm volatile("buffer_wbl2 sc0 sc1" ::: "memory");
        while (t < TSTEPS - 1 && vm_outstanding() > 0) step();
        WAIT_V(0);
        if (flane) ST_SYS((gptr_u)(fptr - (rank * NWG + b) * 4 + WS_DONE + rank * 128 + xcc * 4), (call << 8) | mask);
      }
    }
    while (t < TSTEPS - 1) step();
    return;
  }

  const unsigned call = load_call();
  const int c = lane & 15, g = lane >> 4;
  unsigned ax0 = lbase + c * 128 + ((g ^ (c >> 1)) * 16), ax1 = ax0 ^ 64u, aw[4];
  {
    const int qq = c >> 2, pp = c & 3;
#pragma unroll
    for (int jn = 0; jn < 4; ++jn)
      aw[jn] = lbase + XB * (1 + (wid >> 1)) + (8 * g + qq) * 256 + (((4 * (wid & 1) + jn) ^ (qq | ((g & 1) << 2))) * 32) + 8 * pp;
  }
  f32x4 acc[8][4];
  bf16x8 xf[2][8], wc[2][4], wn[2][4];
  u32x4 pk;
  s16x4 wlo;
#define RD_WL(KK, jn) wlo = __builtin_amdgcn_ds_read_tr16_b64_v4i16((lp4)(aw[jn] + (KK) * 8192))
#define RD_WH(KK, jn) wn[KK][jn] = __builtin_bit_cast(bf16x8, __builtin_shufflevector(wlo, (s16x4)__builtin_amdgcn_ds_read_tr16_b64_v4i16((lp4)(aw[jn] + (KK) * 8192 + 1024)), 0, 1, 2, 3, 4, 5, 6, 7))
#define RD_X0(G) xf[0][G] = *(lp8)(ax0 + (((G) + 2) & 7) * 2048)
#define RD_X1(G) xf[1][G] = *(lp8)(ax1 + (((G) + 2) & 7) * 2048)
#define MFMA(ACC, A, B) asm volatile("v_mfma_f32_16x16x32_bf16 %0, %1, %2, %0" : "+v"(ACC) : "v"(A), "v"(B))
#define MFMA0(ACC, A, B) asm volatile("v_mfma_f32_16x16x32_bf16 %0, %1, %2, 0" : "+v"(ACC) : "v"(A), "v"(B))
#define MFMAI(ACC, A, B) asm volatile("v_mfma_f32_16x16x32_bf16 %0, %1, %2, 0" : "=&v"(ACC) : "v"(A), "v"(B))
#define SLOT(G, S, FILL) do { \
    constexpr int mi_ = ((G) + 2) & 7, jn_ = ((G) & 1) ? 3 - ((S) >> 1) : ((S) >> 1), kk_ = (((S) >> 1) ^ (S)) & 1; \
    if constexpr (decltype(KIND_)::v == 2 && !((S) & 1)) MFMAI(acc[mi_][jn_], wc[kk_][jn_], xf[kk_][G]); \
    else if constexpr (Z && !((S) & 1)) MFMA0(acc[mi_][jn_], wc[kk_][jn_], xf[kk_][G]); else MFMA(acc[mi_][jn_], wc[kk_][jn_], xf[kk_][G]); \
    SCHED_FENCE(); FILL; extra(KIND_, IC<(G) * 8 + (S)>{}); SCHED_FENCE(); } while (0)
#define GROUP(G, F0, F1, F2, F3, F4, F5, F6, F7) do { \
    SLOT(G, 0, F0); SLOT(G, 1, F1); SLOT(G, 2, F2); SLOT(G, 3, F3); SLOT(G, 4, F4); SLOT(G, 5, F5); SLOT(G, 6, F6); SLOT(G, 7, F7); } while (0)
#define NOP_ (void)0
#define STPAD() do { SCHED_FENCE(); asm volatile("s_nop 1"); SCHED_FENCE(); } while (0)

  const int gb = b & ~3;
  const rsrc_t rxd = mkrs(myws + WS_XD, ROUNDS * XD_BUF);
  rsrc_t rxf = mkrs(myws + WS_XF, 0);
  unsigned xd_buf = 0, seq = 0;
  const unsigned stg_addr = lbase + STG + wid * 4096 + lane16;
  auto opaque_l16 = [&]() AINL { unsigned l = lane16; PIN(l); return l; };
  auto pack_part = [&](auto N_) AINL -> u32x4 {
    constexpr int n = decltype(N_)::v, mi = n >> 1, h = n & 1;
    const f32x4 v0 = acc[mi][2 * h], v1 = acc[mi][2 * h + 1];
    return (u32x4){pack2(v0[0], v0[1]), pack2(v0[2], v0[3]), pack2(v1[0], v1[1]), pack2(v1[2], v1[3])};
  };
  auto store_part = [&](auto N_, u32x4 dd) AINL {
    constexpr int n = decltype(N_)::v, u = n >> 2;
    const unsigned so = xd_buf + (unsigned)(gb + ((q + u) & 3)) * XD_WG + (unsigned)((u - 1) * 4 + wid) * 4096u + (n & 3) * 1024u;
    __builtin_amdgcn_raw_buffer_store_b128(dd, rxd, lane16, so, AUX_SYS);
    SCHED_FENCE(); asm volatile("s_nop 0");
  };
  auto stage_own = [&](auto J_) AINL { *(lpq)(stg_addr + decltype(J_)::v * 1024) = pack_part(J_); };
  auto send_flags = [&]() AINL {
    WAIT_V(0);
    const unsigned l = opaque_l16() >> 4;
    unsigned vo = (((q + l) & 3) * 16 + ((l - 1) & 3) * 4) * 4;
    PIN(vo);
    vo = l - 1u < 3u ? vo : 0x80000000u;
    __builtin_amdgcn_raw_buffer_store_b32(seq, rxf, vo, (unsigned)(gb * 16 + wid) * 4u, AUX_SYS);
  };

  int slot = 0;
  auto extra = [&](auto KIND_, auto SL_) AINL {
    constexpr int KIND = decltype(KIND_)::v, SL = decltype(SL_)::v;
#define PS(sl, n) if constexpr (SL == sl) { pk = pack_part(IC<n>{}); PIN(pk); } if constexpr (SL == sl + 1) store_part(IC<n>{}, pk);
    if constexpr (KIND == 5) { PS(13, 4) PS(21, 5) PS(29, 6) PS(37, 7) PS(45, 8) PS(53, 9) PS(61, 10) }
    if constexpr (KIND == 1) {
      PS(5, 11) PS(13, 12) PS(21, 13) PS(29, 14) PS(36, 15)
      if constexpr (SL == 1) stage_own(IC<0>{});
      if constexpr (SL == 9) stage_own(IC<1>{});
      if constexpr (SL == 17) stage_own(IC<2>{});
      if constexpr (SL == 25) stage_own(IC<3>{});
    }
    if constexpr (KIND == 3 && SL == 22) send_flags();
    if constexpr (KIND == 6) {
      PS(13, 4) PS(16, 5) PS(21, 6) PS(24, 7) PS(29, 8) PS(32, 9) PS(37, 10) PS(40, 11) PS(45, 12) PS(48, 13) PS(53, 14) PS(56, 15)
    }
  };
  auto kstep = [&](auto KIND_) AINL {
    constexpr bool Z = decltype(KIND_)::v == 1 || decltype(KIND_)::v == 2;
    slot = (slot + 1 == NS) ? 0 : slot + 1;
    const unsigned dlt = slot == 0 ? (unsigned)(-(NS - 1) * STAGE) : (unsigned)STAGE;
#pragma unroll
    for (int jn = 0; jn < 4; ++jn) { wc[0][jn] = wn[0][jn]; wc[1][jn] = wn[1][jn]; }
    SCHED_FENCE();
    GROUP(0, RD_X0(2), NOP_, NOP_, NOP_, RD_X1(2), NOP_, NOP_, NOP_);
    GROUP(1, RD_X0(3), NOP_, NOP_, NOP_, RD_X1(3), NOP_, NOP_, NOP_);
    GROUP(2, RD_X0(4), NOP_, NOP_, NOP_, RD_X1(4), NOP_, NOP_, aw[0] += dlt; PIN(aw[0]));
    GROUP(3, RD_X0(5), NOP_, RD_X1(5), aw[1] += dlt; PIN(aw[1]), RD_X0(6), NOP_, NOP_, aw[2] += dlt; PIN(aw[2]));
    GROUP(4, RD_X1(6), NOP_, RD_X0(7), RD_X1(7), NOP_, NOP_, aw[3] += dlt; PIN(aw[3]), ax0 += dlt; PIN(ax0));
    SLOT(5, 0, ax1 += dlt; PIN(ax1));
    SLOT(5, 1, NOP_);
    WAIT_L(0);
    BARRIER();
    RD_X0(0);
    SCHED_FENCE();
    SLOT(5, 2, RD_WL(0, 0)); SLOT(5, 3, RD_WH(0, 0)); SLOT(5, 4, RD_X1(0)); SLOT(5, 5, RD_WL(1, 0)); SLOT(5, 6, RD_WH(1, 0)); SLOT(5, 7, RD_WL(1, 1));
    GROUP(6, RD_WH(1, 1), RD_WL(0, 1), RD_WH(0, 1), RD_WL(0, 2), RD_WH(0, 2), RD_WL(1, 2), RD_WH(1, 2), RD_WL(1, 3));
    GROUP(7, RD_WH(1, 3), RD_WL(0, 3), RD_WH(0, 3), RD_X0(1), RD_X1(1), NOP_, NOP_, NOP_);
  };

  BARRIER();
  RD_X0(0); RD_X1(0);
#pragma unroll
  for (int jn = 0; jn < 4; ++jn) { RD_WL(0, jn); RD_WH(0, jn); RD_WL(1, jn); RD_WH(1, jn); }
  RD_X0(1); RD_X1(1);
  SCHED_FENCE();
#define KS0() kstep(IC<0>{})
  kstep(IC<2>{});
#pragma unroll 1
  for (int r = 0; r < ROUNDS; ++r) {
    if (__builtin_expect(r > 0, 1)) {
      kstep(IC<5>{});
      kstep(IC<1>{});
    }
    kstep(IC<3>{});
    KS0(); KS0(); KS0(); KS0(); KS0(); KS0(); KS0(); KS0(); KS0(); KS0(); KS0(); KS0(); KS0();
    rxf = mkrs(myws + WS_XF, NWG * 64);
    xd_buf = r * XD_BUF;
    seq = (call - 1u) * ROUNDS + r + 1u;
  }
  kstep(IC<6>{});
  {
    SCHED_FENCE();
    send_flags();
    {
      const unsigned long long t0 = __builtin_amdgcn_s_memrealtime();
      const unsigned l = opaque_l16();
      for (;;) {
        CFENCE();
        const unsigned f = __builtin_amdgcn_raw_buffer_load_b32(rxf, l < 32u ? l : 32u, (unsigned)(b * 16 + wid) * 4u, AUX_SYS);
        if (__all(f >= seq)) break;
        if (give_up(myws, call, t0)) break;
      }
      CFENCE();
    }
    u32x4 r3[3][4];
#pragma unroll
    for (int s = 0; s < 3; ++s)
#pragma unroll
      for (int j = 0; j < 4; ++j)
        r3[s][j] = __builtin_amdgcn_raw_buffer_load_b128(rxd, lane16, xd_buf + (unsigned)b * XD_WG + (unsigned)(s * 4 + wid) * 4096u + j * 1024u, AUX_SYS);
    const bool bad = __hip_atomic_load((unsigned*)(myws + WS_TMO), __ATOMIC_RELAXED, __HIP_MEMORY_SCOPE_AGENT) == call;
    const Tile tl = TILE_OF(ROUNDS - 1);
    rsrc_t rd = mkrs(p.out[tl.peer], OUT_BYTES);
    const unsigned so = (unsigned)(tl.orow + 32 * q) * (NSH * 2) + (unsigned)tl.ocol * 2u;
    const unsigned l = opaque_l16() >> 4, gg = l >> 4;
    const unsigned vo = (l & 15) * (NSH * 2) + (8 * wid + (gg & 1) * 2 + (gg >> 1)) * 16u;
#pragma unroll
    for (int j = 0; j < 4; ++j) {
      f32x4 v[2] = {acc[j >> 1][2 * (j & 1)], acc[j >> 1][2 * (j & 1) + 1]};
#pragma unroll
      for (int s = 0; s < 3; ++s)
#pragma unroll
        for (int w = 0; w < 8; ++w) v[w >> 2][w & 3] += unpk(r3[s][j][w >> 1], w & 1);
      const unsigned d0 = relu2(pack2(v[0][0], v[0][1])), d1 = relu2(pack2(v[0][2], v[0][3]));
      const unsigned e0 = relu2(pack2(v[1][0], v[1][1])), e1 = relu2(pack2(v[1][2], v[1][3]));
      auto s0 = __builtin_amdgcn_permlane16_swap(d0, e0, false, false);
      auto s1 = __builtin_amdgcn_permlane16_swap(d1, e1, false, false);
      u32x4 o = {s0[0], s1[0], s0[1], s1[1]};
      if (bad) o = (u32x4){0x7fc07fc0u, 0x7fc07fc0u, 0x7fc07fc0u, 0x7fc07fc0u};
      __builtin_amdgcn_raw_buffer_store_b128(o, rd, vo, so + (j >> 1) * (16 * NSH * 2) + (j & 1) * 64u, AUX_SYS);
      STPAD();
    }
  }
  if (wid == 0) {
    if (lane == 0) __hip_atomic_store((unsigned*)(myws + WS_CNT) + b, call, __ATOMIC_RELAXED, __HIP_MEMORY_SCOPE_AGENT);
  } else if (wid == 1 && b < NR) {
    rsrc_t frs = mkrs(myws + WS_DONE, NR * 128), frs2 = mkrs(myws + WS_DONE2 + ((rank + b) & 7) * (NWG * 4), NWG * 4);
    const unsigned long long t0 = __builtin_amdgcn_s_memrealtime();
    for (;;) {
      CFENCE();
      bool ok;
      if (b == 0) {
        const unsigned v = __builtin_amdgcn_raw_buffer_load_b32(frs, (lane >> 3) * 128 + (lane & 7) * 4, 0, AUX_SYS);
        ok = __all(src_done(v, call, lane) || (lane >> 3) == rank);
      } else {
        ok = all_ge(__builtin_amdgcn_raw_buffer_load_b128(frs2, lane16, 0, AUX_SYS), call);
      }
      if (ok) break;
      if (give_up(myws, call, t0)) {
        if (lane == 0) *(volatile unsigned short*)(p.out[rank] + (size_t)rank * MLOC * NSH * 2) = 0x7fc0;
        break;
      }
    }
  }
}

static void place(unsigned* tab, int h, int& pos, int peer, int col, int n, const int* src, const int* rt, int gi) {
  int k = 0;
  while (k < n) {
    const int room = 32 - (pos & 31), m = n - k < room ? n - k : room;
    for (int j = 0; j < m; ++j, ++pos) {
      const int ph = m > 1 && m <= 4 ? gi % m : 0;
      tab[(h * 32 + (pos & 31)) * 4 + (pos >> 5)] = (unsigned)(peer | col << 3 | rt[k + j] << 6 | src[k + j] << 8 | j << 12 | m << 16 | ph << 20);
    }
    k += m;
  }
}
static void build_table(unsigned* tab, int rank) {
  for (int h = 0; h < 2; ++h) {
    int pos = 0, src[8], rt[8];
    for (int u = 0; u < 32; ++u) {
      for (int j = 0; j < 3; ++j) { src[j] = rank; rt[j] = j; }
      place(tab, h, pos, u < 28 ? (rank + 1 + (u + 4 * h) % 7) & 7 : rank, h * 4 + (u < 28 ? u / 7 : u - 28), 3, src, rt, u + h);
    }
    for (int c = 0; c < 4; ++c) {
      for (int j = 0; j < 8; ++j) { src[j] = (rank + j) & 7; rt[j] = 3; }
      place(tab, h, pos, rank, h * 4 + c, 8, src, rt, 0);
    }
  }
}
}

extern "C" void kernel_launch(int rank, int nranks, ncclComm_t comm, hipStream_t stream,
                              void* const* d_in, const int* in_sizes, int n_in,
                              void* d_out, int out_size, const int* in_dtypes, int out_dtype,
                              void* const* d_out_all, void* const* d_ws_all, size_t ws_bytes) {
  if (nranks != ga::NR || (unsigned)rank >= (unsigned)ga::NR || n_in < 2 || ws_bytes < ga::WS_NEED ||
      in_sizes[0] != ga::MLOC * ga::KDIM || in_sizes[1] != ga::KDIM * ga::NDIM || out_size != 4096 * ga::NSH) {
    fprintf(stderr, "kernel_launch: unexpected problem shape\n");
    return;
  }
  ga::Params p;
  p.X = d_in[0]; p.W = d_in[1]; p.myws = (char*)d_ws_all[rank];
  for (int i = 0; i < ga::NR; ++i) { p.out[i] = (char*)d_out_all[i]; p.ws[i] = (char*)d_ws_all[i]; }
  p.rank = rank; p.pad = 0; p.pad2 = 0; p.pace = ga::PACE;
  ga::build_table((unsigned*)p.tab, rank);
  hipLaunchKernelGGL(ga::gemm_a2a, dim3(ga::NWG), dim3(512), 0, stream, p);
}
```
